```python
import numpy as np

kernel_name = "gpu_a2a_gemm_m4096_k4096_n4096_e5m2e5m2_dequant"


ABSMAX_RTOL = 0.05

COMPUTE_DTYPE = "float32"
EPILOGUE = "dequant"
IN_QUANT = None
OUT_QUANT = None
IN_FP8 = "e5m2e5m2"
IN_DTYPES = {"x": "e5m2", "w_mat": "e5m2",
             "scale_x": "float32", "scale_w": "float32"}
WORLD_SIZE = 8

def _snap_fp8(x, mant_bits, max_abs, min_normal):
    a = np.abs(x.astype(np.float64))
    _m, _e = np.frexp(a)
    q = np.where(a >= min_normal, np.ldexp(1.0, _e - 1 - mant_bits),
                 min_normal * 2.0 ** -mant_bits)
    snapped = np.minimum(np.round(a / q) * q, max_abs)
    return (np.sign(x) * snapped).astype(np.float32)

def setup_inputs():

    x = _snap_fp8(np.random.randn(4096, 4096).astype(np.float32), 2, 57344.0, 2.0 ** -14)
    w_mat = _snap_fp8(np.random.randn(4096, 4096).astype(np.float32), 2, 57344.0, 2.0 ** -14)
    scale_x = (0.02 * (1.0 + np.random.rand(1))).astype(np.float32)
    scale_w = (0.02 * (1.0 + np.random.rand(1))).astype(np.float32)
    return {"x": x, "w_mat": w_mat, "scale_x": scale_x, "scale_w": scale_w}

def reference(x, w_mat, scale_x, scale_w):
    acc = (x @ w_mat).astype(np.float32)
    y = (acc * (scale_x * scale_w)).astype(np.float32)
    return y
```

```cpp
#include <hip/hip_runtime.h>
#include <rccl/rccl.h>
#include <cstdio>

#define HIP_CHECK(expr) do { hipError_t _e = (expr); if (_e != hipSuccess) { \
    fprintf(stderr, "HIP error %s at %s:%d\n", hipGetErrorString(_e), __FILE__, __LINE__); \
    return; } } while (0)

typedef unsigned u32x4 __attribute__((ext_vector_type(4)));
typedef int i32x8 __attribute__((ext_vector_type(8)));
typedef int i32x4 __attribute__((ext_vector_type(4)));
typedef int v2i __attribute__((__vector_size__(2 * sizeof(int))));
typedef float f32x4 __attribute__((ext_vector_type(4)));
typedef unsigned long long u64;
#define LDSP __attribute__((address_space(3)))
#define GLBP __attribute__((address_space(1)))

constexpr int NR = 8;
constexpr size_t STAGE_PER = 256u << 10;
constexpr size_t FLAG_OFF = 4u << 20;
constexpr size_t CNT_OFF = FLAG_OFF + (64u << 10);
constexpr size_t TMO_OFF = CNT_OFF + (64u << 10);
constexpr u64 DEADLINE = 20000000ull;
constexpr int LDS_BYTES = 96 * 1024;

struct Params { const unsigned char* x; const unsigned char* w; const float* sx; const float* sw; float* out; unsigned char* ws[NR]; int rank; int pad; };

__global__ __launch_bounds__(256, 1) void a2a_gemm(Params p) {
  extern __shared__ __attribute__((aligned(16))) unsigned char lds[];
  const int b = blockIdx.x, tid = threadIdx.x, lane = tid & 63, w = __builtin_amdgcn_readfirstlane(tid >> 6);
  unsigned char* myws = p.ws[p.rank];
  unsigned* lw = (unsigned*)(lds + 2 * 24576);
  if (tid == 0) lw[0] = __hip_atomic_fetch_add((unsigned*)(myws + CNT_OFF + b * 128), 1u, __ATOMIC_RELAXED, __HIP_MEMORY_SCOPE_AGENT) + 1u;
  const int d = (p.rank + b) & 7, unit = b >> 3;
  u32x4 v0 = *(const GLBP u32x4*)(p.x + (size_t)(d * 512 + unit * 16) * 512 + tid * 16);
  u32x4 v1 = *(const GLBP u32x4*)(p.x + (size_t)(d * 512 + unit * 16) * 512 + 4096 + tid * 16);
  __syncthreads();
  const unsigned call = __builtin_amdgcn_readfirstlane(lw[0]);
  const unsigned par = call & 1u;
  {
    unsigned char* dp = p.ws[d] + (par * 8 + p.rank) * STAGE_PER + unit * 8192 + tid * 16;
    asm volatile("global_store_dwordx4 %0, %1, off sc0 sc1\n\ts_nop 1" :: "v"(dp), "v"(v0) : "memory");
    asm volatile("global_store_dwordx4 %0, %1, off sc0 sc1\n\ts_nop 1" :: "v"(dp + 4096), "v"(v1) : "memory");
    asm volatile("s_waitcnt vmcnt(0)" ::: "memory");
    __syncthreads();
    if (tid == 0) __hip_atomic_store((unsigned*)(p.ws[d] + FLAG_OFF + (p.rank * 32 + unit) * 128), call, __ATOMIC_RELAXED, __HIP_MEMORY_SCOPE_SYSTEM);
  }
  const int xcd = b & 7, idx = b >> 3, mt = idx & 7, nt = xcd * 4 + (idx >> 3);
  unsigned* tmo = (unsigned*)(myws + TMO_OFF);
  f32x4 acc[2][4] = {};
  auto stage = [&](int step, int buf) {
    const int src = (p.rank + (step >> 2)) & 7, ks = step & 3;
    unsigned char* lb = lds + buf * 24576;
#pragma unroll
    for (int i = 0; i < 4; ++i) {
      const int ins = w * 4 + i, row = ins * 8 + (lane >> 3);
      const int f = ((row >> 1) & 3) | (((row >> 5) & 1) << 2);
      const unsigned char* g = p.w + (size_t)(src * 512 + ks * 128 + row) * 4096 + nt * 128 + (((lane & 7) ^ f) << 4);
      __builtin_amdgcn_global_load_lds((const GLBP void*)g, (LDSP void*)(lb + ins * 1024), 16, 0, 0);
    }
#pragma unroll
    for (int i = 0; i < 2; ++i) {
      const int ins = w * 2 + i, row = ins * 8 + (lane >> 3);
      const unsigned char* g = myws + (par * 8 + src) * STAGE_PER + (size_t)(mt * 64 + row) * 512 + ks * 128 + (((lane & 7) ^ (row & 7)) << 4);
      __builtin_amdgcn_global_load_lds((const GLBP void*)g, (LDSP void*)(lb + 16384 + ins * 1024), 16, 0, 17);
    }
  };
  auto waitsrc = [&](int step) {
    if ((step & 3) == 0) {
      if (w == 0) {
        const int src = (p.rank + (step >> 2)) & 7;
        const unsigned* f = (const unsigned*)(myws + FLAG_OFF + (src * 32 + mt * 4 + (lane & 3)) * 128);
        u64 t0 = __builtin_amdgcn_s_memrealtime();
        while (true) {
          unsigned v = 0xffffffffu;
          if (lane < 4) v = __hip_atomic_load(f, __ATOMIC_RELAXED, __HIP_MEMORY_SCOPE_SYSTEM);
          if (__all(v >= call)) break;
          if (__hip_atomic_load(tmo, __ATOMIC_RELAXED, __HIP_MEMORY_SCOPE_AGENT) == call || __builtin_amdgcn_s_memrealtime() - t0 > DEADLINE) {
            __hip_atomic_store(tmo, call, __ATOMIC_RELAXED, __HIP_MEMORY_SCOPE_AGENT); break; }
          __builtin_amdgcn_s_sleep(2);
        }
      }
      __syncthreads();
    }
  };
  waitsrc(0); stage(0, 0);
  const int q = lane >> 4, r = (lane >> 1) & 7, pp = lane & 1, fl = (r >> 1) | ((q & 1) << 2);
  for (int step = 0; step < 32; ++step) {
    asm volatile("s_waitcnt vmcnt(0)" ::: "memory");
    __syncthreads();
    if (step + 1 < 32) { waitsrc(step + 1); stage(step + 1, (step + 1) & 1); }
    unsigned char* lb = lds + (step & 1) * 24576;
    i32x8 wf[2], xf[4];
#pragma unroll
    for (int c = 0; c < 2; ++c)
#pragma unroll
      for (int t = 0; t < 4; ++t) {
        v2i rr = __builtin_amdgcn_ds_read_tr8_b64_v2i32((LDSP v2i*)(lb + (32 * q + 8 * t + r) * 128 + (((2 * w + c) ^ fl) << 4) + 8 * pp));
        wf[c][2 * t] = rr[0]; wf[c][2 * t + 1] = rr[1];
      }
#pragma unroll
    for (int m = 0; m < 4; ++m)
#pragma unroll
      for (int i = 0; i < 2; ++i) {
        const int row = m * 16 + (lane & 15);
        i32x4 rr = *(LDSP i32x4*)(lb + 16384 + row * 128 + (((2 * q + i) ^ (row & 7)) << 4));
        xf[m][4 * i] = rr[0]; xf[m][4 * i + 1] = rr[1]; xf[m][4 * i + 2] = rr[2]; xf[m][4 * i + 3] = rr[3];
      }
#pragma unroll
    for (int c = 0; c < 2; ++c)
#pragma unroll
      for (int m = 0; m < 4; ++m)
        acc[c][m] = __builtin_amdgcn_mfma_scale_f32_16x16x128_f8f6f4(wf[c], xf[m], acc[c][m], 1, 1, 0, 0, 0, 0);
  }
  float s = p.sx[0] * p.sw[0];
  if (__hip_atomic_load(tmo, __ATOMIC_RELAXED, __HIP_MEMORY_SCOPE_AGENT) == call) s = __builtin_nanf("");
#pragma unroll
  for (int c = 0; c < 2; ++c)
#pragma unroll
    for (int m = 0; m < 4; ++m) {
      f32x4 o = acc[c][m] * s;
      *(f32x4*)(p.out + (size_t)(mt * 64 + m * 16 + (lane & 15)) * 4096 + nt * 128 + (2 * w + c) * 16 + q * 4) = o;
    }
}

extern "C" void kernel_launch(int rank, int nranks, ncclComm_t comm,
                              hipStream_t stream,
                              void* const* d_in, const int* in_sizes, int n_in,
                              void* d_out, int out_size,
                              const int* in_dtypes, int out_dtype,
                              void* const* d_out_all,
                              void* const* d_ws_all, size_t ws_bytes) {
  if (nranks != NR || (unsigned)rank >= (unsigned)NR) return;
  static bool init = false;
  if (!init) { HIP_CHECK(hipFuncSetAttribute((const void*)a2a_gemm, hipFuncAttributeMaxDynamicSharedMemorySize, LDS_BYTES)); init = true; }
  Params p{};
  p.x = (const unsigned char*)d_in[0]; p.w = (const unsigned char*)d_in[1];
  p.sx = (const float*)d_in[2]; p.sw = (const float*)d_in[3]; p.out = (float*)d_out;
  for (int i = 0; i < NR; ++i) p.ws[i] = (unsigned char*)d_ws_all[i];
  p.rank = rank; p.pad = 0;
  a2a_gemm<<<256, 256, LDS_BYTES, stream>>>(p);
  HIP_CHECK(hipGetLastError());
}
```

```cpp
#include <hip/hip_runtime.h>
#include <rccl/rccl.h>
#include <cstdio>

#define HIP_CHECK(expr) do { hipError_t _e = (expr); if (_e != hipSuccess) { \
    fprintf(stderr, "HIP error %s at %s:%d\n", hipGetErrorString(_e), __FILE__, __LINE__); \
    return; } } while (0)

typedef unsigned long long u64;
constexpr int NR = 8;
constexpr int LDS_BYTES = 160 * 1024;
constexpr size_t WS_NEED = 14422016;
constexpr size_t STAMP_OFF = 4591616;

struct Params { u64 w, x, ws, out, sx, sw; unsigned rank, pad; u64 peer[NR]; };

__global__ __launch_bounds__(256, 1) void a2a_gemm(Params p) {
  asm volatile(
    "L_top%=:\n"
    "s_load_dwordx16 s[16:31], %[ka], 0x0\n"
    "s_mov_b32 s13, %[bid]\n"
    "s_mul_i32 s38, %[bid], 37\n"
    "s_lshr_b32 s38, s38, 8\n"
    "s_mul_i32 s74, s38, 7\n"
    "s_sub_u32 s35, %[bid], s74\n"
    "s_add_u32 s35, s35, 1\n"
    "s_and_b32 s35, s35, 7\n"
    "s_lshl_b32 s74, s35, 3\n"
    "s_load_dwordx2 s[36:37], %[ka], s74 offset:0x38\n"
    "v_readfirstlane_b32 s12, %[tid]\n"
    "v_and_b32 v12, 63, %[tid]\n"
    "s_lshr_b32 s12, s12, 6\n"
    "s_getpc_b64 s[60:61]\n"
    "L_anchor%=:\n"
    "s_cmp_lt_u32 s13, 24\n"
    "s_cbranch_scc1 L_nocpf%=\n"
    "s_cmp_ge_u32 s13, 40\n"
    "s_cbranch_scc1 L_nocpf%=\n"
    "s_bfe_u32 s74, s13, 0x10005\n"
    "s_lshl2_add_u32 s74, s74, s12\n"
    "s_lshl_b32 s74, s74, 12\n"
    "v_lshl_add_u32 v34, v12, 6, s74\n"
    "v_min_u32 v34, L_end%=-L_anchor%=-4, v34\n"
    "global_load_dword v35, v34, s[60:61]\n"
    "L_nocpf%=:\n"
    "v_lshrrev_b32 v47, 4, v12\n"
    "v_xor_b32 v47, v47, v12\n"
    "v_and_b32 v47, 7, v47\n"
    "s_lshl2_add_u32 s88, s13, s12\n"
    "s_lshl_b32 s77, s88, 7\n"
    "s_add_u32 s75, s77, 4390912\n"
    "v_mov_b32 v46, s75\n"
    "v_mov_b32 v15, 1\n"
    "s_and_b32 s15, s13, 1\n"
    "v_lshrrev_b32 v24, 3, v12\n"
    "v_lshlrev_b32 v33, 9, v24\n"
    "v_lshl_or_b32 v32, v47, 4, v33\n"
    "s_lshl_b32 s51, s12, 7\n"
    "v_or_b32 v32, s51, v32\n"
    "v_xor_b32 v33, 0x1040, v32\n"
    "s_cmp_ge_u32 s12, 2\n"
    "s_cselect_b32 s49, 5, 0\n"
    "s_mul_i32 s39, s38, 10\n"
    "s_mov_b32 s54, 0x200000\n"
    "s_mov_b32 s55, 0x20000\n"
    "s_mov_b32 s42, 0x1000000\n"
    "s_mov_b32 s43, 0x20000\n"
    "s_lshr_b32 s75, s13, 3\n"
    "s_sub_u32 s75, s75, 8\n"
    "s_mul_i32 s76, s75, 43\n"
    "s_lshr_b32 s76, s76, 7\n"
    "s_mul_i32 s74, s76, 3\n"
    "s_sub_u32 s14, s75, s74\n"
    "s_bfe_u32 s74, s13, 0x20003\n"
    "s_lshl1_add_u32 s74, s74, s15\n"
    "s_cmp_lt_u32 s13, 64\n"
    "s_cselect_b32 s76, s74, s76\n"
    "s_cselect_b32 s14, 3, s14\n"
    "s_bfe_u32 s74, s13, 0x20001\n"
    "s_lshr_b32 s75, s76, 1\n"
    "s_xor_b32 s74, s74, s75\n"
    "s_lshl2_add_u32 s34, s76, s74\n"
    "v_lshlrev_b32 v14, 12, v24\n"
    "v_lshl_or_b32 v14, v47, 4, v14\n"
    "s_lshl_b32 s74, s34, 7\n"
    "s_lshl_b32 s75, s15, 20\n"
    "s_add_u32 s74, s74, s75\n"
    "s_lshl_b32 s75, s12, 17\n"
    "s_add_u32 s74, s74, s75\n"
    "v_add_u32 v16, s74, v14\n"
    "v_add_u32 v17, 31744, v16\n"
    "v_xor_b32 v18, 64, v16\n"
    "v_add_u32 v19, 95232, v18\n"
    "v_add_u32 v18, 63488, v18\n"
    "s_lshl_b32 s85, s12, 12\n"
    "s_add_u32 s85, s85, 73728\n"
    "s_mov_b32 m0, s85\n"
    "s_lshl1_add_u32 s98, s14, 1\n"
    "s_lshr_b32 s60, s13, 3\n"
    "s_sub_u32 s60, s60, 8\n"
    "s_lshl1_add_u32 s60, s60, s12\n"
    "v_lshlrev_b32 v36, 7, v12\n"
    "s_mul_i32 s61, s60, 47\n"
    "s_lshr_b32 s61, s61, 9\n"
    "s_mul_i32 s62, s61, 11\n"
    "s_sub_u32 s62, s60, s62\n"
    "s_and_b32 s63, s13, 7\n"
    "s_lshl3_add_u32 s63, s61, s63\n"
    "s_mul_i32 s64, s63, 37\n"
    "s_lshr_b32 s64, s64, 8\n"
    "s_mul_i32 s65, s64, 7\n"
    "s_sub_u32 s65, s63, s65\n"
    "s_add_u32 s59, s64, 20\n"
    "s_mul_i32 s64, s64, 10\n"
    "s_cmp_eq_u32 s62, 10\n"
    "s_cselect_b32 s64, s59, s64\n"
    "s_add_u32 s62, s62, s64\n"
    "s_cmp_lt_u32 s12, 2\n"
    "s_cselect_b32 s63, s63, 99\n"
    "s_lshr_b32 s60, s13, 3\n"
    "s_sub_u32 s60, s60, 20\n"
    "s_lshr_b32 s56, s60, 2\n"
    "s_and_b32 s57, s60, 3\n"
    "s_sub_u32 s61, s56, 1\n"
    "s_mul_i32 s61, s61, 7\n"
    "s_lshl1_add_u32 s56, s56, 2\n"
    "s_sub_u32 s59, s60, 99\n"
    "s_cmp_eq_u32 s12, 3\n"
    "s_cselect_b32 s56, s61, s56\n"
    "s_cselect_b32 s60, s59, s60\n"
    "s_cmp_ge_u32 s12, 2\n"
    "s_cselect_b32 s60, s60, 99\n"
    "s_bfe_u32 s45, s13, 0x20001\n"
    "s_xor_b32 s45, s45, 3\n"
    "s_lshl_b32 s45, s45, 10\n"
    "s_add_u32 s45, s45, 384\n"
    "s_lshl2_add_u32 s57, s15, s57\n"
    "s_lshl_b32 s57, s57, 18\n"
    "s_waitcnt lgkmcnt(0)\n"
    "s_mov_b32 s52, s18\n"
    "s_and_b32 s53, s19, 0xffff\n"
    "s_mov_b32 s40, s16\n"
    "s_and_b32 s41, s17, 0xffff\n"
    "s_cmp_ge_u32 s13, 21\n"
    "s_cbranch_scc1 L_np%=\n"
    "L_yp%=:\n"
    "s_add_u32 s50, s28, s35\n"
    "s_and_b32 s50, s50, 7\n"
    "s_lshl_b32 s50, s50, 5\n"
    "s_add_u32 s50, s50, s39\n"
    "s_mov_b64 exec, 1\n"
    "global_atomic_add v45, v46, v15, s[20:21] sc0\n"
    "s_mov_b64 exec, -1\n"
    "s_add_u32 s51, s50, s49\n"
    "s_lshl_b32 s51, s51, 13\n"
    "s_add_u32 s49, s49, 1\n"
    "buffer_load_dwordx4 v[48:51], v32, s[52:55], s51 offen\n"
    "buffer_load_dwordx4 v[52:55], v33, s[52:55], s51 offen\n"
    "s_add_u32 s51, s50, s49\n"
    "s_lshl_b32 s51, s51, 13\n"
    "s_add_u32 s49, s49, 1\n"
    "buffer_load_dwordx4 v[56:59], v32, s[52:55], s51 offen\n"
    "buffer_load_dwordx4 v[60:63], v33, s[52:55], s51 offen\n"
    "s_branch L_zush2%=\n"
    "L_np%=:\n"
    "s_cmp_lt_u32 s13, 64\n"
    "s_cbranch_scc1 L_help%=\n"
    "s_cmp_lt_u32 s63, 21\n"
    "s_cbranch_scc0 L_nx%=\n"
    "s_add_u32 s65, s65, s28\n"
    "s_add_u32 s65, s65, 1\n"
    "s_and_b32 s65, s65, 7\n"
    "s_lshl_b32 s65, s65, 5\n"
    "s_add_u32 s65, s65, s62\n"
    "s_lshl_b32 s65, s65, 13\n"
    "buffer_load_dword v37, v36, s[52:55], s65 offen\n"
    "L_nx%=:\n"
    "s_cmp_lt_u32 s60, 12\n"
    "s_cbranch_scc0 L_na%=\n"
    "s_add_u32 s56, s56, s28\n"
    "s_and_b32 s56, s56, 7\n"
    "s_lshl_b32 s56, s56, 21\n"
    "s_add_u32 s56, s56, s57\n"
    "v_lshl_add_u32 v36, v12, 12, s45\n"
    "buffer_load_dword v37, v36, s[40:43], s56 offen\n"
    "L_na%=:\n"
    "s_add_u32 s98, s98, s28\n"
    "s_and_b32 s98, s98, 7\n"
    "s_lshl_b32 s98, s98, 21\n"
    "s_add_u32 s99, s98, 0x80000\n"
    "buffer_load_dwordx4 v16, s[40:43], s98 offen offset:0 lds\n"
    "buffer_load_dwordx4 v17, s[40:43], s98 offen offset:1024 lds\n"
    "buffer_load_dwordx4 v18, s[40:43], s98 offen offset:2048 lds\n"
    "buffer_load_dwordx4 v19, s[40:43], s98 offen offset:3072 lds\n"
    "s_add_u32 m0, s85, 16384\n"
    "s_nop 0\n"
    "buffer_load_dwordx4 v16, s[40:43], s99 offen offset:0 lds\n"
    "buffer_load_dwordx4 v17, s[40:43], s99 offen offset:1024 lds\n"
    "buffer_load_dwordx4 v18, s[40:43], s99 offen offset:2048 lds\n"
    "buffer_load_dwordx4 v19, s[40:43], s99 offen offset:3072 lds\n"
    "s_branch L_gemm%=\n"
    "L_help%=:\n"
    "s_cmp_lt_u32 s13, 24\n"
    "s_cbranch_scc1 L_end%=\n"
    "s_cmp_ge_u32 s13, 32\n"
    "s_cbranch_scc1 L_e0%=\n"
    "s_lshr_b32 s47, s13, 3\n"
    "s_sub_u32 s47, s47, 3\n"
    "s_lshl2_add_u32 s47, s47, s12\n"
    "s_sub_u32 s60, s47, 4\n"
    "s_cmp_ge_u32 s60, 12\n"
    "s_cbranch_scc1 L_h0%=\n"
    "s_bfe_u32 s44, s13, 0x20001\n"
    "s_xor_b32 s45, s44, 3\n"
    "s_lshl_b32 s45, s45, 10\n"
    "s_add_u32 s45, s45, 384\n"
    "v_lshl_add_u32 v36, v12, 12, s45\n"
    "s_lshr_b32 s56, s60, 2\n"
    "s_lshl1_add_u32 s56, s56, s28\n"
    "s_add_u32 s56, s56, 2\n"
    "s_and_b32 s56, s56, 7\n"
    "s_lshl_b32 s56, s56, 21\n"
    "s_and_b32 s57, s60, 3\n"
    "s_lshl2_add_u32 s57, s15, s57\n"
    "s_lshl_b32 s57, s57, 18\n"
    "s_add_u32 s56, s56, s57\n"
    "buffer_load_dword v37, v36, s[40:43], s56 offen\n"
    "s_branch L_e0%=\n"
    "L_h0%=:\n"
    "v_lshrrev_b32 v36, 1, v12\n"
    "v_lshlrev_b32 v36, 9, v36\n"
    "v_and_b32 v34, 1, v12\n"
    "v_lshl_or_b32 v36, v34, 7, v36\n"
    "s_min_u32 s60, s60, s47\n"
    "s_lshl_b32 s58, s28, 18\n"
    "s_lshl_b32 s57, s60, 14\n"
    "s_add_u32 s57, s57, s58\n"
    "s_lshl_b32 s58, s15, 8\n"
    "s_add_u32 s57, s57, s58\n"
    "buffer_load_dword v37, v36, s[52:55], s57 offen\n"
    "s_cmp_ge_u32 s13, 32\n"
    "s_cbranch_scc1 L_e0%=\n"
    "s_add_u32 s57, s57, 0x10000\n"
    "buffer_load_dword v37, v36, s[52:55], s57 offen\n"
    "s_add_u32 s57, s57, 0x10000\n"
    "buffer_load_dword v37, v36, s[52:55], s57 offen\n"
    "s_add_u32 s57, s57, 0x10000\n"
    "buffer_load_dword v37, v36, s[52:55], s57 offen\n"
    "s_endpgm\n"
    "L_e0%=:\n"
    "s_lshl_b32 s98, s28, 21\n"
    "s_and_b32 s63, s34, 4\n"
    "s_lshl_b32 s63, s63, 8\n"
    "s_sub_u32 s63, 512, s63\n"
    "buffer_load_dwordx4 v16, s[40:43], s98 offen offset:0 lds\n"
    "buffer_load_dwordx4 v17, s[40:43], s98 offen offset:1024 lds\n"
    "buffer_load_dwordx4 v18, s[40:43], s98 offen offset:2048 lds\n"
    "buffer_load_dwordx4 v19, s[40:43], s98 offen offset:3072 lds\n"
    "s_add_u32 m0, s85, 16384\n"
    "v_add_u32 v48, s63, v16\n"
    "v_add_u32 v49, s63, v17\n"
    "v_add_u32 v50, s63, v18\n"
    "v_add_u32 v51, s63, v19\n"
    "buffer_load_dwordx4 v48, s[40:43], s98 offen offset:0 lds\n"
    "buffer_load_dwordx4 v49, s[40:43], s98 offen offset:1024 lds\n"
    "buffer_load_dwordx4 v50, s[40:43], s98 offen offset:2048 lds\n"
    "buffer_load_dwordx4 v51, s[40:43], s98 offen offset:3072 lds\n"
    "s_branch L_gemm%=\n"
    "L_zush2%=:\n"
    "s_add_u32 s51, s50, s49\n"
    "s_lshl_b32 s51, s51, 13\n"
    "s_add_u32 s49, s49, 1\n"
    "buffer_load_dwordx4 v[64:67], v32, s[52:55], s51 offen\n"
    "buffer_load_dwordx4 v[68:71], v33, s[52:55], s51 offen\n"
    "s_lshr_b32 s64, s12, 1\n"
    "s_lshl1_add_u32 s51, s39, s64\n"
    "s_lshl_b32 s51, s51, 15\n"
    "s_and_b32 s59, s12, 1\n"
    "s_add_u32 s59, s59, 14\n"
    "s_lshl_b32 s56, s35, 1\n"
    "s_sub_u32 s59, s59, s56\n"
    "s_lshl_b32 s65, s59, 11\n"
    "s_lshl_b32 s56, s38, 2\n"
    "s_sub_u32 s56, s59, s56\n"
    "s_add_u32 s57, s56, 12\n"
    "s_min_u32 s56, s56, s57\n"
    "s_cmp_lt_u32 s59, 12\n"
    "s_cselect_b32 s59, s56, s59\n"
    "s_lshl_b32 s59, s59, 11\n"
    "s_add_u32 s51, s51, s59\n"
    "s_add_u32 s44, s36, s51\n"
    "s_addc_u32 s45, s37, 0\n"
    "s_mul_i32 s51, s28, 5\n"
    "s_add_u32 s51, s51, s38\n"
    "s_lshl_b32 s51, s51, 7\n"
    "s_add_u32 s51, s51, 4194304\n"
    "s_add_u32 s59, s77, 4526080\n"
    "s_add_u32 s66, s51, 384\n"
    "s_cmp_eq_u32 s12, 0\n"
    "s_cselect_b32 s51, s51, s59\n"
    "s_cselect_b64 s[46:47], s[36:37], s[20:21]\n"
    "s_add_u32 s46, s46, s51\n"
    "s_addc_u32 s47, s47, 0\n"
    "s_add_u32 s51, s38, 30\n"
    "s_lshl1_add_u32 s51, s51, s64\n"
    "s_lshl_b32 s51, s51, 15\n"
    "s_add_u32 s65, s65, s51\n"
    "s_lshr_b32 s51, s38, 1\n"
    "s_or_b32 s51, s51, s12\n"
    "s_cmp_eq_u32 s51, 0\n"
    "s_cselect_b32 s66, s66, s59\n"
    "s_cselect_b64 s[62:63], s[36:37], s[20:21]\n"
    "s_add_u32 s62, s62, s66\n"
    "s_addc_u32 s63, s63, 0\n"
    "v_lshlrev_b32 v18, 4, v12\n"
    "v_lshlrev_b32 v19, 13, v12\n"
    "s_cmp_eq_u32 s12, 0\n"
    "s_cselect_b32 s68, 3, 1\n"
    "s_mov_b32 s69, 0\n"
    "s_mov_b32 s71, 0\n"
    "s_cmp_lt_u32 s38, 2\n"
    "s_cselect_b32 s70, s68, 1\n"
    "s_waitcnt vmcnt(4)\n"
    "v_readfirstlane_b32 s51, v45\n"
    "s_add_u32 s51, s51, 1\n"
    "s_lshl_b32 s58, s51, 4\n"
    "s_and_b32 s67, s51, 1\n"
    "s_lshl_b32 s67, s67, 21\n"
    "s_add_u32 s44, s44, s67\n"
    "s_addc_u32 s45, s45, 0\n"
    "s_memrealtime s[72:73]\n"
    "global_store_dwordx4 v18, v[48:51], s[44:45] offset:0 sc0 sc1\n"
    "global_store_dwordx4 v18, v[52:55], s[44:45] offset:1024 sc0 sc1\n"
    "s_add_u32 s44, s44, 0x10000\n"
    "s_addc_u32 s45, s45, 0\n"
    "s_add_u32 s51, s50, s49\n"
    "s_lshl_b32 s51, s51, 13\n"
    "s_add_u32 s49, s49, 1\n"
    "buffer_load_dwordx4 v[72:75], v32, s[52:55], s51 offen\n"
    "buffer_load_dwordx4 v[76:79], v33, s[52:55], s51 offen\n"
    "s_waitcnt vmcnt(6)\n"
    "L_pc2%=:\n"
    "s_memrealtime s[74:75]\n"
    "s_waitcnt lgkmcnt(0)\n"
    "s_sub_u32 s74, s74, s72\n"
    "s_cmp_lt_i32 s74, 14\n"
    "s_cbranch_scc1 L_pc2%=\n"
    "global_store_dwordx4 v18, v[56:59], s[44:45] offset:0 sc0 sc1\n"
    "L_pc3%=:\n"
    "s_memrealtime s[74:75]\n"
    "s_waitcnt lgkmcnt(0)\n"
    "s_sub_u32 s74, s74, s72\n"
    "s_cmp_lt_i32 s74, 36\n"
    "s_cbranch_scc1 L_pc3%=\n"
    "global_store_dwordx4 v18, v[60:63], s[44:45] offset:1024 sc0 sc1\n"
    "s_add_u32 s44, s44, 0x10000\n"
    "s_addc_u32 s45, s45, 0\n"
    "s_add_u32 s51, s50, s49\n"
    "s_lshl_b32 s51, s51, 13\n"
    "s_add_u32 s49, s49, 1\n"
    "buffer_load_dwordx4 v[80:83], v32, s[52:55], s51 offen\n"
    "buffer_load_dwordx4 v[84:87], v33, s[52:55], s51 offen\n"
    "s_cmp_eq_u32 s49, 10\n"
    "s_cselect_b32 s49, 0, s49\n"
    "s_waitcnt vmcnt(7)\n"
    "L_pc4%=:\n"
    "s_memrealtime s[74:75]\n"
    "s_waitcnt lgkmcnt(0)\n"
    "s_sub_u32 s74, s74, s72\n"
    "s_cmp_lt_i32 s74, 58\n"
    "s_cbranch_scc1 L_pc4%=\n"
    "global_store_dwordx4 v18, v[64:67], s[44:45] offset:0 sc0 sc1\n"
    "s_waitcnt vmcnt(7)\n"
    "s_barrier\n"
    "s_add_u32 s51, s58, 1\n"
    "v_mov_b32 v20, s51\n"
    "s_mov_b64 exec, s[68:69]\n"
    "global_store_dword v19, v20, s[46:47] sc0 sc1\n"
    "s_mov_b64 exec, -1\n"
    "L_pc5%=:\n"
    "s_memrealtime s[74:75]\n"
    "s_waitcnt lgkmcnt(0)\n"
    "s_sub_u32 s74, s74, s72\n"
    "s_cmp_lt_i32 s74, 80\n"
    "s_cbranch_scc1 L_pc5%=\n"
    "global_store_dwordx4 v18, v[68:71], s[44:45] offset:1024 sc0 sc1\n"
    "s_add_u32 s44, s44, 0x10000\n"
    "s_addc_u32 s45, s45, 0\n"
    "s_add_u32 s51, s50, s49\n"
    "s_lshl_b32 s51, s51, 13\n"
    "s_add_u32 s49, s49, 1\n"
    "buffer_load_dwordx4 v[88:91], v32, s[52:55], s51 offen\n"
    "buffer_load_dwordx4 v[92:95], v33, s[52:55], s51 offen\n"
    "s_waitcnt vmcnt(8)\n"
    "L_pc6%=:\n"
    "s_memrealtime s[74:75]\n"
    "s_waitcnt lgkmcnt(0)\n"
    "s_sub_u32 s74, s74, s72\n"
    "s_cmp_lt_i32 s74, 102\n"
    "s_cbranch_scc1 L_pc6%=\n"
    "global_store_dwordx4 v18, v[72:75], s[44:45] offset:0 sc0 sc1\n"
    "s_waitcnt vmcnt(8)\n"
    "s_barrier\n"
    "s_add_u32 s51, s58, 2\n"
    "v_mov_b32 v20, s51\n"
    "s_mov_b64 exec, s[68:69]\n"
    "global_store_dword v19, v20, s[46:47] sc0 sc1\n"
    "s_mov_b64 exec, -1\n"
    "L_pc7%=:\n"
    "s_memrealtime s[74:75]\n"
    "s_waitcnt lgkmcnt(0)\n"
    "s_sub_u32 s74, s74, s72\n"
    "s_cmp_lt_i32 s74, 124\n"
    "s_cbranch_scc1 L_pc7%=\n"
    "global_store_dwordx4 v18, v[76:79], s[44:45] offset:1024 sc0 sc1\n"
    "s_add_u32 s44, s44, 0x10000\n"
    "s_addc_u32 s45, s45, 0\n"
    "s_or_b32 s51, s50, 31\n"
    "s_sub_u32 s51, s51, 1\n"
    "s_add_u32 s51, s51, s38\n"
    "s_lshl_b32 s51, s51, 13\n"
    "buffer_load_dwordx4 v[96:99], v32, s[52:55], s51 offen\n"
    "buffer_load_dwordx4 v[100:103], v33, s[52:55], s51 offen\n"
    "s_waitcnt vmcnt(9)\n"
    "L_pc8%=:\n"
    "s_memrealtime s[74:75]\n"
    "s_waitcnt lgkmcnt(0)\n"
    "s_sub_u32 s74, s74, s72\n"
    "s_cmp_lt_i32 s74, 146\n"
    "s_cbranch_scc1 L_pc8%=\n"
    "global_store_dwordx4 v18, v[80:83], s[44:45] offset:0 sc0 sc1\n"
    "s_waitcnt vmcnt(8)\n"
    "s_barrier\n"
    "s_add_u32 s51, s58, 3\n"
    "v_mov_b32 v20, s51\n"
    "s_mov_b64 exec, s[68:69]\n"
    "global_store_dword v19, v20, s[46:47] sc0 sc1\n"
    "s_mov_b64 exec, -1\n"
    "L_pc9%=:\n"
    "s_memrealtime s[74:75]\n"
    "s_waitcnt lgkmcnt(0)\n"
    "s_sub_u32 s74, s74, s72\n"
    "s_cmp_lt_i32 s74, 168\n"
    "s_cbranch_scc1 L_pc9%=\n"
    "global_store_dwordx4 v18, v[84:87], s[44:45] offset:1024 sc0 sc1\n"
    "s_add_u32 s44, s44, 0x10000\n"
    "s_addc_u32 s45, s45, 0\n"
    "s_add_u32 s51, s50, s49\n"
    "s_lshl_b32 s51, s51, 13\n"
    "s_add_u32 s49, s49, 1\n"
    "buffer_load_dwordx4 v[104:107], v32, s[52:55], s51 offen\n"
    "buffer_load_dwordx4 v[108:111], v33, s[52:55], s51 offen\n"
    "s_waitcnt vmcnt(9)\n"
    "L_pc10%=:\n"
    "s_memrealtime s[74:75]\n"
    "s_waitcnt lgkmcnt(0)\n"
    "s_sub_u32 s74, s74, s72\n"
    "s_cmp_lt_i32 s74, 190\n"
    "s_cbranch_scc1 L_pc10%=\n"
    "global_store_dwordx4 v18, v[88:91], s[44:45] offset:0 sc0 sc1\n"
    "s_waitcnt vmcnt(8)\n"
    "s_barrier\n"
    "s_add_u32 s51, s58, 4\n"
    "v_mov_b32 v20, s51\n"
    "s_mov_b64 exec, s[68:69]\n"
    "global_store_dword v19, v20, s[46:47] sc0 sc1\n"
    "s_mov_b64 exec, -1\n"
    "L_pc11%=:\n"
    "s_memrealtime s[74:75]\n"
    "s_waitcnt lgkmcnt(0)\n"
    "s_sub_u32 s74, s74, s72\n"
    "s_cmp_lt_i32 s74, 212\n"
    "s_cbranch_scc1 L_pc11%=\n"
    "global_store_dwordx4 v18, v[92:95], s[44:45] offset:1024 sc0 sc1\n"
    "s_add_u32 s44, s44, 0x10000\n"
    "s_addc_u32 s45, s45, 0\n"
    "s_add_u32 s51, s50, s49\n"
    "s_lshl_b32 s51, s51, 13\n"
    "s_add_u32 s49, s49, 1\n"
    "buffer_load_dwordx4 v[112:115], v32, s[52:55], s51 offen\n"
    "buffer_load_dwordx4 v[116:119], v33, s[52:55], s51 offen\n"
    "s_waitcnt vmcnt(9)\n"
    "L_pc12%=:\n"
    "s_memrealtime s[74:75]\n"
    "s_waitcnt lgkmcnt(0)\n"
    "s_sub_u32 s74, s74, s72\n"
    "s_cmp_lt_i32 s74, 234\n"
    "s_cbranch_scc1 L_pc12%=\n"
    "s_add_u32 s65, s65, s67\n"
    "s_lshl_b32 s51, s77, 4\n"
    "s_add_u32 s51, s51, 4853760\n"
    "s_cmp_lt_u32 s38, 2\n"
    "s_cselect_b32 s65, s65, s51\n"
    "s_cselect_b64 s[60:61], s[36:37], s[20:21]\n"
    "s_add_u32 s60, s60, s65\n"
    "s_addc_u32 s61, s61, 0\n"
    "global_store_dwordx4 v18, v[96:99], s[60:61] offset:0 sc0 sc1\n"
    "s_waitcnt vmcnt(8)\n"
    "s_barrier\n"
    "s_add_u32 s51, s58, 5\n"
    "v_mov_b32 v20, s51\n"
    "s_mov_b64 exec, s[68:69]\n"
    "global_store_dword v19, v20, s[46:47] sc0 sc1\n"
    "s_mov_b64 exec, -1\n"
    "L_pc13%=:\n"
    "s_memrealtime s[74:75]\n"
    "s_waitcnt lgkmcnt(0)\n"
    "s_sub_u32 s74, s74, s72\n"
    "s_cmp_lt_i32 s74, 256\n"
    "s_cbranch_scc1 L_pc13%=\n"
    "global_store_dwordx4 v18, v[100:103], s[60:61] offset:1024 sc0 sc1\n"
    "s_add_u32 s51, s50, s49\n"
    "s_lshl_b32 s51, s51, 13\n"
    "s_add_u32 s49, s49, 1\n"
    "buffer_load_dwordx4 v[120:123], v32, s[52:55], s51 offen\n"
    "buffer_load_dwordx4 v[124:127], v33, s[52:55], s51 offen\n"
    "s_waitcnt vmcnt(9)\n"
    "L_pc14%=:\n"
    "s_memrealtime s[74:75]\n"
    "s_waitcnt lgkmcnt(0)\n"
    "s_sub_u32 s74, s74, s72\n"
    "s_cmp_lt_i32 s74, 278\n"
    "s_cbranch_scc1 L_pc14%=\n"
    "global_store_dwordx4 v18, v[104:107], s[44:45] offset:0 sc0 sc1\n"
    "s_waitcnt vmcnt(8)\n"
    "s_barrier\n"
    "s_add_u32 s51, s58, 6\n"
    "v_mov_b32 v20, s51\n"
    "s_mov_b64 exec, s[68:69]\n"
    "global_store_dword v19, v20, s[46:47] sc0 sc1\n"
    "s_mov_b64 exec, -1\n"
    "L_pc15%=:\n"
    "s_memrealtime s[74:75]\n"
    "s_waitcnt lgkmcnt(0)\n"
    "s_sub_u32 s74, s74, s72\n"
    "s_cmp_lt_i32 s74, 300\n"
    "s_cbranch_scc1 L_pc15%=\n"
    "global_store_dwordx4 v18, v[108:111], s[44:45] offset:1024 sc0 sc1\n"
    "s_add_u32 s44, s44, 0x10000\n"
    "s_addc_u32 s45, s45, 0\n"
    "s_add_u32 s51, s50, s49\n"
    "s_lshl_b32 s51, s51, 13\n"
    "s_add_u32 s49, s49, 1\n"
    "buffer_load_dwordx4 v[128:131], v32, s[52:55], s51 offen\n"
    "buffer_load_dwordx4 v[132:135], v33, s[52:55], s51 offen\n"
    "s_waitcnt vmcnt(9)\n"
    "L_pc16%=:\n"
    "s_memrealtime s[74:75]\n"
    "s_waitcnt lgkmcnt(0)\n"
    "s_sub_u32 s74, s74, s72\n"
    "s_cmp_lt_i32 s74, 322\n"
    "s_cbranch_scc1 L_pc16%=\n"
    "global_store_dwordx4 v18, v[112:115], s[44:45] offset:0 sc0 sc1\n"
    "s_waitcnt vmcnt(8)\n"
    "s_barrier\n"
    "s_add_u32 s51, s58, 1\n"
    "v_mov_b32 v20, s51\n"
    "s_mov_b64 exec, s[70:71]\n"
    "global_store_dword v19, v20, s[62:63] sc0 sc1\n"
    "s_mov_b64 exec, -1\n"
    "L_pc17%=:\n"
    "s_memrealtime s[74:75]\n"
    "s_waitcnt lgkmcnt(0)\n"
    "s_sub_u32 s74, s74, s72\n"
    "s_cmp_lt_i32 s74, 344\n"
    "s_cbranch_scc1 L_pc17%=\n"
    "global_store_dwordx4 v18, v[116:119], s[44:45] offset:1024 sc0 sc1\n"
    "s_add_u32 s44, s44, 0x10000\n"
    "s_addc_u32 s45, s45, 0\n"
    "s_waitcnt vmcnt(7)\n"
    "L_pc18%=:\n"
    "s_memrealtime s[74:75]\n"
    "s_waitcnt lgkmcnt(0)\n"
    "s_sub_u32 s74, s74, s72\n"
    "s_cmp_lt_i32 s74, 366\n"
    "s_cbranch_scc1 L_pc18%=\n"
    "global_store_dwordx4 v18, v[120:123], s[44:45] offset:0 sc0 sc1\n"
    "s_waitcnt vmcnt(6)\n"
    "s_barrier\n"
    "s_add_u32 s51, s58, 7\n"
    "v_mov_b32 v20, s51\n"
    "s_mov_b64 exec, s[68:69]\n"
    "global_store_dword v19, v20, s[46:47] sc0 sc1\n"
    "s_mov_b64 exec, -1\n"
    "L_pc19%=:\n"
    "s_memrealtime s[74:75]\n"
    "s_waitcnt lgkmcnt(0)\n"
    "s_sub_u32 s74, s74, s72\n"
    "s_cmp_lt_i32 s74, 388\n"
    "s_cbranch_scc1 L_pc19%=\n"
    "global_store_dwordx4 v18, v[124:127], s[44:45] offset:1024 sc0 sc1\n"
    "s_add_u32 s44, s44, 0x10000\n"
    "s_addc_u32 s45, s45, 0\n"
    "s_waitcnt vmcnt(5)\n"
    "L_pc20%=:\n"
    "s_memrealtime s[74:75]\n"
    "s_waitcnt lgkmcnt(0)\n"
    "s_sub_u32 s74, s74, s72\n"
    "s_cmp_lt_i32 s74, 410\n"
    "s_cbranch_scc1 L_pc20%=\n"
    "global_store_dwordx4 v18, v[128:131], s[44:45] offset:0 sc0 sc1\n"
    "s_waitcnt vmcnt(4)\n"
    "s_barrier\n"
    "s_add_u32 s51, s58, 8\n"
    "v_mov_b32 v20, s51\n"
    "s_mov_b64 exec, s[68:69]\n"
    "global_store_dword v19, v20, s[46:47] sc0 sc1\n"
    "s_mov_b64 exec, -1\n"
    "L_pc21%=:\n"
    "s_memrealtime s[74:75]\n"
    "s_waitcnt lgkmcnt(0)\n"
    "s_sub_u32 s74, s74, s72\n"
    "s_cmp_lt_i32 s74, 432\n"
    "s_cbranch_scc1 L_pc21%=\n"
    "global_store_dwordx4 v18, v[132:135], s[44:45] offset:1024 sc0 sc1\n"
    "s_add_u32 s44, s44, 0x10000\n"
    "s_addc_u32 s45, s45, 0\n"
    "s_waitcnt vmcnt(3)\n"
    "s_barrier\n"
    "s_add_u32 s51, s58, 9\n"
    "v_mov_b32 v20, s51\n"
    "s_mov_b64 exec, s[68:69]\n"
    "global_store_dword v19, v20, s[46:47] sc0 sc1\n"
    "s_mov_b64 exec, -1\n"
    "s_waitcnt vmcnt(1)\n"
    "s_barrier\n"
    "s_add_u32 s51, s58, 10\n"
    "v_mov_b32 v20, s51\n"
    "s_mov_b64 exec, s[68:69]\n"
    "global_store_dword v19, v20, s[46:47] sc0 sc1\n"
    "s_mov_b64 exec, -1\n"
    "s_endpgm\n"
    "L_gemm%=:\n"
    "s_lshl_b32 s74, s14, 2\n"
    "v_add_u32 v13, s74, v12\n"
    "v_subrev_u32 v14, 12, v13\n"
    "v_cmp_gt_u32 vcc, 12, v12\n"
    "v_min_u32 v13, v13, v14\n"
    "s_nop 0\n"
    "v_cndmask_b32 v13, v12, v13, vcc\n"
    "v_add_u32 v13, 2, v13\n"
    "v_and_b32 v13, 15, v13\n"
    "v_lshrrev_b32 v44, 1, v13\n"
    "v_and_b32 v13, 1, v13\n"
    "v_add_u32 v44, s28, v44\n"
    "v_and_b32 v44, 7, v44\n"
    "v_lshl_or_b32 v44, v44, 2, v13\n"
    "v_lshlrev_b32 v44, 19, v44\n"
    "s_mov_b64 exec, 1\n"
    "global_atomic_add v45, v46, v15, s[20:21] sc0\n"
    "s_mov_b64 exec, -1\n"
    "s_load_dword s83, s[24:25], 0x0\n"
    "s_load_dword s84, s[26:27], 0x0\n"
    "v_readlane_b32 s98, v44, 4\n"
    "s_add_u32 s75, s77, 4259840\n"
    "s_add_u32 s56, s20, s75\n"
    "s_addc_u32 s57, s21, 0\n"
    "s_and_b32 s57, s57, 0xffff\n"
    "s_mov_b32 s58, 4\n"
    "s_mov_b32 s59, 0x20000\n"
    "s_cmp_eq_u32 s12, 0\n"
    "s_cselect_b32 s58, 4, 0\n"
    "s_sub_u32 s88, s88, 128\n"
    "s_mul_i32 s75, s88, 10240\n"
    "s_add_u32 s75, s75, 5246976\n"
    "s_add_u32 s60, s20, s75\n"
    "s_addc_u32 s61, s21, 0\n"
    "s_xor_b32 s88, s88, 4\n"
    "s_mul_i32 s75, s88, 10240\n"
    "s_add_u32 s75, s75, 5242880\n"
    "s_add_u32 s48, s20, s75\n"
    "s_addc_u32 s49, s21, 0\n"
    "s_and_b32 s49, s49, 0xffff\n"
    "s_mov_b32 s50, 10240\n"
    "s_mov_b32 s51, 0x20000\n"
    "s_mul_i32 s86, s14, 10\n"
    "s_mul_i32 s73, s15, 5\n"
    "s_xor_b32 s74, s34, 4\n"
    "s_cmp_eq_u32 s14, 3\n"
    "s_cselect_b32 s65, 2, 10\n"
    "s_cselect_b32 s73, 5, s73\n"
    "s_cselect_b32 s66, s74, s34\n"
    "s_and_b32 s74, s12, 1\n"
    "s_mul_i32 s74, s74, 4160\n"
    "v_lshlrev_b32 v24, 9, v24\n"
    "v_lshl_or_b32 v24, v47, 4, v24\n"
    "v_xor_b32 v24, s74, v24\n"
    "s_lshr_b32 s74, s12, 1\n"
    "s_lshl1_add_u32 s74, s15, s74\n"
    "s_lshl_b32 s74, s74, 7\n"
    "v_or_b32 v24, s74, v24\n"
    "v_lshrrev_b32 v47, 4, v12\n"
    "v_bfe_u32 v13, v12, 1, 3\n"
    "v_lshl_or_b32 v14, v47, 4, v13\n"
    "v_lshlrev_b32 v20, 7, v14\n"
    "v_bfe_u32 v13, v12, 2, 3\n"
    "s_lshl_b32 s74, s12, 1\n"
    "v_xor_b32 v13, s74, v13\n"
    "v_lshl_or_b32 v20, v13, 4, v20\n"
    "v_and_b32 v13, 1, v12\n"
    "v_lshl_or_b32 v20, v13, 3, v20\n"
    "v_add_u32 v20, 73728, v20\n"
    "v_xor_b32 v21, 16, v20\n"
    "v_add_u32 v22, 65536, v20\n"
    "v_add_u32 v23, 65536, v21\n"
    "v_and_b32 v13, 15, v12\n"
    "v_bfe_u32 v14, v12, 1, 3\n"
    "v_xor_b32 v14, v47, v14\n"
    "v_lshlrev_b32 v26, 7, v13\n"
    "v_lshl_or_b32 v26, v14, 4, v26\n"
    "v_xor_b32 v27, 64, v26\n"
    "v_add_u32 v28, 28672, v26\n"
    "v_add_u32 v29, 28672, v27\n"
    "v_lshlrev_b32 v40, 4, v12\n"
    "v_add_u32 v39, 0x2000, v40\n"
    "v_lshrrev_b32 v13, 5, v12\n"
    "v_and_b32 v14, 31, v12\n"
    "v_lshlrev_b32 v13, 17, v13\n"
    "v_lshl_or_b32 v13, v14, 2, v13\n"
    "s_lshl2_add_u32 s74, s66, s12\n"
    "s_lshl_b32 s74, s74, 7\n"
    "v_add_u32 v64, s74, v13\n"
    "v_add_u32 v65, 16384, v64\n"
    "v_add_u32 v66, 32768, v64\n"
    "v_add_u32 v67, 49152, v64\n"
    "v_add_u32 v68, 65536, v64\n"
    "v_add_u32 v69, 81920, v64\n"
    "v_add_u32 v70, 98304, v64\n"
    "v_add_u32 v71, 114688, v64\n"
    "s_mul_i32 s72, s12, 7168\n"
    "v_add_u32 v25, s72, v40\n"
    "s_mul_i32 s75, s12, 7\n"
    "s_lshr_b32 s75, s75, 1\n"
    "s_lshl_b32 s74, s14, 2\n"
    "v_min_u32 v47, 2, v47\n"
    "v_min_u32 v13, 1, v47\n"
    "v_add3_u32 v47, v47, v13, s75\n"
    "v_add_u32 v13, s74, v47\n"
    "v_subrev_u32 v14, 12, v13\n"
    "v_cmp_gt_u32 vcc, 12, v47\n"
    "v_min_u32 v13, v13, v14\n"
    "s_nop 0\n"
    "v_cndmask_b32 v13, v47, v13, vcc\n"
    "v_lshrrev_b32 v13, 1, v13\n"
    "v_add3_u32 v13, s28, v13, 1\n"
    "v_and_b32 v13, 7, v13\n"
    "v_mul_u32_u24 v13, 5, v13\n"
    "v_add_u32 v13, s14, v13\n"
    "s_and_b32 s74, s13, 1\n"
    "s_lshl_b32 s74, s74, 13\n"
    "s_add_u32 s74, s74, 4194304\n"
    "v_lshl_add_u32 v41, v13, 7, s74\n"
    "v_mov_b32 v38, v41\n"
    "s_add_u32 s74, s88, 128\n"
    "s_andn2_b32 s74, s74, 3\n"
    "s_lshl_b32 s74, s74, 7\n"
    "s_add_u32 s74, s74, 4259840\n"
    "s_cmp_eq_u32 s12, 0\n"
    "s_cselect_b32 s74, s74, 4521984\n"
    "s_mov_b32 s71, s74\n"
    "s_mov_b32 s62, 0\n"
    "s_mov_b32 exec_lo, 0\n"
    "s_mov_b32 exec_hi, 0x00ff0000\n"
    "v_mov_b32 v41, s74\n"
    "s_mov_b32 exec_hi, 0xff000000\n"
    "v_mov_b32 v41, 4521984\n"
    "s_mov_b64 exec, -1\n"
    "s_mov_b32 s95, 0\n"
    "s_lshl_b32 s97, s65, 16\n"
    "s_mov_b32 s93, 0\n"
    "v_add_u32 v30, 0x10000, v26\n"
    "v_add_u32 v31, 0x10000, v27\n"
    "s_sub_u32 s74, s86, s73\n"
    "s_add_u32 s74, s74, 5\n"
    "s_lshl_b32 s74, s74, 18\n"
    "s_add_u32 s22, s22, s74\n"
    "s_addc_u32 s23, s23, 0\n"
    "v_subrev_u32 v13, 16, v12\n"
    "v_add_u32 v13, s73, v13\n"
    "v_subrev_u32 v14, s65, v13\n"
    "v_min_u32 v13, v13, v14\n"
    "s_lshl_b32 s74, s28, 5\n"
    "s_add_u32 s74, s74, s86\n"
    "v_add_lshl_u32 v13, s74, v13, 13\n"
    "s_mov_b32 exec_lo, 0xffff0000\n"
    "v_mov_b32 v44, v13\n"
    "s_mov_b32 exec_lo, -1\n"
    "s_lshl_b32 s87, s12, 10\n"
    "s_add_u32 s87, s87, 28672\n"
    "s_cmp_eq_u32 s14, 3\n"
    "s_cbranch_scc1 L_ext%=\n"
    "s_add_u32 m0, s85, 32768\n"
    "v_readlane_b32 s99, v44, 8\n"
    "buffer_load_dwordx4 v16, s[40:43], s98 offen offset:0 lds\n"
    "buffer_load_dwordx4 v17, s[40:43], s98 offen offset:1024 lds\n"
    "buffer_load_dwordx4 v18, s[40:43], s98 offen offset:2048 lds\n"
    "buffer_load_dwordx4 v19, s[40:43], s98 offen offset:3072 lds\n"
    "s_waitcnt lgkmcnt(0)\n"
    "v_mov_b32 v43, s83\n"
    "v_mul_f32 v43, s84, v43\n"
    "s_waitcnt vmcnt(9)\n"
    "s_barrier\n"
    "s_add_u32 m0, s85, 49152\n"
    "ds_read_b64_tr_b8 a[0:1], v20 offset:0\n"
    "v_readlane_b32 s98, v44, 2\n"
    "ds_read_b64_tr_b8 a[2:3], v20 offset:1024\n"
    "buffer_load_dwordx4 v16, s[40:43], s99 offen offset:0 lds\n"
    "ds_read_b64_tr_b8 a[4:5], v20 offset:8192\n"
    "buffer_load_dwordx4 v17, s[40:43], s99 offen offset:1024 lds\n"
    "ds_read_b64_tr_b8 a[6:7], v20 offset:9216\n"
    "ds_read_b64_tr_b8 a[8:9], v21 offset:0\n"
    "ds_read_b64_tr_b8 a[10:11], v21 offset:1024\n"
    "ds_read_b64_tr_b8 a[12:13], v21 offset:8192\n"
    "ds_read_b64_tr_b8 a[14:15], v21 offset:9216\n"
    "buffer_load_dwordx4 v18, s[40:43], s99 offen offset:2048 lds\n"
    "buffer_load_dwordx4 v19, s[40:43], s99 offen offset:3072 lds\n"
    "s_waitcnt vmcnt(9)\n"
    "s_barrier\n"
    "s_add_u32 m0, s85, 65536\n"
    "ds_read_b64_tr_b8 a[16:17], v20 offset:16384\n"
    "v_readlane_b32 s99, v44, 5\n"
    "ds_read_b64_tr_b8 a[18:19], v20 offset:17408\n"
    "buffer_load_dwordx4 v16, s[40:43], s98 offen offset:0 lds\n"
    "ds_read_b64_tr_b8 a[20:21], v20 offset:24576\n"
    "buffer_load_dwordx4 v17, s[40:43], s98 offen offset:1024 lds\n"
    "ds_read_b64_tr_b8 a[22:23], v20 offset:25600\n"
    "ds_read_b64_tr_b8 a[24:25], v21 offset:16384\n"
    "ds_read_b64_tr_b8 a[26:27], v21 offset:17408\n"
    "ds_read_b64_tr_b8 a[28:29], v21 offset:24576\n"
    "s_waitcnt lgkmcnt(11)\n"
    "ds_read_b64_tr_b8 a[30:31], v21 offset:25600\n"
    "buffer_load_dwordx4 v18, s[40:43], s98 offen offset:2048 lds\n"
    "buffer_load_dwordx4 v19, s[40:43], s98 offen offset:3072 lds\n"
    "s_waitcnt vmcnt(8) lgkmcnt(8)\n"
    "s_barrier\n"
    "s_add_u32 m0, s85, 0\n"
    "ds_read_b64_tr_b8 a[64:65], v20 offset:32768\n"
    "v_readlane_b32 s98, v44, 9\n"
    "ds_read_b64_tr_b8 a[66:67], v20 offset:33792\n"
    "buffer_load_dwordx4 v16, s[40:43], s99 offen offset:0 lds\n"
    "ds_read_b64_tr_b8 a[68:69], v20 offset:40960\n"
    "buffer_load_dwordx4 v17, s[40:43], s99 offen offset:1024 lds\n"
    "ds_read_b64_tr_b8 a[70:71], v20 offset:41984\n"
    "ds_read_b64_tr_b8 a[72:73], v21 offset:32768\n"
    "ds_read_b64_tr_b8 a[74:75], v21 offset:33792\n"
    "ds_read_b64_tr_b8 a[76:77], v21 offset:40960\n"
    "s_waitcnt lgkmcnt(11)\n"
    "ds_read_b64_tr_b8 a[78:79], v21 offset:41984\n"
    "buffer_load_dwordx4 v18, s[40:43], s99 offen offset:2048 lds\n"
    "buffer_load_dwordx4 v19, s[40:43], s99 offen offset:3072 lds\n"
    "s_waitcnt vmcnt(8) lgkmcnt(8)\n"
    "s_barrier\n"
    "s_add_u32 m0, s85, 16384\n"
    "ds_read_b64_tr_b8 a[128:129], v20 offset:49152\n"
    "v_readlane_b32 s99, v44, 3\n"
    "ds_read_b64_tr_b8 a[130:131], v20 offset:50176\n"
    "buffer_load_dwordx4 v16, s[40:43], s98 offen offset:0 lds\n"
    "ds_read_b64_tr_b8 a[132:133], v20 offset:57344\n"
    "buffer_load_dwordx4 v17, s[40:43], s98 offen offset:1024 lds\n"
    "ds_read_b64_tr_b8 a[134:135], v20 offset:58368\n"
    "ds_read_b64_tr_b8 a[136:137], v21 offset:49152\n"
    "ds_read_b64_tr_b8 a[138:139], v21 offset:50176\n"
    "ds_read_b64_tr_b8 a[140:141], v21 offset:57344\n"
    "s_waitcnt lgkmcnt(11)\n"
    "ds_read_b64_tr_b8 a[142:143], v21 offset:58368\n"
    "buffer_load_dwordx4 v18, s[40:43], s98 offen offset:2048 lds\n"
    "buffer_load_dwordx4 v19, s[40:43], s98 offen offset:3072 lds\n"
    "v_readfirstlane_b32 s74, v45\n"
    "s_add_u32 s74, s74, 1\n"
    "s_lshl_b32 s70, s74, 4\n"
    "s_and_b32 s74, s74, 1\n"
    "s_lshl_b32 s74, s74, 21\n"
    "s_lshl1_add_u32 s75, s86, s15\n"
    "s_lshl_b32 s75, s75, 15\n"
    "s_add_u32 s74, s74, s75\n"
    "s_add_u32 s44, s20, s74\n"
    "s_addc_u32 s45, s21, 0\n"
    "s_and_b32 s45, s45, 0xffff\n"
    "s_mov_b32 s46, 0\n"
    "s_mov_b32 s47, 0x20000\n"
    "s_mov_b32 s64, s70\n"
    "s_mov_b32 s67, s70\n"
    "s_add_u32 s69, s70, 1\n"
    "s_cmp_eq_u32 s12, 0\n"
    "s_cselect_b32 s94, -1, 0\n"
    "s_and_b32 s89, s69, s94\n"
    "v_mov_b32 v46, s70\n"
    "s_mov_b32 s63, s70\n"
    "s_waitcnt vmcnt(8) lgkmcnt(8)\n"
    "s_barrier\n"
    "s_add_u32 m0, s85, 32768\n"
    "ds_read_b64_tr_b8 a[32:33], v22 offset:0\n"
    "v_readlane_b32 s98, v44, 14\n"
    "ds_read_b64_tr_b8 a[34:35], v22 offset:1024\n"
    "buffer_load_dwordx4 v16, s[40:43], s99 offen offset:0 lds\n"
    "ds_read_b64_tr_b8 a[36:37], v22 offset:8192\n"
    "buffer_load_dwordx4 v17, s[40:43], s99 offen offset:1024 lds\n"
    "ds_read_b64_tr_b8 a[38:39], v22 offset:9216\n"
    "ds_read_b64_tr_b8 a[40:41], v23 offset:0\n"
    "ds_read_b64_tr_b8 a[42:43], v23 offset:1024\n"
    "ds_read_b64_tr_b8 a[44:45], v23 offset:8192\n"
    "s_waitcnt lgkmcnt(11)\n"
    "ds_read_b64_tr_b8 a[46:47], v23 offset:9216\n"
    "buffer_load_dwordx4 v18, s[40:43], s99 offen offset:2048 lds\n"
    "buffer_load_dwordx4 v19, s[40:43], s99 offen offset:3072 lds\n"
    "s_waitcnt vmcnt(8) lgkmcnt(8)\n"
    "s_barrier\n"
    "s_add_u32 m0, s85, 49152\n"
    "ds_read_b64_tr_b8 a[80:81], v20 offset:0\n"
    "v_readlane_b32 s99, v44, 15\n"
    "ds_read_b64_tr_b8 a[82:83], v20 offset:1024\n"
    "buffer_load_dwordx4 v16, s[40:43], s98 offen offset:0 lds\n"
    "ds_read_b64_tr_b8 a[84:85], v20 offset:8192\n"
    "buffer_load_dwordx4 v17, s[40:43], s98 offen offset:1024 lds\n"
    "ds_read_b64_tr_b8 a[86:87], v20 offset:9216\n"
    "ds_read_b64_tr_b8 a[88:89], v21 offset:0\n"
    "ds_read_b64_tr_b8 a[90:91], v21 offset:1024\n"
    "ds_read_b64_tr_b8 a[92:93], v21 offset:8192\n"
    "s_waitcnt lgkmcnt(11)\n"
    "ds_read_b64_tr_b8 a[94:95], v21 offset:9216\n"
    "buffer_load_dwordx4 v18, s[40:43], s98 offen offset:2048 lds\n"
    "buffer_load_dwordx4 v19, s[40:43], s98 offen offset:3072 lds\n"
    "s_waitcnt vmcnt(8) lgkmcnt(8)\n"
    "s_barrier\n"
    "s_add_u32 m0, s85, 65536\n"
    "ds_read_b64_tr_b8 a[144:145], v20 offset:16384\n"
    "v_readlane_b32 s98, v44, 6\n"
    "ds_read_b64_tr_b8 a[146:147], v20 offset:17408\n"
    "buffer_load_dwordx4 v16, s[40:43], s99 offen offset:0 lds\n"
    "ds_read_b64_tr_b8 a[148:149], v20 offset:24576\n"
    "buffer_load_dwordx4 v17, s[40:43], s99 offen offset:1024 lds\n"
    "ds_read_b64_tr_b8 a[150:151], v20 offset:25600\n"
    "ds_read_b64_tr_b8 a[152:153], v21 offset:16384\n"
    "ds_read_b64_tr_b8 a[154:155], v21 offset:17408\n"
    "ds_read_b64_tr_b8 a[156:157], v21 offset:24576\n"
    "s_waitcnt lgkmcnt(11)\n"
    "ds_read_b64_tr_b8 a[158:159], v21 offset:25600\n"
    "buffer_load_dwordx4 v18, s[40:43], s99 offen offset:2048 lds\n"
    "buffer_load_dwordx4 v19, s[40:43], s99 offen offset:3072 lds\n"
    "v_readlane_b32 s76, v44, 16\n"
    "s_nop 2\n"
    "s_add_u32 m0, s87, 0\n"
    "v_readlane_b32 s77, v44, 17\n"
    "s_nop 2\n"
    "buffer_load_dwordx4 v24, s[52:55], s76 offen lds\n"
    "s_add_u32 m0, s87, 4096\n"
    "v_readlane_b32 s76, v44, 18\n"
    "s_nop 2\n"
    "buffer_load_dwordx4 v24, s[52:55], s77 offen lds\n"
    "s_add_u32 m0, s87, 8192\n"
    "v_readlane_b32 s77, v44, 19\n"
    "s_nop 2\n"
    "buffer_load_dwordx4 v24, s[52:55], s76 offen lds\n"
    "s_add_u32 m0, s87, 12288\n"
    "v_readlane_b32 s76, v44, 20\n"
    "s_nop 2\n"
    "buffer_load_dwordx4 v24, s[52:55], s77 offen lds\n"
    "s_add_u32 m0, s87, 16384\n"
    "v_readlane_b32 s77, v44, 21\n"
    "s_nop 2\n"
    "buffer_load_dwordx4 v24, s[52:55], s76 offen lds\n"
    "s_add_u32 m0, s87, 20480\n"
    "v_readlane_b32 s76, v44, 22\n"
    "s_nop 2\n"
    "buffer_load_dwordx4 v24, s[52:55], s77 offen lds\n"
    "s_add_u32 m0, s87, 24576\n"
    "v_readlane_b32 s77, v44, 23\n"
    "s_nop 2\n"
    "buffer_load_dwordx4 v24, s[52:55], s76 offen lds\n"
    "s_add_u32 m0, s87, 28672\n"
    "v_readlane_b32 s76, v44, 24\n"
    "s_nop 2\n"
    "buffer_load_dwordx4 v24, s[52:55], s77 offen lds\n"
    "s_add_u32 m0, s87, 32768\n"
    "v_readlane_b32 s77, v44, 25\n"
    "s_nop 2\n"
    "buffer_load_dwordx4 v24, s[52:55], s76 offen lds\n"
    "s_add_u32 m0, s87, 36864\n"
    "v_readlane_b32 s76, v44, 26\n"
    "s_nop 2\n"
    "buffer_load_dwordx4 v24, s[52:55], s77 offen lds\n"
    "s_waitcnt vmcnt(18) lgkmcnt(8)\n"
    "s_barrier\n"
    "s_add_u32 m0, s85, 0\n"
    "ds_read_b64_tr_b8 a[48:49], v20 offset:32768\n"
    "v_readlane_b32 s99, v44, 10\n"
    "ds_read_b64_tr_b8 a[50:51], v20 offset:33792\n"
    "buffer_load_dwordx4 v16, s[40:43], s98 offen offset:0 lds\n"
    "ds_read_b64_tr_b8 a[52:53], v20 offset:40960\n"
    "buffer_load_dwordx4 v17, s[40:43], s98 offen offset:1024 lds\n"
    "ds_read_b64_tr_b8 a[54:55], v20 offset:41984\n"
    "ds_read_b64_tr_b8 a[56:57], v21 offset:32768\n"
    "ds_read_b64_tr_b8 a[58:59], v21 offset:33792\n"
    "ds_read_b64_tr_b8 a[60:61], v21 offset:40960\n"
    "s_waitcnt lgkmcnt(11)\n"
    "ds_read_b64_tr_b8 a[62:63], v21 offset:41984\n"
    "buffer_load_dwordx4 v18, s[40:43], s98 offen offset:2048 lds\n"
    "buffer_load_dwordx4 v19, s[40:43], s98 offen offset:3072 lds\n"
    "s_waitcnt vmcnt(18) lgkmcnt(8)\n"
    "s_barrier\n"
    "s_add_u32 m0, s85, 16384\n"
    "ds_read_b64_tr_b8 a[224:225], v20 offset:49152\n"
    "v_readlane_b32 s98, v44, 7\n"
    "ds_read_b64_tr_b8 a[226:227], v20 offset:50176\n"
    "buffer_load_dwordx4 v16, s[40:43], s99 offen offset:0 lds\n"
    "ds_read_b64_tr_b8 a[228:229], v20 offset:57344\n"
    "buffer_load_dwordx4 v17, s[40:43], s99 offen offset:1024 lds\n"
    "ds_read_b64_tr_b8 a[230:231], v20 offset:58368\n"
    "ds_read_b64_tr_b8 a[232:233], v21 offset:49152\n"
    "ds_read_b64_tr_b8 a[234:235], v21 offset:50176\n"
    "ds_read_b64_tr_b8 a[236:237], v21 offset:57344\n"
    "s_waitcnt lgkmcnt(11)\n"
    "ds_read_b64_tr_b8 a[238:239], v21 offset:58368\n"
    "buffer_load_dwordx4 v18, s[40:43], s99 offen offset:2048 lds\n"
    "buffer_load_dwordx4 v19, s[40:43], s99 offen offset:3072 lds\n"
    "s_waitcnt vmcnt(18) lgkmcnt(8)\n"
    "s_barrier\n"
    "s_add_u32 m0, s85, 32768\n"
    "ds_read_b64_tr_b8 a[240:241], v22 offset:0\n"
    "v_readlane_b32 s99, v44, 11\n"
    "ds_read_b64_tr_b8 a[242:243], v22 offset:1024\n"
    "buffer_load_dwordx4 v16, s[40:43], s98 offen offset:0 lds\n"
    "ds_read_b64_tr_b8 a[244:245], v22 offset:8192\n"
    "buffer_load_dwordx4 v17, s[40:43], s98 offen offset:1024 lds\n"
    "ds_read_b64_tr_b8 a[246:247], v22 offset:9216\n"
    "ds_read_b64_tr_b8 a[248:249], v23 offset:0\n"
    "ds_read_b64_tr_b8 a[250:251], v23 offset:1024\n"
    "ds_read_b64_tr_b8 a[252:253], v23 offset:8192\n"
    "s_waitcnt lgkmcnt(11)\n"
    "ds_read_b64_tr_b8 a[254:255], v23 offset:9216\n"
    "buffer_load_dwordx4 v18, s[40:43], s98 offen offset:2048 lds\n"
    "buffer_load_dwordx4 v19, s[40:43], s98 offen offset:3072 lds\n"
    "s_waitcnt vmcnt(8) lgkmcnt(8)\n"
    "s_barrier\n"
    "ds_read_b128 v[104:107], v28 offset:0\n"
    "ds_read_b128 v[108:111], v29 offset:0\n"
    "ds_read_b128 v[112:115], v28 offset:2048\n"
    "ds_read_b128 v[116:119], v29 offset:2048\n"
    "ds_read_b128 v[120:123], v28 offset:4096\n"
    "ds_read_b128 v[124:127], v29 offset:4096\n"
    "global_load_dword v42, v38, s[20:21] sc0 sc1\n"
    "ds_read_b64_tr_b8 a[96:97], v20 offset:0\n"
    "s_add_u32 m0, s85, 49152\n"
    "s_waitcnt lgkmcnt(11)\n"
    "ds_read_b64_tr_b8 a[98:99], v20 offset:1024\n"
    "v_readlane_b32 s98, v44, 12\n"
    "ds_read_b64_tr_b8 a[100:101], v20 offset:8192\n"
    "buffer_load_dwordx4 v16, s[40:43], s99 offen offset:0 lds\n"
    "ds_read_b64_tr_b8 a[102:103], v20 offset:9216\n"
    "ds_read_b64_tr_b8 a[104:105], v21 offset:0\n"
    "s_waitcnt lgkmcnt(11)\n"
    "ds_read_b64_tr_b8 a[106:107], v21 offset:1024\n"
    "ds_read_b64_tr_b8 a[108:109], v21 offset:8192\n"
    "ds_read_b64_tr_b8 a[110:111], v21 offset:9216\n"
    "buffer_load_dwordx4 v17, s[40:43], s99 offen offset:1024 lds\n"
    "buffer_load_dwordx4 v18, s[40:43], s99 offen offset:2048 lds\n"
    "buffer_load_dwordx4 v19, s[40:43], s99 offen offset:3072 lds\n"
    "s_waitcnt lgkmcnt(12)\n"
    "v_mfma_f32_16x16x128_f8f6f4 v[152:155], v[104:111], a[224:231], 0 cbsz:1 blgp:1\n"
    "ds_read_b128 v[128:131], v28 offset:6144\n"
    "ds_read_b128 v[132:135], v29 offset:6144\n"
    "v_mfma_f32_16x16x128_f8f6f4 v[156:159], v[104:111], a[232:239], 0 cbsz:1 blgp:1\n"
    "s_waitcnt lgkmcnt(12)\n"
    "v_mfma_f32_16x16x128_f8f6f4 v[152:155], v[112:119], a[240:247], v[152:155] cbsz:1 blgp:1\n"
    "ds_read_b128 v[136:139], v28 offset:8192\n"
    "ds_read_b128 v[140:143], v29 offset:8192\n"
    "v_mfma_f32_16x16x128_f8f6f4 v[156:159], v[112:119], a[248:255], v[156:159] cbsz:1 blgp:1\n"
    "s_waitcnt vmcnt(9)\n"
    "s_barrier\n"
    "s_waitcnt lgkmcnt(12)\n"
    "v_mfma_f32_16x16x128_f8f6f4 v[160:163], v[120:127], a[224:231], 0 cbsz:1 blgp:1\n"
    "ds_read_b128 v[144:147], v28 offset:10240\n"
    "ds_read_b128 v[148:151], v29 offset:10240\n"
    "s_add_u32 m0, s85, 65536\n"
    "ds_read_b64_tr_b8 a[160:161], v20 offset:16384\n"
    "v_mfma_f32_16x16x128_f8f6f4 v[164:167], v[120:127], a[232:239], 0 cbsz:1 blgp:1\n"
    "v_readlane_b32 s99, v44, 13\n"
    "s_waitcnt lgkmcnt(11)\n"
    "ds_read_b64_tr_b8 a[162:163], v20 offset:17408\n"
    "buffer_load_dwordx4 v16, s[40:43], s98 offen offset:0 lds\n"
    "ds_read_b64_tr_b8 a[164:165], v20 offset:24576\n"
    "s_waitcnt lgkmcnt(7)\n"
    "v_mfma_f32_16x16x128_f8f6f4 v[160:163], v[128:135], a[240:247], v[160:163] cbsz:1 blgp:1\n"
    "ds_read_b128 v[96:99], v28 offset:12288\n"
    "ds_read_b128 v[100:103], v29 offset:12288\n"
    "buffer_load_dwordx4 v17, s[40:43], s98 offen offset:1024 lds\n"
    "ds_read_b64_tr_b8 a[166:167], v20 offset:25600\n"
    "v_mfma_f32_16x16x128_f8f6f4 v[164:167], v[128:135], a[248:255], v[164:167] cbsz:1 blgp:1\n"
    "buffer_load_dwordx4 v18, s[40:43], s98 offen offset:2048 lds\n"
    "ds_read_b64_tr_b8 a[168:169], v21 offset:16384\n"
    "ds_read_b64_tr_b8 a[170:171], v21 offset:17408\n"
    "ds_read_b64_tr_b8 a[172:173], v21 offset:24576\n"
    "s_waitcnt lgkmcnt(11)\n"
    "v_mfma_f32_16x16x128_f8f6f4 v[168:171], v[136:143], a[224:231], 0 cbsz:1 blgp:1\n"
    "ds_read_b128 v[104:107], v28 offset:14336\n"
    "ds_read_b128 v[108:111], v29 offset:14336\n"
    "buffer_load_dwordx4 v19, s[40:43], s98 offen offset:3072 lds\n"
    "ds_read_b64_tr_b8 a[174:175], v21 offset:25600\n"
    "v_mfma_f32_16x16x128_f8f6f4 v[172:175], v[136:143], a[232:239], 0 cbsz:1 blgp:1\n"
    "s_waitcnt lgkmcnt(12)\n"
    "v_mfma_f32_16x16x128_f8f6f4 v[168:171], v[144:151], a[240:247], v[168:171] cbsz:1 blgp:1\n"
    "ds_read_b128 v[112:115], v28 offset:16384\n"
    "ds_read_b128 v[116:119], v29 offset:16384\n"
    "v_mfma_f32_16x16x128_f8f6f4 v[172:175], v[144:151], a[248:255], v[172:175] cbsz:1 blgp:1\n"
    "s_waitcnt vmcnt(8)\n"
    "s_barrier\n"
    "s_waitcnt lgkmcnt(9)\n"
    "v_mfma_f32_16x16x128_f8f6f4 v[176:179], v[96:103], a[224:231], 0 cbsz:1 blgp:1\n"
    "ds_read_b128 v[120:123], v28 offset:18432\n"
    "ds_read_b128 v[124:127], v29 offset:18432\n"
    "s_add_u32 m0, s85, 0\n"
    "ds_read_b64_tr_b8 a[112:113], v20 offset:32768\n"
    "v_mfma_f32_16x16x128_f8f6f4 v[180:183], v[96:103], a[232:239], 0 cbsz:1 blgp:1\n"
    "s_nop 0\n"
    "ds_read_b64_tr_b8 a[114:115], v20 offset:33792\n"
    "buffer_load_dwordx4 v16, s[40:43], s99 offen offset:0 lds\n"
    "ds_read_b64_tr_b8 a[116:117], v20 offset:40960\n"
    "s_waitcnt lgkmcnt(8)\n"
    "v_mfma_f32_16x16x128_f8f6f4 v[176:179], v[104:111], a[240:247], v[176:179] cbsz:1 blgp:1\n"
    "ds_read_b128 v[128:131], v28 offset:20480\n"
    "ds_read_b128 v[132:135], v29 offset:20480\n"
    "buffer_load_dwordx4 v17, s[40:43], s99 offen offset:1024 lds\n"
    "ds_read_b64_tr_b8 a[118:119], v20 offset:41984\n"
    "v_mfma_f32_16x16x128_f8f6f4 v[180:183], v[104:111], a[248:255], v[180:183] cbsz:1 blgp:1\n"
    "buffer_load_dwordx4 v18, s[40:43], s99 offen offset:2048 lds\n"
    "ds_read_b64_tr_b8 a[120:121], v21 offset:32768\n"
    "ds_read_b64_tr_b8 a[122:123], v21 offset:33792\n"
    "ds_read_b64_tr_b8 a[124:125], v21 offset:40960\n"
    "s_waitcnt lgkmcnt(11)\n"
    "v_mfma_f32_16x16x128_f8f6f4 v[184:187], v[112:119], a[224:231], 0 cbsz:1 blgp:1\n"
    "ds_read_b128 v[136:139], v28 offset:22528\n"
    "ds_read_b128 v[140:143], v29 offset:22528\n"
    "buffer_load_dwordx4 v19, s[40:43], s99 offen offset:3072 lds\n"
    "ds_read_b64_tr_b8 a[126:127], v21 offset:41984\n"
    "v_mfma_f32_16x16x128_f8f6f4 v[188:191], v[112:119], a[232:239], 0 cbsz:1 blgp:1\n"
    "v_readlane_b32 s74, v42, 0\n"
    "v_readlane_b32 s75, v42, 16\n"
    "v_readlane_b32 s76, v42, 32\n"
    "s_min_u32 s74, s74, s75\n"
    "s_waitcnt lgkmcnt(12)\n"
    "v_mfma_f32_16x16x128_f8f6f4 v[184:187], v[120:127], a[240:247], v[184:187] cbsz:1 blgp:1\n"
    "ds_read_b128 v[144:147], v28 offset:24576\n"
    "ds_read_b128 v[148:151], v29 offset:24576\n"
    "s_min_u32 s96, s76, s74\n"
    "s_min_u32 s79, s96, s69\n"
    "v_mfma_f32_16x16x128_f8f6f4 v[188:191], v[120:127], a[248:255], v[188:191] cbsz:1 blgp:1\n"
    "s_mov_b32 s68, s67\n"
    "s_cmp_lt_u32 s67, s79\n"
    "s_cselect_b32 s46, s97, 0\n"
    "s_cselect_b32 s87, s72, 155648\n"
    "s_cselect_b32 s73, 0x8000, 0\n"
    "s_addc_u32 s67, s67, 0\n"
    "s_mov_b32 m0, s87\n"
    "s_add_u32 s82, s95, 0x1000\n"
    "s_waitcnt vmcnt(8)\n"
    "s_barrier\n"
    "s_waitcnt lgkmcnt(9)\n"
    "v_mfma_f32_16x16x128_f8f6f4 v[192:195], v[128:135], a[224:231], 0 cbsz:1 blgp:1\n"
    "ds_read_b128 v[96:99], v28 offset:26624\n"
    "ds_read_b128 v[100:103], v29 offset:26624\n"
    "buffer_load_dwordx4 v25, s[44:47], s95 offen offset:0 sc0 sc1 lds\n"
    "ds_read_b64_tr_b8 a[176:177], v20 offset:49152\n"
    "v_mfma_f32_16x16x128_f8f6f4 v[196:199], v[128:135], a[232:239], 0 cbsz:1 blgp:1\n"
    "s_add_u32 s72, s72, s73\n"
    "ds_read_b64_tr_b8 a[178:179], v20 offset:50176\n"
    "s_and_b32 s72, s72, 0x1ffff\n"
    "ds_read_b64_tr_b8 a[180:181], v20 offset:57344\n"
    "s_waitcnt lgkmcnt(8)\n"
    "v_mfma_f32_16x16x128_f8f6f4 v[192:195], v[136:143], a[240:247], v[192:195] cbsz:1 blgp:1\n"
    "ds_read_b128 v[104:107], v28 offset:28672\n"
    "ds_read_b128 v[108:111], v29 offset:28672\n"
    "buffer_load_dwordx4 v25, s[44:47], s95 offen offset:1024 sc0 sc1 lds\n"
    "ds_read_b64_tr_b8 a[182:183], v20 offset:58368\n"
    "v_mfma_f32_16x16x128_f8f6f4 v[196:199], v[136:143], a[248:255], v[196:199] cbsz:1 blgp:1\n"
    "buffer_load_dwordx4 v25, s[44:47], s95 offen offset:2048 sc0 sc1 lds\n"
    "ds_read_b64_tr_b8 a[184:185], v21 offset:49152\n"
    "ds_read_b64_tr_b8 a[186:187], v21 offset:50176\n"
    "ds_read_b64_tr_b8 a[188:189], v21 offset:57344\n"
    "s_waitcnt lgkmcnt(11)\n"
    "v_mfma_f32_16x16x128_f8f6f4 v[200:203], v[144:151], a[224:231], 0 cbsz:1 blgp:1\n"
    "ds_read_b128 v[112:115], v28 offset:30720\n"
    "ds_read_b128 v[116:119], v29 offset:30720\n"
    "buffer_load_dwordx4 v25, s[44:47], s95 offen offset:3072 sc0 sc1 lds\n"
    "ds_read_b64_tr_b8 a[190:191], v21 offset:58368\n"
    "v_mfma_f32_16x16x128_f8f6f4 v[204:207], v[144:151], a[232:239], 0 cbsz:1 blgp:1\n"
    "s_add_u32 m0, s87, 0x1000\n"
    "s_lshl1_add_u32 s95, s73, s95\n"
    "s_waitcnt lgkmcnt(12)\n"
    "v_mfma_f32_16x16x128_f8f6f4 v[200:203], v[96:103], a[240:247], v[200:203] cbsz:1 blgp:1\n"
    "ds_read_b128 v[120:123], v28 offset:32768\n"
    "ds_read_b128 v[124:127], v29 offset:32768\n"
    "v_mfma_f32_16x16x128_f8f6f4 v[204:207], v[96:103], a[248:255], v[204:207] cbsz:1 blgp:1\n"
    "s_waitcnt vmcnt(8)\n"
    "s_barrier\n"
    "s_waitcnt lgkmcnt(9)\n"
    "v_mfma_f32_16x16x128_f8f6f4 v[208:211], v[104:111], a[224:231], 0 cbsz:1 blgp:1\n"
    "ds_read_b128 v[128:131], v28 offset:34816\n"
    "ds_read_b128 v[132:135], v29 offset:34816\n"
    "buffer_load_dwordx4 v25, s[44:47], s82 offen offset:0 sc0 sc1 lds\n"
    "ds_read_b64_tr_b8 a[192:193], v22 offset:0\n"
    "v_mfma_f32_16x16x128_f8f6f4 v[212:215], v[104:111], a[232:239], 0 cbsz:1 blgp:1\n"
    "buffer_load_dwordx4 v25, s[44:47], s82 offen offset:1024 sc0 sc1 lds\n"
    "ds_read_b64_tr_b8 a[194:195], v22 offset:1024\n"
    "ds_read_b64_tr_b8 a[196:197], v22 offset:8192\n"
    "ds_read_b64_tr_b8 a[198:199], v22 offset:9216\n"
    "s_waitcnt lgkmcnt(9)\n"
    "v_mfma_f32_16x16x128_f8f6f4 v[208:211], v[112:119], a[240:247], v[208:211] cbsz:1 blgp:1\n"
    "ds_read_b128 v[136:139], v28 offset:36864\n"
    "ds_read_b128 v[140:143], v29 offset:36864\n"
    "buffer_load_dwordx4 v25, s[44:47], s82 offen offset:2048 sc0 sc1 lds\n"
    "ds_read_b64_tr_b8 a[200:201], v23 offset:0\n"
    "v_mfma_f32_16x16x128_f8f6f4 v[212:215], v[112:119], a[248:255], v[212:215] cbsz:1 blgp:1\n"
    "global_load_dword v42, v38, s[20:21] sc0 sc1\n"
    "ds_read_b64_tr_b8 a[202:203], v23 offset:1024\n"
    "ds_read_b64_tr_b8 a[204:205], v23 offset:8192\n"
    "ds_read_b64_tr_b8 a[206:207], v23 offset:9216\n"
    "s_waitcnt lgkmcnt(12)\n"
    "v_mfma_f32_16x16x128_f8f6f4 v[216:219], v[120:127], a[224:231], 0 cbsz:1 blgp:1\n"
    "ds_read_b128 v[144:147], v28 offset:38912\n"
    "ds_read_b128 v[148:151], v29 offset:38912\n"
    "v_mfma_f32_16x16x128_f8f6f4 v[220:223], v[120:127], a[232:239], 0 cbsz:1 blgp:1\n"
    "s_waitcnt vmcnt(8)\n"
    "s_barrier\n"
    "ds_read_b64_tr_b8 a[208:209], v20 offset:0\n"
    "s_waitcnt lgkmcnt(11)\n"
    "ds_read_b64_tr_b8 a[210:211], v20 offset:1024\n"
    "ds_read_b64_tr_b8 a[212:213], v20 offset:8192\n"
    "ds_read_b64_tr_b8 a[214:215], v20 offset:9216\n"
    "ds_read_b64_tr_b8 a[216:217], v21 offset:0\n"
    "s_waitcnt lgkmcnt(11)\n"
    "ds_read_b64_tr_b8 a[218:219], v21 offset:1024\n"
    "ds_read_b64_tr_b8 a[220:221], v21 offset:8192\n"
    "ds_read_b64_tr_b8 a[222:223], v21 offset:9216\n"
    "s_waitcnt vmcnt(0) lgkmcnt(0)\n"
    "s_cmp_lt_u32 s64, s67\n"
    "s_cbranch_scc0 L_c0%=\n"
    "L_r0%=:\n"
    "s_barrier\n"
    "ds_read_b128 v[96:99], v26 offset:0\n"
    "ds_read_b128 v[100:103], v27 offset:0\n"
    "v_mfma_f32_16x16x128_f8f6f4 v[216:219], v[128:135], a[240:247], v[216:219] cbsz:1 blgp:1\n"
    "ds_read_b128 v[104:107], v26 offset:2048\n"
    "ds_read_b128 v[108:111], v27 offset:2048\n"
    "s_add_u32 s69, s70, 3\n"
    "v_readlane_b32 s74, v42, 0\n"
    "v_readlane_b32 s75, v42, 16\n"
    "v_mfma_f32_16x16x128_f8f6f4 v[220:223], v[128:135], a[248:255], v[220:223] cbsz:1 blgp:1\n"
    "v_readlane_b32 s76, v42, 32\n"
    "s_min_u32 s74, s74, s75\n"
    "s_min_u32 s96, s76, s74\n"
    "s_min_u32 s79, s96, s69\n"
    "s_mov_b32 s68, s67\n"
    "v_mfma_f32_16x16x128_f8f6f4 v[224:227], v[136:143], a[224:231], 0 cbsz:1 blgp:1\n"
    "ds_read_b128 v[112:115], v26 offset:4096\n"
    "ds_read_b128 v[116:119], v27 offset:4096\n"
    "s_cmp_lt_u32 s67, s79\n"
    "s_cselect_b32 s46, s97, 0\n"
    "s_cselect_b32 s87, s72, 155648\n"
    "v_mfma_f32_16x16x128_f8f6f4 v[228:231], v[136:143], a[232:239], 0 cbsz:1 blgp:1\n"
    "s_cselect_b32 s73, 0x8000, 0\n"
    "s_addc_u32 s67, s67, 0\n"
    "s_mov_b32 m0, s87\n"
    "s_add_u32 s82, s95, 0x1000\n"
    "buffer_load_dwordx4 v25, s[44:47], s95 offen offset:0 sc0 sc1 lds\n"
    "v_mfma_f32_16x16x128_f8f6f4 v[224:227], v[144:151], a[240:247], v[224:227] cbsz:1 blgp:1\n"
    "ds_read_b128 v[120:123], v26 offset:6144\n"
    "ds_read_b128 v[124:127], v27 offset:6144\n"
    "s_add_u32 s72, s72, s73\n"
    "s_and_b32 s72, s72, 0x1ffff\n"
    "buffer_load_dwordx4 v25, s[44:47], s95 offen offset:1024 sc0 sc1 lds\n"
    "v_mfma_f32_16x16x128_f8f6f4 v[228:231], v[144:151], a[248:255], v[228:231] cbsz:1 blgp:1\n"
    "buffer_load_dwordx4 v25, s[44:47], s95 offen offset:2048 sc0 sc1 lds\n"
    "s_waitcnt lgkmcnt(6)\n"
    "v_mfma_f32_16x16x128_f8f6f4 v[152:155], v[96:103], a[0:7], v[152:155] cbsz:1 blgp:1\n"
    "ds_read_b128 v[128:131], v26 offset:8192\n"
    "ds_read_b128 v[132:135], v27 offset:8192\n"
    "v_mfma_f32_16x16x128_f8f6f4 v[156:159], v[96:103], a[8:15], v[156:159] cbsz:1 blgp:1\n"
    "buffer_load_dwordx4 v25, s[44:47], s95 offen offset:3072 sc0 sc1 lds\n"
    "s_add_u32 m0, s87, 0x1000\n"
    "s_lshl1_add_u32 s95, s73, s95\n"
    "s_waitcnt lgkmcnt(6)\n"
    "v_mfma_f32_16x16x128_f8f6f4 v[152:155], v[104:111], a[16:23], v[152:155] cbsz:1 blgp:1\n"
    "ds_read_b128 v[136:139], v26 offset:10240\n"
    "ds_read_b128 v[140:143], v27 offset:10240\n"
    "buffer_load_dwordx4 v25, s[44:47], s82 offen offset:0 sc0 sc1 lds\n"
    "v_mfma_f32_16x16x128_f8f6f4 v[156:159], v[104:111], a[24:31], v[156:159] cbsz:1 blgp:1\n"
    "buffer_load_dwordx4 v25, s[44:47], s82 offen offset:1024 sc0 sc1 lds\n"
    "s_waitcnt lgkmcnt(6)\n"
    "v_mfma_f32_16x16x128_f8f6f4 v[152:155], v[112:119], a[32:39], v[152:155] cbsz:1 blgp:1\n"
    "ds_read_b128 v[144:147], v26 offset:12288\n"
    "ds_read_b128 v[148:151], v27 offset:12288\n"
    "v_mfma_f32_16x16x128_f8f6f4 v[156:159], v[112:119], a[40:47], v[156:159] cbsz:1 blgp:1\n"
    "buffer_load_dwordx4 v25, s[44:47], s82 offen offset:2048 sc0 sc1 lds\n"
    "s_add_u32 s74, s70, 4\n"
    "s_cmp_lt_u32 s96, s74\n"
    "s_cbranch_scc0 L_n0%=\n"
    "global_load_dword v42, v38, s[20:21] sc0 sc1\n"
    "L_n0%=:\n"
    "v_readlane_b32 s74, v42, 0\n"
    "v_readlane_b32 s75, v42, 16\n"
    "v_readlane_b32 s76, v42, 32\n"
    "s_waitcnt lgkmcnt(6)\n"
    "v_mfma_f32_16x16x128_f8f6f4 v[152:155], v[120:127], a[48:55], v[152:155] cbsz:1 blgp:1\n"
    "ds_read_b128 v[96:99], v26 offset:14336\n"
    "ds_read_b128 v[100:103], v27 offset:14336\n"
    "s_min_u32 s74, s74, s75\n"
    "s_min_u32 s96, s76, s74\n"
    "s_min_u32 s79, s96, s69\n"
    "v_mfma_f32_16x16x128_f8f6f4 v[156:159], v[120:127], a[56:63], v[156:159] cbsz:1 blgp:1\n"
    "s_mov_b32 s68, s67\n"
    "s_cmp_lt_u32 s67, s79\n"
    "s_cselect_b32 s46, s97, 0\n"
    "s_cselect_b32 s87, s72, 155648\n"
    "s_cselect_b32 s73, 0x8000, 0\n"
    "s_waitcnt lgkmcnt(6)\n"
    "v_mfma_f32_16x16x128_f8f6f4 v[152:155], v[128:135], a[64:71], v[152:155] cbsz:1 blgp:1\n"
    "ds_read_b128 v[104:107], v26 offset:16384\n"
    "ds_read_b128 v[108:111], v27 offset:16384\n"
    "s_addc_u32 s67, s67, 0\n"
    "s_mov_b32 m0, s87\n"
    "s_add_u32 s82, s95, 0x1000\n"
    "v_mfma_f32_16x16x128_f8f6f4 v[156:159], v[128:135], a[72:79], v[156:159] cbsz:1 blgp:1\n"
    "buffer_load_dwordx4 v25, s[44:47], s95 offen offset:0 sc0 sc1 lds\n"
    "s_add_u32 s72, s72, s73\n"
    "s_and_b32 s72, s72, 0x1ffff\n"
    "s_waitcnt lgkmcnt(6)\n"
    "v_mfma_f32_16x16x128_f8f6f4 v[152:155], v[136:143], a[80:87], v[152:155] cbsz:1 blgp:1\n"
    "ds_read_b128 v[112:115], v26 offset:18432\n"
    "ds_read_b128 v[116:119], v27 offset:18432\n"
    "v_mfma_f32_16x16x128_f8f6f4 v[156:159], v[136:143], a[88:95], v[156:159] cbsz:1 blgp:1\n"
    "buffer_load_dwordx4 v25, s[44:47], s95 offen offset:1024 sc0 sc1 lds\n"
    "s_waitcnt lgkmcnt(6)\n"
    "v_mfma_f32_16x16x128_f8f6f4 v[152:155], v[144:151], a[96:103], v[152:155] cbsz:1 blgp:1\n"
    "ds_read_b128 v[120:123], v26 offset:20480\n"
    "ds_read_b128 v[124:127], v27 offset:20480\n"
    "v_mfma_f32_16x16x128_f8f6f4 v[156:159], v[144:151], a[104:111], v[156:159] cbsz:1 blgp:1\n"
    "buffer_load_dwordx4 v25, s[44:47], s95 offen offset:2048 sc0 sc1 lds\n"
    "s_waitcnt lgkmcnt(6)\n"
    "v_mfma_f32_16x16x128_f8f6f4 v[152:155], v[96:103], a[112:119], v[152:155] cbsz:1 blgp:1\n"
    "ds_read_b128 v[128:131], v26 offset:22528\n"
    "ds_read_b128 v[132:135], v27 offset:22528\n"
    "buffer_load_dwordx4 v25, s[44:47], s95 offen offset:3072 sc0 sc1 lds\n"
    "s_add_u32 m0, s87, 0x1000\n"
    "s_lshl1_add_u32 s95, s73, s95\n"
    "v_mfma_f32_16x16x128_f8f6f4 v[156:159], v[96:103], a[120:127], v[156:159] cbsz:1 blgp:1\n"
    "buffer_load_dwordx4 v25, s[44:47], s82 offen offset:0 sc0 sc1 lds\n"
    "s_waitcnt lgkmcnt(6)\n"
    "v_mfma_f32_16x16x128_f8f6f4 v[152:155], v[104:111], a[128:135], v[152:155] cbsz:1 blgp:1\n"
    "ds_read_b128 v[136:139], v26 offset:24576\n"
    "ds_read_b128 v[140:143], v27 offset:24576\n"
    "v_mfma_f32_16x16x128_f8f6f4 v[156:159], v[104:111], a[136:143], v[156:159] cbsz:1 blgp:1\n"
    "buffer_load_dwordx4 v25, s[44:47], s82 offen offset:1024 sc0 sc1 lds\n"
    "s_waitcnt lgkmcnt(6)\n"
    "v_mfma_f32_16x16x128_f8f6f4 v[152:155], v[112:119], a[144:151], v[152:155] cbsz:1 blgp:1\n"
    "ds_read_b128 v[144:147], v26 offset:26624\n"
    "ds_read_b128 v[148:151], v27 offset:26624\n"
    "v_mfma_f32_16x16x128_f8f6f4 v[156:159], v[112:119], a[152:159], v[156:159] cbsz:1 blgp:1\n"
    "buffer_load_dwordx4 v25, s[44:47], s82 offen offset:2048 sc0 sc1 lds\n"
    "s_add_u32 s64, s64, 1\n"
    "s_add_u32 s69, s64, 3\n"
    "s_waitcnt vmcnt(7)\n"
    "s_cmp_lt_u32 s64, s68\n"
    "s_cbranch_scc0 L_c1%=\n"
    "L_r1%=:\n"
    "s_waitcnt lgkmcnt(6)\n"
    "v_mfma_f32_16x16x128_f8f6f4 v[152:155], v[120:127], a[160:167], v[152:155] cbsz:1 blgp:1\n"
    "s_barrier\n"
    "ds_read_b128 v[96:99], v26 offset:32768\n"
    "ds_read_b128 v[100:103], v27 offset:32768\n"
    "s_add_u32 s64, s64, 1\n"
    "v_mfma_f32_16x16x128_f8f6f4 v[156:159], v[120:127], a[168:175], v[156:159] cbsz:1 blgp:1\n"
    "s_cmp_le_u32 s67, s64\n"
    "s_cbranch_scc1 L_u1_0%=\n"
    "L_v1_0%=:\n"
    "s_waitcnt lgkmcnt(6)\n"
    "v_mfma_f32_16x16x128_f8f6f4 v[152:155], v[128:135], a[176:183], v[152:155] cbsz:1 blgp:1\n"
    "ds_read_b128 v[104:107], v26 offset:34816\n"
    "ds_read_b128 v[108:111], v27 offset:34816\n"
    "v_mfma_f32_16x16x128_f8f6f4 v[156:159], v[128:135], a[184:191], v[156:159] cbsz:1 blgp:1\n"
    "s_add_u32 s74, s70, 5\n"
    "s_cmp_lt_u32 s96, s74\n"
    "s_cbranch_scc0 L_n1%=\n"
    "global_load_dword v42, v38, s[20:21] sc0 sc1\n"
    "L_n1%=:\n"
    "s_waitcnt lgkmcnt(6)\n"
    "v_mfma_f32_16x16x128_f8f6f4 v[152:155], v[136:143], a[192:199], v[152:155] cbsz:1 blgp:1\n"
    "ds_read_b128 v[112:115], v26 offset:36864\n"
    "ds_read_b128 v[116:119], v27 offset:36864\n"
    "v_mfma_f32_16x16x128_f8f6f4 v[156:159], v[136:143], a[200:207], v[156:159] cbsz:1 blgp:1\n"
    "s_waitcnt lgkmcnt(6)\n"
    "v_mfma_f32_16x16x128_f8f6f4 v[152:155], v[144:151], a[208:215], v[152:155] cbsz:1 blgp:1\n"
    "ds_read_b128 v[120:123], v26 offset:38912\n"
    "ds_read_b128 v[124:127], v27 offset:38912\n"
    "v_readlane_b32 s74, v42, 0\n"
    "v_readlane_b32 s75, v42, 16\n"
    "v_readlane_b32 s76, v42, 32\n"
    "v_mfma_f32_16x16x128_f8f6f4 v[156:159], v[144:151], a[216:223], v[156:159] cbsz:1 blgp:1\n"
    "s_min_u32 s74, s74, s75\n"
    "s_min_u32 s96, s76, s74\n"
    "s_min_u32 s79, s96, s69\n"
    "s_add_u32 s74, s64, 1\n"
    "s_min_u32 s74, s74, s79\n"
    "s_waitcnt lgkmcnt(6)\n"
    "v_mfma_f32_16x16x128_f8f6f4 v[160:163], v[96:103], a[0:7], v[160:163] cbsz:1 blgp:1\n"
    "ds_read_b128 v[128:131], v26 offset:40960\n"
    "ds_read_b128 v[132:135], v27 offset:40960\n"
    "s_cmp_lt_u32 s67, s74\n"
    "s_cbranch_scc1 L_e1%=\n"
    "L_f1%=:\n"
    "s_mov_b32 s68, s67\n"
    "v_mfma_f32_16x16x128_f8f6f4 v[164:167], v[96:103], a[8:15], v[164:167] cbsz:1 blgp:1\n"
    "s_cmp_lt_u32 s67, s79\n"
    "s_cselect_b32 s46, s97, 0\n"
    "s_cselect_b32 s87, s72, 155648\n"
    "s_cselect_b32 s73, 0x8000, 0\n"
    "s_addc_u32 s67, s67, 0\n"
    "s_waitcnt lgkmcnt(6)\n"
    "v_mfma_f32_16x16x128_f8f6f4 v[160:163], v[104:111], a[16:23], v[160:163] cbsz:1 blgp:1\n"
    "ds_read_b128 v[136:139], v26 offset:43008\n"
    "ds_read_b128 v[140:143], v27 offset:43008\n"
    "v_mul_f32 v152, v43, v152\n"
    "v_mul_f32 v153, v43, v153\n"
    "v_mul_f32 v154, v43, v154\n"
    "v_mul_f32 v155, v43, v155\n"
    "v_mfma_f32_16x16x128_f8f6f4 v[164:167], v[104:111], a[24:31], v[164:167] cbsz:1 blgp:1\n"
    "global_store_dwordx4 v40, v[152:155], s[60:61] offset:-4096 sc0 sc1\n"
    "s_waitcnt lgkmcnt(6)\n"
    "v_mfma_f32_16x16x128_f8f6f4 v[160:163], v[112:119], a[32:39], v[160:163] cbsz:1 blgp:1\n"
    "ds_read_b128 v[144:147], v26 offset:45056\n"
    "ds_read_b128 v[148:151], v27 offset:45056\n"
    "v_mul_f32 v156, v43, v156\n"
    "v_mul_f32 v157, v43, v157\n"
    "v_mul_f32 v158, v43, v158\n"
    "v_mul_f32 v159, v43, v159\n"
    "v_mfma_f32_16x16x128_f8f6f4 v[164:167], v[112:119], a[40:47], v[164:167] cbsz:1 blgp:1\n"
    "global_store_dwordx4 v40, v[156:159], s[60:61] offset:-3072 sc0 sc1\n"
    "s_waitcnt lgkmcnt(6)\n"
    "v_mfma_f32_16x16x128_f8f6f4 v[160:163], v[120:127], a[48:55], v[160:163] cbsz:1 blgp:1\n"
    "ds_read_b128 v[96:99], v26 offset:47104\n"
    "ds_read_b128 v[100:103], v27 offset:47104\n"
    "s_mov_b32 m0, s87\n"
    "s_add_u32 s82, s95, 0x1000\n"
    "v_mfma_f32_16x16x128_f8f6f4 v[164:167], v[120:127], a[56:63], v[164:167] cbsz:1 blgp:1\n"
    "buffer_load_dwordx4 v25, s[44:47], s95 offen offset:0 sc0 sc1 lds\n"
    "s_add_u32 s72, s72, s73\n"
    "s_and_b32 s72, s72, 0x1ffff\n"
    "s_waitcnt lgkmcnt(6)\n"
    "v_mfma_f32_16x16x128_f8f6f4 v[160:163], v[128:135], a[64:71], v[160:163] cbsz:1 blgp:1\n"
    "ds_read_b128 v[104:107], v26 offset:49152\n"
    "ds_read_b128 v[108:111], v27 offset:49152\n"
    "s_add_u32 s69, s64, 3\n"
    "v_mfma_f32_16x16x128_f8f6f4 v[164:167], v[128:135], a[72:79], v[164:167] cbsz:1 blgp:1\n"
    "buffer_load_dwordx4 v25, s[44:47], s95 offen offset:1024 sc0 sc1 lds\n"
    "s_waitcnt lgkmcnt(6)\n"
    "v_mfma_f32_16x16x128_f8f6f4 v[160:163], v[136:143], a[80:87], v[160:163] cbsz:1 blgp:1\n"
    "ds_read_b128 v[112:115], v26 offset:51200\n"
    "ds_read_b128 v[116:119], v27 offset:51200\n"
    "v_mfma_f32_16x16x128_f8f6f4 v[164:167], v[136:143], a[88:95], v[164:167] cbsz:1 blgp:1\n"
    "buffer_load_dwordx4 v25, s[44:47], s95 offen offset:2048 sc0 sc1 lds\n"
    "s_waitcnt lgkmcnt(6)\n"
    "v_mfma_f32_16x16x128_f8f6f4 v[160:163], v[144:151], a[96:103], v[160:163] cbsz:1 blgp:1\n"
    "ds_read_b128 v[120:123], v26 offset:53248\n"
    "ds_read_b128 v[124:127], v27 offset:53248\n"
    "v_mfma_f32_16x16x128_f8f6f4 v[164:167], v[144:151], a[104:111], v[164:167] cbsz:1 blgp:1\n"
    "buffer_load_dwordx4 v25, s[44:47], s95 offen offset:3072 sc0 sc1 lds\n"
    "s_lshl1_add_u32 s95, s73, s95\n"
    "s_add_u32 m0, s87, 0x1000\n"
    "s_waitcnt lgkmcnt(6)\n"
    "v_mfma_f32_16x16x128_f8f6f4 v[160:163], v[96:103], a[112:119], v[160:163] cbsz:1 blgp:1\n"
    "ds_read_b128 v[128:131], v26 offset:55296\n"
    "ds_read_b128 v[132:135], v27 offset:55296\n"
    "v_mfma_f32_16x16x128_f8f6f4 v[164:167], v[96:103], a[120:127], v[164:167] cbsz:1 blgp:1\n"
    "buffer_load_dwordx4 v25, s[44:47], s82 offen offset:0 sc0 sc1 lds\n"
    "s_waitcnt lgkmcnt(6)\n"
    "v_mfma_f32_16x16x128_f8f6f4 v[160:163], v[104:111], a[128:135], v[160:163] cbsz:1 blgp:1\n"
    "ds_read_b128 v[136:139], v26 offset:57344\n"
    "ds_read_b128 v[140:143], v27 offset:57344\n"
    "v_mfma_f32_16x16x128_f8f6f4 v[164:167], v[104:111], a[136:143], v[164:167] cbsz:1 blgp:1\n"
    "buffer_load_dwordx4 v25, s[44:47], s82 offen offset:1024 sc0 sc1 lds\n"
    "s_waitcnt lgkmcnt(6)\n"
    "v_mfma_f32_16x16x128_f8f6f4 v[160:163], v[112:119], a[144:151], v[160:163] cbsz:1 blgp:1\n"
    "ds_read_b128 v[144:147], v26 offset:59392\n"
    "ds_read_b128 v[148:151], v27 offset:59392\n"
    "v_mfma_f32_16x16x128_f8f6f4 v[164:167], v[112:119], a[152:159], v[164:167] cbsz:1 blgp:1\n"
    "buffer_load_dwordx4 v25, s[44:47], s82 offen offset:2048 sc0 sc1 lds\n"
    "s_waitcnt vmcnt(9)\n"
    "s_cmp_lt_u32 s64, s68\n"
    "s_cbranch_scc0 L_c2%=\n"
    "L_r2%=:\n"
    "s_waitcnt lgkmcnt(6)\n"
    "v_mfma_f32_16x16x128_f8f6f4 v[160:163], v[120:127], a[160:167], v[160:163] cbsz:1 blgp:1\n"
    "s_barrier\n"
    "ds_read_b128 v[96:99], v30 offset:0\n"
    "ds_read_b128 v[100:103], v31 offset:0\n"
    "s_add_u32 s64, s64, 1\n"
    "v_mfma_f32_16x16x128_f8f6f4 v[164:167], v[120:127], a[168:175], v[164:167] cbsz:1 blgp:1\n"
    "s_cmp_le_u32 s67, s64\n"
    "s_cbranch_scc1 L_u2_0%=\n"
    "L_v2_0%=:\n"
    "s_waitcnt lgkmcnt(6)\n"
    "v_mfma_f32_16x16x128_f8f6f4 v[160:163], v[128:135], a[176:183], v[160:163] cbsz:1 blgp:1\n"
    "ds_read_b128 v[104:107], v30 offset:2048\n"
    "ds_read_b128 v[108:111], v31 offset:2048\n"
    "v_mfma_f32_16x16x128_f8f6f4 v[164:167], v[128:135], a[184:191], v[164:167] cbsz:1 blgp:1\n"
    "s_add_u32 s74, s70, 6\n"
    "s_cmp_lt_u32 s96, s74\n"
    "s_cbranch_scc0 L_n2%=\n"
    "global_load_dword v42, v38, s[20:21] sc0 sc1\n"
    "L_n2%=:\n"
    "s_waitcnt lgkmcnt(6)\n"
    "v_mfma_f32_16x16x128_f8f6f4 v[160:163], v[136:143], a[192:199], v[160:163] cbsz:1 blgp:1\n"
    "ds_read_b128 v[112:115], v30 offset:4096\n"
    "ds_read_b128 v[116:119], v31 offset:4096\n"
    "v_mfma_f32_16x16x128_f8f6f4 v[164:167], v[136:143], a[200:207], v[164:167] cbsz:1 blgp:1\n"
    "s_waitcnt lgkmcnt(6)\n"
    "v_mfma_f32_16x16x128_f8f6f4 v[160:163], v[144:151], a[208:215], v[160:163] cbsz:1 blgp:1\n"
    "ds_read_b128 v[120:123], v30 offset:6144\n"
    "ds_read_b128 v[124:127], v31 offset:6144\n"
    "v_readlane_b32 s74, v42, 0\n"
    "v_readlane_b32 s75, v42, 16\n"
    "v_readlane_b32 s76, v42, 32\n"
    "v_mfma_f32_16x16x128_f8f6f4 v[164:167], v[144:151], a[216:223], v[164:167] cbsz:1 blgp:1\n"
    "s_min_u32 s74, s74, s75\n"
    "s_min_u32 s96, s76, s74\n"
    "s_min_u32 s79, s96, s69\n"
    "s_add_u32 s74, s64, 1\n"
    "s_min_u32 s74, s74, s79\n"
    "s_waitcnt lgkmcnt(6)\n"
    "v_mfma_f32_16x16x128_f8f6f4 v[168:171], v[96:103], a[0:7], v[168:171] cbsz:1 blgp:1\n"
    "ds_read_b128 v[128:131], v30 offset:8192\n"
    "ds_read_b128 v[132:135], v31 offset:8192\n"
    "s_cmp_lt_u32 s67, s74\n"
    "s_cbranch_scc1 L_e2%=\n"
    "L_f2%=:\n"
    "s_mov_b32 s68, s67\n"
    "v_mfma_f32_16x16x128_f8f6f4 v[172:175], v[96:103], a[8:15], v[172:175] cbsz:1 blgp:1\n"
    "s_cmp_lt_u32 s67, s79\n"
    "s_cselect_b32 s46, s97, 0\n"
    "s_cselect_b32 s87, s72, 155648\n"
    "s_cselect_b32 s73, 0x8000, 0\n"
    "s_addc_u32 s67, s67, 0\n"
    "s_waitcnt lgkmcnt(6)\n"
    "v_mfma_f32_16x16x128_f8f6f4 v[168:171], v[104:111], a[16:23], v[168:171] cbsz:1 blgp:1\n"
    "ds_read_b128 v[136:139], v30 offset:10240\n"
    "ds_read_b128 v[140:143], v31 offset:10240\n"
    "v_mul_f32 v160, v43, v160\n"
    "v_mul_f32 v161, v43, v161\n"
    "v_mul_f32 v162, v43, v162\n"
    "v_mul_f32 v163, v43, v163\n"
    "v_mfma_f32_16x16x128_f8f6f4 v[172:175], v[104:111], a[24:31], v[172:175] cbsz:1 blgp:1\n"
    "global_store_dwordx4 v40, v[160:163], s[60:61] offset:-2048 sc0 sc1\n"
    "s_waitcnt lgkmcnt(6)\n"
    "v_mfma_f32_16x16x128_f8f6f4 v[168:171], v[112:119], a[32:39], v[168:171] cbsz:1 blgp:1\n"
    "ds_read_b128 v[144:147], v30 offset:12288\n"
    "ds_read_b128 v[148:151], v31 offset:12288\n"
    "v_mul_f32 v164, v43, v164\n"
    "v_mul_f32 v165, v43, v165\n"
    "v_mul_f32 v166, v43, v166\n"
    "v_mul_f32 v167, v43, v167\n"
    "v_mfma_f32_16x16x128_f8f6f4 v[172:175], v[112:119], a[40:47], v[172:175] cbsz:1 blgp:1\n"
    "global_store_dwordx4 v40, v[164:167], s[60:61] offset:-1024 sc0 sc1\n"
    "s_waitcnt lgkmcnt(6)\n"
    "v_mfma_f32_16x16x128_f8f6f4 v[168:171], v[120:127], a[48:55], v[168:171] cbsz:1 blgp:1\n"
    "ds_read_b128 v[96:99], v30 offset:14336\n"
    "ds_read_b128 v[100:103], v31 offset:14336\n"
    "s_mov_b32 m0, s87\n"
    "s_add_u32 s82, s95, 0x1000\n"
    "v_mfma_f32_16x16x128_f8f6f4 v[172:175], v[120:127], a[56:63], v[172:175] cbsz:1 blgp:1\n"
    "buffer_load_dwordx4 v25, s[44:47], s95 offen offset:0 sc0 sc1 lds\n"
    "s_add_u32 s72, s72, s73\n"
    "s_and_b32 s72, s72, 0x1ffff\n"
    "s_waitcnt lgkmcnt(6)\n"
    "v_mfma_f32_16x16x128_f8f6f4 v[168:171], v[128:135], a[64:71], v[168:171] cbsz:1 blgp:1\n"
    "ds_read_b128 v[104:107], v30 offset:16384\n"
    "ds_read_b128 v[108:111], v31 offset:16384\n"
    "s_add_u32 s69, s64, 3\n"
    "v_mfma_f32_16x16x128_f8f6f4 v[172:175], v[128:135], a[72:79], v[172:175] cbsz:1 blgp:1\n"
    "buffer_load_dwordx4 v25, s[44:47], s95 offen offset:1024 sc0 sc1 lds\n"
    "s_waitcnt lgkmcnt(6)\n"
    "v_mfma_f32_16x16x128_f8f6f4 v[168:171], v[136:143], a[80:87], v[168:171] cbsz:1 blgp:1\n"
    "ds_read_b128 v[112:115], v30 offset:18432\n"
    "ds_read_b128 v[116:119], v31 offset:18432\n"
    "v_mfma_f32_16x16x128_f8f6f4 v[172:175], v[136:143], a[88:95], v[172:175] cbsz:1 blgp:1\n"
    "buffer_load_dwordx4 v25, s[44:47], s95 offen offset:2048 sc0 sc1 lds\n"
    "s_waitcnt lgkmcnt(6)\n"
    "v_mfma_f32_16x16x128_f8f6f4 v[168:171], v[144:151], a[96:103], v[168:171] cbsz:1 blgp:1\n"
    "ds_read_b128 v[120:123], v30 offset:20480\n"
    "ds_read_b128 v[124:127], v31 offset:20480\n"
    "v_mfma_f32_16x16x128_f8f6f4 v[172:175], v[144:151], a[104:111], v[172:175] cbsz:1 blgp:1\n"
    "buffer_load_dwordx4 v25, s[44:47], s95 offen offset:3072 sc0 sc1 lds\n"
    "s_lshl1_add_u32 s95, s73, s95\n"
    "s_add_u32 m0, s87, 0x1000\n"
    "s_waitcnt lgkmcnt(6)\n"
    "v_mfma_f32_16x16x128_f8f6f4 v[168:171], v[96:103], a[112:119], v[168:171] cbsz:1 blgp:1\n"
    "ds_read_b128 v[128:131], v30 offset:22528\n"
    "ds_read_b128 v[132:135], v31 offset:22528\n"
    "s_waitcnt vmcnt(13)\n"
    "s_barrier\n"
    "v_add_u32 v46, 1, v46\n"
    "buffer_store_dword v46, v40, s[56:59], 0 offen sc0 sc1\n"
    "v_mfma_f32_16x16x128_f8f6f4 v[172:175], v[96:103], a[120:127], v[172:175] cbsz:1 blgp:1\n"
    "buffer_load_dwordx4 v25, s[44:47], s82 offen offset:0 sc0 sc1 lds\n"
    "s_waitcnt lgkmcnt(6)\n"
    "v_mfma_f32_16x16x128_f8f6f4 v[168:171], v[104:111], a[128:135], v[168:171] cbsz:1 blgp:1\n"
    "ds_read_b128 v[136:139], v30 offset:24576\n"
    "ds_read_b128 v[140:143], v31 offset:24576\n"
    "v_mfma_f32_16x16x128_f8f6f4 v[172:175], v[104:111], a[136:143], v[172:175] cbsz:1 blgp:1\n"
    "buffer_load_dwordx4 v25, s[44:47], s82 offen offset:1024 sc0 sc1 lds\n"
    "s_waitcnt lgkmcnt(6)\n"
    "v_mfma_f32_16x16x128_f8f6f4 v[168:171], v[112:119], a[144:151], v[168:171] cbsz:1 blgp:1\n"
    "ds_read_b128 v[144:147], v30 offset:26624\n"
    "ds_read_b128 v[148:151], v31 offset:26624\n"
    "v_mfma_f32_16x16x128_f8f6f4 v[172:175], v[112:119], a[152:159], v[172:175] cbsz:1 blgp:1\n"
    "buffer_load_dwordx4 v25, s[44:47], s82 offen offset:2048 sc0 sc1 lds\n"
    "s_waitcnt vmcnt(10)\n"
    "s_cmp_lt_u32 s64, s68\n"
    "s_cbranch_scc0 L_c3%=\n"
    "L_r3%=:\n"
    "s_waitcnt lgkmcnt(6)\n"
    "v_mfma_f32_16x16x128_f8f6f4 v[168:171], v[120:127], a[160:167], v[168:171] cbsz:1 blgp:1\n"
    "s_barrier\n"
    "ds_read_b128 v[96:99], v30 offset:32768\n"
    "ds_read_b128 v[100:103], v31 offset:32768\n"
    "s_add_u32 s64, s64, 1\n"
    "v_mfma_f32_16x16x128_f8f6f4 v[172:175], v[120:127], a[168:175], v[172:175] cbsz:1 blgp:1\n"
    "s_cmp_le_u32 s67, s64\n"
    "s_cbranch_scc1 L_u3_0%=\n"
    "L_v3_0%=:\n"
    "s_waitcnt lgkmcnt(6)\n"
    "v_mfma_f32_16x16x128_f8f6f4 v[168:171], v[128:135], a[176:183], v[168:171] cbsz:1 blgp:1\n"
    "ds_read_b128 v[104:107], v30 offset:34816\n"
    "ds_read_b128 v[108:111], v31 offset:34816\n"
    "v_mfma_f32_16x16x128_f8f6f4 v[172:175], v[128:135], a[184:191], v[172:175] cbsz:1 blgp:1\n"
    "s_add_u32 s74, s70, 7\n"
    "s_cmp_lt_u32 s96, s74\n"
    "s_cbranch_scc0 L_n3%=\n"
    "global_load_dword v42, v38, s[20:21] sc0 sc1\n"
    "L_n3%=:\n"
    "s_waitcnt lgkmcnt(6)\n"
    "v_mfma_f32_16x16x128_f8f6f4 v[168:171], v[136:143], a[192:199], v[168:171] cbsz:1 blgp:1\n"
    "ds_read_b128 v[112:115], v30 offset:36864\n"
    "ds_read_b128 v[116:119], v31 offset:36864\n"
    "v_mfma_f32_16x16x128_f8f6f4 v[172:175], v[136:143], a[200:207], v[172:175] cbsz:1 blgp:1\n"
    "s_waitcnt lgkmcnt(6)\n"
    "v_mfma_f32_16x16x128_f8f6f4 v[168:171], v[144:151], a[208:215], v[168:171] cbsz:1 blgp:1\n"
    "ds_read_b128 v[120:123], v30 offset:38912\n"
    "ds_read_b128 v[124:127], v31 offset:38912\n"
    "v_readlane_b32 s74, v42, 0\n"
    "v_readlane_b32 s75, v42, 16\n"
    "v_readlane_b32 s76, v42, 32\n"
    "v_mfma_f32_16x16x128_f8f6f4 v[172:175], v[144:151], a[216:223], v[172:175] cbsz:1 blgp:1\n"
    "s_min_u32 s74, s74, s75\n"
    "s_min_u32 s96, s76, s74\n"
    "s_min_u32 s79, s96, s69\n"
    "s_add_u32 s74, s64, 1\n"
    "s_min_u32 s74, s74, s79\n"
    "s_waitcnt lgkmcnt(6)\n"
    "v_mfma_f32_16x16x128_f8f6f4 v[176:179], v[96:103], a[0:7], v[176:179] cbsz:1 blgp:1\n"
    "ds_read_b128 v[128:131], v30 offset:40960\n"
    "ds_read_b128 v[132:135], v31 offset:40960\n"
    "s_cmp_lt_u32 s67, s74\n"
    "s_cbranch_scc1 L_e3%=\n"
    "L_f3%=:\n"
    "s_mov_b32 s68, s67\n"
    "v_mfma_f32_16x16x128_f8f6f4 v[180:183], v[96:103], a[8:15], v[180:183] cbsz:1 blgp:1\n"
    "s_cmp_lt_u32 s67, s79\n"
    "s_cselect_b32 s46, s97, 0\n"
    "s_cselect_b32 s87, s72, 155648\n"
    "s_cselect_b32 s73, 0x8000, 0\n"
    "s_addc_u32 s67, s67, 0\n"
    "s_waitcnt lgkmcnt(6)\n"
    "v_mfma_f32_16x16x128_f8f6f4 v[176:179], v[104:111], a[16:23], v[176:179] cbsz:1 blgp:1\n"
    "ds_read_b128 v[136:139], v30 offset:43008\n"
    "ds_read_b128 v[140:143], v31 offset:43008\n"
    "v_mul_f32 v168, v43, v168\n"
    "v_mul_f32 v169, v43, v169\n"
    "v_mul_f32 v170, v43, v170\n"
    "v_mul_f32 v171, v43, v171\n"
    "v_mfma_f32_16x16x128_f8f6f4 v[180:183], v[104:111], a[24:31], v[180:183] cbsz:1 blgp:1\n"
    "global_store_dwordx4 v40, v[168:171], s[60:61] offset:0 sc0 sc1\n"
    "s_waitcnt lgkmcnt(6)\n"
    "v_mfma_f32_16x16x128_f8f6f4 v[176:179], v[112:119], a[32:39], v[176:179] cbsz:1 blgp:1\n"
    "ds_read_b128 v[144:147], v30 offset:45056\n"
    "ds_read_b128 v[148:151], v31 offset:45056\n"
    "v_mul_f32 v172, v43, v172\n"
    "v_mul_f32 v173, v43, v173\n"
    "v_mul_f32 v174, v43, v174\n"
    "v_mul_f32 v175, v43, v175\n"
    "v_mfma_f32_16x16x128_f8f6f4 v[180:183], v[112:119], a[40:47], v[180:183] cbsz:1 blgp:1\n"
    "global_store_dwordx4 v40, v[172:175], s[60:61] offset:1024 sc0 sc1\n"
    "s_waitcnt lgkmcnt(6)\n"
    "v_mfma_f32_16x16x128_f8f6f4 v[176:179], v[120:127], a[48:55], v[176:179] cbsz:1 blgp:1\n"
    "ds_read_b128 v[96:99], v30 offset:47104\n"
    "ds_read_b128 v[100:103], v31 offset:47104\n"
    "s_mov_b32 m0, s87\n"
    "s_add_u32 s82, s95, 0x1000\n"
    "v_mfma_f32_16x16x128_f8f6f4 v[180:183], v[120:127], a[56:63], v[180:183] cbsz:1 blgp:1\n"
    "buffer_load_dwordx4 v25, s[44:47], s95 offen offset:0 sc0 sc1 lds\n"
    "s_add_u32 s72, s72, s73\n"
    "s_and_b32 s72, s72, 0x1ffff\n"
    "s_waitcnt lgkmcnt(6)\n"
    "v_mfma_f32_16x16x128_f8f6f4 v[176:179], v[128:135], a[64:71], v[176:179] cbsz:1 blgp:1\n"
    "ds_read_b128 v[104:107], v30 offset:49152\n"
    "ds_read_b128 v[108:111], v31 offset:49152\n"
    "s_add_u32 s69, s64, 3\n"
    "v_mfma_f32_16x16x128_f8f6f4 v[180:183], v[128:135], a[72:79], v[180:183] cbsz:1 blgp:1\n"
    "buffer_load_dwordx4 v25, s[44:47], s95 offen offset:1024 sc0 sc1 lds\n"
    "s_waitcnt lgkmcnt(6)\n"
    "v_mfma_f32_16x16x128_f8f6f4 v[176:179], v[136:143], a[80:87], v[176:179] cbsz:1 blgp:1\n"
    "ds_read_b128 v[112:115], v30 offset:51200\n"
    "ds_read_b128 v[116:119], v31 offset:51200\n"
    "v_mfma_f32_16x16x128_f8f6f4 v[180:183], v[136:143], a[88:95], v[180:183] cbsz:1 blgp:1\n"
    "buffer_load_dwordx4 v25, s[44:47], s95 offen offset:2048 sc0 sc1 lds\n"
    "s_waitcnt lgkmcnt(6)\n"
    "v_mfma_f32_16x16x128_f8f6f4 v[176:179], v[144:151], a[96:103], v[176:179] cbsz:1 blgp:1\n"
    "ds_read_b128 v[120:123], v30 offset:53248\n"
    "ds_read_b128 v[124:127], v31 offset:53248\n"
    "v_mfma_f32_16x16x128_f8f6f4 v[180:183], v[144:151], a[104:111], v[180:183] cbsz:1 blgp:1\n"
    "buffer_load_dwordx4 v25, s[44:47], s95 offen offset:3072 sc0 sc1 lds\n"
    "s_lshl1_add_u32 s95, s73, s95\n"
    "s_add_u32 m0, s87, 0x1000\n"
    "s_waitcnt lgkmcnt(6)\n"
    "v_mfma_f32_16x16x128_f8f6f4 v[176:179], v[96:103], a[112:119], v[176:179] cbsz:1 blgp:1\n"
    "ds_read_b128 v[128:131], v30 offset:55296\n"
    "ds_read_b128 v[132:135], v31 offset:55296\n"
    "s_waitcnt vmcnt(14)\n"
    "s_barrier\n"
    "v_add_u32 v46, 1, v46\n"
    "buffer_store_dword v46, v40, s[56:59], 0 offen sc0 sc1\n"
    "v_mfma_f32_16x16x128_f8f6f4 v[180:183], v[96:103], a[120:127], v[180:183] cbsz:1 blgp:1\n"
    "buffer_load_dwordx4 v25, s[44:47], s82 offen offset:0 sc0 sc1 lds\n"
    "s_waitcnt lgkmcnt(6)\n"
    "v_mfma_f32_16x16x128_f8f6f4 v[176:179], v[104:111], a[128:135], v[176:179] cbsz:1 blgp:1\n"
    "ds_read_b128 v[136:139], v30 offset:57344\n"
    "ds_read_b128 v[140:143], v31 offset:57344\n"
    "v_mfma_f32_16x16x128_f8f6f4 v[180:183], v[104:111], a[136:143], v[180:183] cbsz:1 blgp:1\n"
    "buffer_load_dwordx4 v25, s[44:47], s82 offen offset:1024 sc0 sc1 lds\n"
    "s_waitcnt lgkmcnt(6)\n"
    "v_mfma_f32_16x16x128_f8f6f4 v[176:179], v[112:119], a[144:151], v[176:179] cbsz:1 blgp:1\n"
    "ds_read_b128 v[144:147], v30 offset:59392\n"
    "ds_read_b128 v[148:151], v31 offset:59392\n"
    "v_mfma_f32_16x16x128_f8f6f4 v[180:183], v[112:119], a[152:159], v[180:183] cbsz:1 blgp:1\n"
    "buffer_load_dwordx4 v25, s[44:47], s82 offen offset:2048 sc0 sc1 lds\n"
    "s_waitcnt vmcnt(10)\n"
    "s_cmp_lt_u32 s64, s68\n"
    "s_cbranch_scc0 L_c4%=\n"
    "L_r4%=:\n"
    "s_waitcnt lgkmcnt(6)\n"
    "v_mfma_f32_16x16x128_f8f6f4 v[176:179], v[120:127], a[160:167], v[176:179] cbsz:1 blgp:1\n"
    "s_barrier\n"
    "ds_read_b128 v[96:99], v26 offset:0\n"
    "ds_read_b128 v[100:103], v27 offset:0\n"
    "s_add_u32 s64, s64, 1\n"
    "v_mfma_f32_16x16x128_f8f6f4 v[180:183], v[120:127], a[168:175], v[180:183] cbsz:1 blgp:1\n"
    "s_cmp_le_u32 s67, s64\n"
    "s_cbranch_scc1 L_u4_0%=\n"
    "L_v4_0%=:\n"
    "s_waitcnt lgkmcnt(6)\n"
    "v_mfma_f32_16x16x128_f8f6f4 v[176:179], v[128:135], a[176:183], v[176:179] cbsz:1 blgp:1\n"
    "ds_read_b128 v[104:107], v26 offset:2048\n"
    "ds_read_b128 v[108:111], v27 offset:2048\n"
    "v_mfma_f32_16x16x128_f8f6f4 v[180:183], v[128:135], a[184:191], v[180:183] cbsz:1 blgp:1\n"
    "s_add_u32 s74, s70, 8\n"
    "s_cmp_lt_u32 s96, s74\n"
    "s_cbranch_scc0 L_n4%=\n"
    "global_load_dword v42, v38, s[20:21] sc0 sc1\n"
    "L_n4%=:\n"
    "s_waitcnt lgkmcnt(6)\n"
    "v_mfma_f32_16x16x128_f8f6f4 v[176:179], v[136:143], a[192:199], v[176:179] cbsz:1 blgp:1\n"
    "ds_read_b128 v[112:115], v26 offset:4096\n"
    "ds_read_b128 v[116:119], v27 offset:4096\n"
    "v_mfma_f32_16x16x128_f8f6f4 v[180:183], v[136:143], a[200:207], v[180:183] cbsz:1 blgp:1\n"
    "s_cmp_eq_u32 s12, 0\n"
    "s_cbranch_scc0 L_s4%=\n"
    "s_load_dword s62, s[20:21], s71 glc\n"
    "L_s4%=:\n"
    "s_waitcnt lgkmcnt(6)\n"
    "v_mfma_f32_16x16x128_f8f6f4 v[176:179], v[144:151], a[208:215], v[176:179] cbsz:1 blgp:1\n"
    "ds_read_b128 v[120:123], v26 offset:6144\n"
    "ds_read_b128 v[124:127], v27 offset:6144\n"
    "v_readlane_b32 s74, v42, 0\n"
    "v_readlane_b32 s75, v42, 16\n"
    "v_readlane_b32 s76, v42, 32\n"
    "v_mfma_f32_16x16x128_f8f6f4 v[180:183], v[144:151], a[216:223], v[180:183] cbsz:1 blgp:1\n"
    "s_min_u32 s74, s74, s75\n"
    "s_min_u32 s96, s76, s74\n"
    "s_min_u32 s79, s96, s69\n"
    "s_add_u32 s74, s64, 1\n"
    "s_min_u32 s74, s74, s79\n"
    "s_waitcnt lgkmcnt(6)\n"
    "v_mfma_f32_16x16x128_f8f6f4 v[184:187], v[96:103], a[0:7], v[184:187] cbsz:1 blgp:1\n"
    "ds_read_b128 v[128:131], v26 offset:8192\n"
    "ds_read_b128 v[132:135], v27 offset:8192\n"
    "s_cmp_lt_u32 s67, s74\n"
    "s_cbranch_scc1 L_e4%=\n"
    "L_f4%=:\n"
    "s_mov_b32 s68, s67\n"
    "v_mfma_f32_16x16x128_f8f6f4 v[188:191], v[96:103], a[8:15], v[188:191] cbsz:1 blgp:1\n"
    "s_cmp_lt_u32 s67, s79\n"
    "s_cselect_b32 s46, s97, 0\n"
    "s_cselect_b32 s87, s72, 155648\n"
    "s_cselect_b32 s73, 0x8000, 0\n"
    "s_addc_u32 s67, s67, 0\n"
    "s_waitcnt lgkmcnt(6)\n"
    "v_mfma_f32_16x16x128_f8f6f4 v[184:187], v[104:111], a[16:23], v[184:187] cbsz:1 blgp:1\n"
    "ds_read_b128 v[136:139], v26 offset:10240\n"
    "ds_read_b128 v[140:143], v27 offset:10240\n"
    "v_mul_f32 v176, v43, v176\n"
    "v_mul_f32 v177, v43, v177\n"
    "v_mul_f32 v178, v43, v178\n"
    "v_mul_f32 v179, v43, v179\n"
    "v_mfma_f32_16x16x128_f8f6f4 v[188:191], v[104:111], a[24:31], v[188:191] cbsz:1 blgp:1\n"
    "global_store_dwordx4 v40, v[176:179], s[60:61] offset:2048 sc0 sc1\n"
    "s_waitcnt lgkmcnt(6)\n"
    "v_mfma_f32_16x16x128_f8f6f4 v[184:187], v[112:119], a[32:39], v[184:187] cbsz:1 blgp:1\n"
    "ds_read_b128 v[144:147], v26 offset:12288\n"
    "ds_read_b128 v[148:151], v27 offset:12288\n"
    "v_mul_f32 v180, v43, v180\n"
    "v_mul_f32 v181, v43, v181\n"
    "v_mul_f32 v182, v43, v182\n"
    "v_mul_f32 v183, v43, v183\n"
    "v_mfma_f32_16x16x128_f8f6f4 v[188:191], v[112:119], a[40:47], v[188:191] cbsz:1 blgp:1\n"
    "global_store_dwordx4 v40, v[180:183], s[60:61] offset:3072 sc0 sc1\n"
    "s_waitcnt lgkmcnt(6)\n"
    "v_mfma_f32_16x16x128_f8f6f4 v[184:187], v[120:127], a[48:55], v[184:187] cbsz:1 blgp:1\n"
    "ds_read_b128 v[96:99], v26 offset:14336\n"
    "ds_read_b128 v[100:103], v27 offset:14336\n"
    "s_mov_b32 m0, s87\n"
    "s_add_u32 s82, s95, 0x1000\n"
    "v_mfma_f32_16x16x128_f8f6f4 v[188:191], v[120:127], a[56:63], v[188:191] cbsz:1 blgp:1\n"
    "buffer_load_dwordx4 v25, s[44:47], s95 offen offset:0 sc0 sc1 lds\n"
    "s_add_u32 s72, s72, s73\n"
    "s_and_b32 s72, s72, 0x1ffff\n"
    "s_waitcnt lgkmcnt(6)\n"
    "v_mfma_f32_16x16x128_f8f6f4 v[184:187], v[128:135], a[64:71], v[184:187] cbsz:1 blgp:1\n"
    "ds_read_b128 v[104:107], v26 offset:16384\n"
    "ds_read_b128 v[108:111], v27 offset:16384\n"
    "s_add_u32 s69, s64, 3\n"
    "v_mfma_f32_16x16x128_f8f6f4 v[188:191], v[128:135], a[72:79], v[188:191] cbsz:1 blgp:1\n"
    "buffer_load_dwordx4 v25, s[44:47], s95 offen offset:1024 sc0 sc1 lds\n"
    "s_waitcnt lgkmcnt(6)\n"
    "v_mfma_f32_16x16x128_f8f6f4 v[184:187], v[136:143], a[80:87], v[184:187] cbsz:1 blgp:1\n"
    "ds_read_b128 v[112:115], v26 offset:18432\n"
    "ds_read_b128 v[116:119], v27 offset:18432\n"
    "v_mfma_f32_16x16x128_f8f6f4 v[188:191], v[136:143], a[88:95], v[188:191] cbsz:1 blgp:1\n"
    "buffer_load_dwordx4 v25, s[44:47], s95 offen offset:2048 sc0 sc1 lds\n"
    "s_waitcnt lgkmcnt(6)\n"
    "v_mfma_f32_16x16x128_f8f6f4 v[184:187], v[144:151], a[96:103], v[184:187] cbsz:1 blgp:1\n"
    "ds_read_b128 v[120:123], v26 offset:20480\n"
    "ds_read_b128 v[124:127], v27 offset:20480\n"
    "v_mfma_f32_16x16x128_f8f6f4 v[188:191], v[144:151], a[104:111], v[188:191] cbsz:1 blgp:1\n"
    "buffer_load_dwordx4 v25, s[44:47], s95 offen offset:3072 sc0 sc1 lds\n"
    "s_lshl1_add_u32 s95, s73, s95\n"
    "s_add_u32 m0, s87, 0x1000\n"
    "s_waitcnt lgkmcnt(6)\n"
    "v_mfma_f32_16x16x128_f8f6f4 v[184:187], v[96:103], a[112:119], v[184:187] cbsz:1 blgp:1\n"
    "ds_read_b128 v[128:131], v26 offset:22528\n"
    "ds_read_b128 v[132:135], v27 offset:22528\n"
    "s_waitcnt vmcnt(14)\n"
    "s_barrier\n"
    "v_add_u32 v46, 1, v46\n"
    "buffer_store_dword v46, v40, s[56:59], 0 offen sc0 sc1\n"
    "v_mfma_f32_16x16x128_f8f6f4 v[188:191], v[96:103], a[120:127], v[188:191] cbsz:1 blgp:1\n"
    "buffer_load_dwordx4 v25, s[44:47], s82 offen offset:0 sc0 sc1 lds\n"
    "s_waitcnt lgkmcnt(6)\n"
    "v_mfma_f32_16x16x128_f8f6f4 v[184:187], v[104:111], a[128:135], v[184:187] cbsz:1 blgp:1\n"
    "ds_read_b128 v[136:139], v26 offset:24576\n"
    "ds_read_b128 v[140:143], v27 offset:24576\n"
    "v_mfma_f32_16x16x128_f8f6f4 v[188:191], v[104:111], a[136:143], v[188:191] cbsz:1 blgp:1\n"
    "buffer_load_dwordx4 v25, s[44:47], s82 offen offset:1024 sc0 sc1 lds\n"
    "s_waitcnt lgkmcnt(6)\n"
    "v_mfma_f32_16x16x128_f8f6f4 v[184:187], v[112:119], a[144:151], v[184:187] cbsz:1 blgp:1\n"
    "ds_read_b128 v[144:147], v26 offset:26624\n"
    "ds_read_b128 v[148:151], v27 offset:26624\n"
    "s_cmp_ge_u32 s62, s89\n"
    "s_cbranch_scc0 L_p4%=\n"
    "L_q4%=:\n"
    "v_mfma_f32_16x16x128_f8f6f4 v[188:191], v[112:119], a[152:159], v[188:191] cbsz:1 blgp:1\n"
    "buffer_load_dwordx4 v25, s[44:47], s82 offen offset:2048 sc0 sc1 lds\n"
    "s_waitcnt vmcnt(10)\n"
    "s_cmp_lt_u32 s64, s68\n"
    "s_cbranch_scc0 L_c5%=\n"
    "L_r5%=:\n"
    "s_waitcnt lgkmcnt(6)\n"
    "v_mfma_f32_16x16x128_f8f6f4 v[184:187], v[120:127], a[160:167], v[184:187] cbsz:1 blgp:1\n"
    "s_barrier\n"
    "ds_read_b128 v[96:99], v26 offset:32768\n"
    "ds_read_b128 v[100:103], v27 offset:32768\n"
    "s_add_u32 s64, s64, 1\n"
    "v_mfma_f32_16x16x128_f8f6f4 v[188:191], v[120:127], a[168:175], v[188:191] cbsz:1 blgp:1\n"
    "buffer_load_dwordx4 v[152:155], v40, s[48:51], s93 offen offset:0 sc0 sc1\n"
    "s_cmp_le_u32 s67, s64\n"
    "s_cbranch_scc1 L_u5_0%=\n"
    "L_v5_0%=:\n"
    "s_waitcnt lgkmcnt(6)\n"
    "v_mfma_f32_16x16x128_f8f6f4 v[184:187], v[128:135], a[176:183], v[184:187] cbsz:1 blgp:1\n"
    "ds_read_b128 v[104:107], v26 offset:34816\n"
    "ds_read_b128 v[108:111], v27 offset:34816\n"
    "buffer_load_dwordx4 v[156:159], v40, s[48:51], s93 offen offset:1024 sc0 sc1\n"
    "s_sub_u32 s89, s89, s94\n"
    "s_add_u32 s93, s93, 0x800\n"
    "v_mfma_f32_16x16x128_f8f6f4 v[188:191], v[128:135], a[184:191], v[188:191] cbsz:1 blgp:1\n"
    "s_add_u32 s74, s70, 9\n"
    "s_cmp_lt_u32 s96, s74\n"
    "s_cbranch_scc0 L_n5%=\n"
    "global_load_dword v42, v38, s[20:21] sc0 sc1\n"
    "L_n5%=:\n"
    "s_waitcnt lgkmcnt(6)\n"
    "v_mfma_f32_16x16x128_f8f6f4 v[184:187], v[136:143], a[192:199], v[184:187] cbsz:1 blgp:1\n"
    "ds_read_b128 v[112:115], v26 offset:36864\n"
    "ds_read_b128 v[116:119], v27 offset:36864\n"
    "v_mfma_f32_16x16x128_f8f6f4 v[188:191], v[136:143], a[200:207], v[188:191] cbsz:1 blgp:1\n"
    "s_cmp_eq_u32 s12, 0\n"
    "s_cbranch_scc0 L_s5%=\n"
    "s_load_dword s62, s[20:21], s71 glc\n"
    "L_s5%=:\n"
    "s_waitcnt lgkmcnt(6)\n"
    "v_mfma_f32_16x16x128_f8f6f4 v[184:187], v[144:151], a[208:215], v[184:187] cbsz:1 blgp:1\n"
    "ds_read_b128 v[120:123], v26 offset:38912\n"
    "ds_read_b128 v[124:127], v27 offset:38912\n"
    "v_readlane_b32 s74, v42, 0\n"
    "v_readlane_b32 s75, v42, 16\n"
    "v_readlane_b32 s76, v42, 32\n"
    "v_mfma_f32_16x16x128_f8f6f4 v[188:191], v[144:151], a[216:223], v[188:191] cbsz:1 blgp:1\n"
    "s_min_u32 s74, s74, s75\n"
    "s_min_u32 s96, s76, s74\n"
    "s_min_u32 s79, s96, s69\n"
    "s_add_u32 s74, s64, 1\n"
    "s_min_u32 s74, s74, s79\n"
    "s_waitcnt lgkmcnt(6)\n"
    "v_mfma_f32_16x16x128_f8f6f4 v[192:195], v[96:103], a[0:7], v[192:195] cbsz:1 blgp:1\n"
    "ds_read_b128 v[128:131], v26 offset:40960\n"
    "ds_read_b128 v[132:135], v27 offset:40960\n"
    "s_cmp_lt_u32 s67, s74\n"
    "s_cbranch_scc1 L_e5%=\n"
    "L_f5%=:\n"
    "s_mov_b32 s68, s67\n"
    "v_mfma_f32_16x16x128_f8f6f4 v[196:199], v[96:103], a[8:15], v[196:199] cbsz:1 blgp:1\n"
    "s_cmp_lt_u32 s67, s79\n"
    "s_cselect_b32 s46, s97, 0\n"
    "s_cselect_b32 s87, s72, 155648\n"
    "s_cselect_b32 s73, 0x8000, 0\n"
    "s_addc_u32 s67, s67, 0\n"
    "s_waitcnt lgkmcnt(6)\n"
    "v_mfma_f32_16x16x128_f8f6f4 v[192:195], v[104:111], a[16:23], v[192:195] cbsz:1 blgp:1\n"
    "ds_read_b128 v[136:139], v26 offset:43008\n"
    "ds_read_b128 v[140:143], v27 offset:43008\n"
    "v_mul_f32 v184, v43, v184\n"
    "v_mul_f32 v185, v43, v185\n"
    "v_mul_f32 v186, v43, v186\n"
    "v_mul_f32 v187, v43, v187\n"
    "v_mfma_f32_16x16x128_f8f6f4 v[196:199], v[104:111], a[24:31], v[196:199] cbsz:1 blgp:1\n"
    "global_store_dwordx4 v39, v[184:187], s[60:61] offset:-4096 sc0 sc1\n"
    "s_waitcnt lgkmcnt(6)\n"
    "v_mfma_f32_16x16x128_f8f6f4 v[192:195], v[112:119], a[32:39], v[192:195] cbsz:1 blgp:1\n"
    "ds_read_b128 v[144:147], v26 offset:45056\n"
    "ds_read_b128 v[148:151], v27 offset:45056\n"
    "v_mul_f32 v188, v43, v188\n"
    "v_mul_f32 v189, v43, v189\n"
    "v_mul_f32 v190, v43, v190\n"
    "v_mul_f32 v191, v43, v191\n"
    "v_mfma_f32_16x16x128_f8f6f4 v[196:199], v[112:119], a[40:47], v[196:199] cbsz:1 blgp:1\n"
    "global_store_dwordx4 v39, v[188:191], s[60:61] offset:-3072 sc0 sc1\n"
    "s_waitcnt lgkmcnt(6)\n"
    "v_mfma_f32_16x16x128_f8f6f4 v[192:195], v[120:127], a[48:55], v[192:195] cbsz:1 blgp:1\n"
    "ds_read_b128 v[96:99], v26 offset:47104\n"
    "ds_read_b128 v[100:103], v27 offset:47104\n"
    "s_mov_b32 m0, s87\n"
    "s_add_u32 s82, s95, 0x1000\n"
    "v_mfma_f32_16x16x128_f8f6f4 v[196:199], v[120:127], a[56:63], v[196:199] cbsz:1 blgp:1\n"
    "buffer_load_dwordx4 v25, s[44:47], s95 offen offset:0 sc0 sc1 lds\n"
    "s_add_u32 s72, s72, s73\n"
    "s_and_b32 s72, s72, 0x1ffff\n"
    "s_waitcnt lgkmcnt(6)\n"
    "v_mfma_f32_16x16x128_f8f6f4 v[192:195], v[128:135], a[64:71], v[192:195] cbsz:1 blgp:1\n"
    "ds_read_b128 v[104:107], v26 offset:49152\n"
    "ds_read_b128 v[108:111], v27 offset:49152\n"
    "s_add_u32 s69, s64, 3\n"
    "v_mfma_f32_16x16x128_f8f6f4 v[196:199], v[128:135], a[72:79], v[196:199] cbsz:1 blgp:1\n"
    "buffer_load_dwordx4 v25, s[44:47], s95 offen offset:1024 sc0 sc1 lds\n"
    "s_waitcnt lgkmcnt(6)\n"
    "v_mfma_f32_16x16x128_f8f6f4 v[192:195], v[136:143], a[80:87], v[192:195] cbsz:1 blgp:1\n"
    "ds_read_b128 v[112:115], v26 offset:51200\n"
    "ds_read_b128 v[116:119], v27 offset:51200\n"
    "v_mfma_f32_16x16x128_f8f6f4 v[196:199], v[136:143], a[88:95], v[196:199] cbsz:1 blgp:1\n"
    "buffer_load_dwordx4 v25, s[44:47], s95 offen offset:2048 sc0 sc1 lds\n"
    "s_waitcnt lgkmcnt(6)\n"
    "v_mfma_f32_16x16x128_f8f6f4 v[192:195], v[144:151], a[96:103], v[192:195] cbsz:1 blgp:1\n"
    "ds_read_b128 v[120:123], v26 offset:53248\n"
    "ds_read_b128 v[124:127], v27 offset:53248\n"
    "v_mfma_f32_16x16x128_f8f6f4 v[196:199], v[144:151], a[104:111], v[196:199] cbsz:1 blgp:1\n"
    "buffer_load_dwordx4 v25, s[44:47], s95 offen offset:3072 sc0 sc1 lds\n"
    "s_lshl1_add_u32 s95, s73, s95\n"
    "s_add_u32 m0, s87, 0x1000\n"
    "s_waitcnt lgkmcnt(6)\n"
    "v_mfma_f32_16x16x128_f8f6f4 v[192:195], v[96:103], a[112:119], v[192:195] cbsz:1 blgp:1\n"
    "ds_read_b128 v[128:131], v26 offset:55296\n"
    "ds_read_b128 v[132:135], v27 offset:55296\n"
    "s_waitcnt vmcnt(16)\n"
    "s_barrier\n"
    "v_add_u32 v46, 1, v46\n"
    "buffer_store_dword v46, v40, s[56:59], 0 offen sc0 sc1\n"
    "v_mfma_f32_16x16x128_f8f6f4 v[196:199], v[96:103], a[120:127], v[196:199] cbsz:1 blgp:1\n"
    "buffer_load_dwordx4 v25, s[44:47], s82 offen offset:0 sc0 sc1 lds\n"
    "s_waitcnt lgkmcnt(6)\n"
    "v_mfma_f32_16x16x128_f8f6f4 v[192:195], v[104:111], a[128:135], v[192:195] cbsz:1 blgp:1\n"
    "ds_read_b128 v[136:139], v26 offset:57344\n"
    "ds_read_b128 v[140:143], v27 offset:57344\n"
    "v_mfma_f32_16x16x128_f8f6f4 v[196:199], v[104:111], a[136:143], v[196:199] cbsz:1 blgp:1\n"
    "buffer_load_dwordx4 v25, s[44:47], s82 offen offset:1024 sc0 sc1 lds\n"
    "s_waitcnt lgkmcnt(6)\n"
    "v_mfma_f32_16x16x128_f8f6f4 v[192:195], v[112:119], a[144:151], v[192:195] cbsz:1 blgp:1\n"
    "ds_read_b128 v[144:147], v26 offset:59392\n"
    "ds_read_b128 v[148:151], v27 offset:59392\n"
    "s_cmp_ge_u32 s62, s89\n"
    "s_cbranch_scc0 L_p5%=\n"
    "L_q5%=:\n"
    "v_mfma_f32_16x16x128_f8f6f4 v[196:199], v[112:119], a[152:159], v[196:199] cbsz:1 blgp:1\n"
    "buffer_load_dwordx4 v25, s[44:47], s82 offen offset:2048 sc0 sc1 lds\n"
    "s_waitcnt vmcnt(10)\n"
    "s_cmp_lt_u32 s64, s68\n"
    "s_cbranch_scc0 L_c6%=\n"
    "L_r6%=:\n"
    "s_waitcnt lgkmcnt(6)\n"
    "v_mfma_f32_16x16x128_f8f6f4 v[192:195], v[120:127], a[160:167], v[192:195] cbsz:1 blgp:1\n"
    "s_barrier\n"
    "ds_read_b128 v[96:99], v30 offset:0\n"
    "ds_read_b128 v[100:103], v31 offset:0\n"
    "s_add_u32 s64, s64, 1\n"
    "v_mfma_f32_16x16x128_f8f6f4 v[196:199], v[120:127], a[168:175], v[196:199] cbsz:1 blgp:1\n"
    "buffer_load_dwordx4 v[160:163], v40, s[48:51], s93 offen offset:0 sc0 sc1\n"
    "s_cmp_le_u32 s67, s64\n"
    "s_cbranch_scc1 L_u6_0%=\n"
    "L_v6_0%=:\n"
    "s_waitcnt lgkmcnt(6)\n"
    "v_mfma_f32_16x16x128_f8f6f4 v[192:195], v[128:135], a[176:183], v[192:195] cbsz:1 blgp:1\n"
    "ds_read_b128 v[104:107], v30 offset:2048\n"
    "ds_read_b128 v[108:111], v31 offset:2048\n"
    "buffer_load_dwordx4 v[164:167], v40, s[48:51], s93 offen offset:1024 sc0 sc1\n"
    "s_sub_u32 s89, s89, s94\n"
    "s_add_u32 s93, s93, 0x800\n"
    "v_mfma_f32_16x16x128_f8f6f4 v[196:199], v[128:135], a[184:191], v[196:199] cbsz:1 blgp:1\n"
    "s_add_u32 s74, s70, 10\n"
    "s_cmp_lt_u32 s96, s74\n"
    "s_cbranch_scc0 L_n6%=\n"
    "global_load_dword v42, v38, s[20:21] sc0 sc1\n"
    "L_n6%=:\n"
    "s_waitcnt lgkmcnt(6)\n"
    "v_mfma_f32_16x16x128_f8f6f4 v[192:195], v[136:143], a[192:199], v[192:195] cbsz:1 blgp:1\n"
    "ds_read_b128 v[112:115], v30 offset:4096\n"
    "ds_read_b128 v[116:119], v31 offset:4096\n"
    "v_mfma_f32_16x16x128_f8f6f4 v[196:199], v[136:143], a[200:207], v[196:199] cbsz:1 blgp:1\n"
    "s_cmp_eq_u32 s12, 0\n"
    "s_cbranch_scc0 L_s6%=\n"
    "s_load_dword s62, s[20:21], s71 glc\n"
    "L_s6%=:\n"
    "s_waitcnt lgkmcnt(6)\n"
    "v_mfma_f32_16x16x128_f8f6f4 v[192:195], v[144:151], a[208:215], v[192:195] cbsz:1 blgp:1\n"
    "ds_read_b128 v[120:123], v30 offset:6144\n"
    "ds_read_b128 v[124:127], v31 offset:6144\n"
    "v_readlane_b32 s74, v42, 0\n"
    "v_readlane_b32 s75, v42, 16\n"
    "v_readlane_b32 s76, v42, 32\n"
    "v_mfma_f32_16x16x128_f8f6f4 v[196:199], v[144:151], a[216:223], v[196:199] cbsz:1 blgp:1\n"
    "s_min_u32 s74, s74, s75\n"
    "s_min_u32 s96, s76, s74\n"
    "s_min_u32 s79, s96, s69\n"
    "s_add_u32 s74, s64, 1\n"
    "s_min_u32 s74, s74, s79\n"
    "s_waitcnt lgkmcnt(6)\n"
    "v_mfma_f32_16x16x128_f8f6f4 v[200:203], v[96:103], a[0:7], v[200:203] cbsz:1 blgp:1\n"
    "ds_read_b128 v[128:131], v30 offset:8192\n"
    "ds_read_b128 v[132:135], v31 offset:8192\n"
    "s_cmp_lt_u32 s67, s74\n"
    "s_cbranch_scc1 L_e6%=\n"
    "L_f6%=:\n"
    "s_mov_b32 s68, s67\n"
    "v_mfma_f32_16x16x128_f8f6f4 v[204:207], v[96:103], a[8:15], v[204:207] cbsz:1 blgp:1\n"
    "s_cmp_lt_u32 s67, s79\n"
    "s_cselect_b32 s46, s97, 0\n"
    "s_cselect_b32 s87, s72, 155648\n"
    "s_cselect_b32 s73, 0x8000, 0\n"
    "s_addc_u32 s67, s67, 0\n"
    "s_waitcnt lgkmcnt(6)\n"
    "v_mfma_f32_16x16x128_f8f6f4 v[200:203], v[104:111], a[16:23], v[200:203] cbsz:1 blgp:1\n"
    "ds_read_b128 v[136:139], v30 offset:10240\n"
    "ds_read_b128 v[140:143], v31 offset:10240\n"
    "v_fma_f32 v192, v192, v43, v152\n"
    "v_fma_f32 v193, v193, v43, v153\n"
    "v_fma_f32 v194, v194, v43, v154\n"
    "v_fma_f32 v195, v195, v43, v155\n"
    "v_mfma_f32_16x16x128_f8f6f4 v[204:207], v[104:111], a[24:31], v[204:207] cbsz:1 blgp:1\n"
    "v_fma_f32 v196, v196, v43, v156\n"
    "v_fma_f32 v197, v197, v43, v157\n"
    "v_fma_f32 v198, v198, v43, v158\n"
    "v_fma_f32 v199, v199, v43, v159\n"
    "s_waitcnt lgkmcnt(6)\n"
    "v_mfma_f32_16x16x128_f8f6f4 v[200:203], v[112:119], a[32:39], v[200:203] cbsz:1 blgp:1\n"
    "ds_read_b128 v[144:147], v30 offset:12288\n"
    "ds_read_b128 v[148:151], v31 offset:12288\n"
    "v_permlane16_swap_b32 v192, v196\n"
    "v_permlane16_swap_b32 v193, v197\n"
    "v_permlane16_swap_b32 v194, v198\n"
    "v_permlane16_swap_b32 v195, v199\n"
    "v_mfma_f32_16x16x128_f8f6f4 v[204:207], v[112:119], a[40:47], v[204:207] cbsz:1 blgp:1\n"
    "global_store_dword v64, v192, s[22:23] sc0 sc1\n"
    "s_waitcnt lgkmcnt(6)\n"
    "v_mfma_f32_16x16x128_f8f6f4 v[200:203], v[120:127], a[48:55], v[200:203] cbsz:1 blgp:1\n"
    "ds_read_b128 v[96:99], v30 offset:14336\n"
    "ds_read_b128 v[100:103], v31 offset:14336\n"
    "global_store_dword v68, v196, s[22:23] sc0 sc1\n"
    "s_mov_b32 m0, s87\n"
    "s_add_u32 s82, s95, 0x1000\n"
    "v_mfma_f32_16x16x128_f8f6f4 v[204:207], v[120:127], a[56:63], v[204:207] cbsz:1 blgp:1\n"
    "buffer_load_dwordx4 v25, s[44:47], s95 offen offset:0 sc0 sc1 lds\n"
    "s_add_u32 s72, s72, s73\n"
    "s_and_b32 s72, s72, 0x1ffff\n"
    "s_waitcnt lgkmcnt(6)\n"
    "v_mfma_f32_16x16x128_f8f6f4 v[200:203], v[128:135], a[64:71], v[200:203] cbsz:1 blgp:1\n"
    "ds_read_b128 v[104:107], v30 offset:16384\n"
    "ds_read_b128 v[108:111], v31 offset:16384\n"
    "global_store_dword v65, v193, s[22:23] sc0 sc1\n"
    "s_add_u32 s69, s64, 3\n"
    "v_mfma_f32_16x16x128_f8f6f4 v[204:207], v[128:135], a[72:79], v[204:207] cbsz:1 blgp:1\n"
    "buffer_load_dwordx4 v25, s[44:47], s95 offen offset:1024 sc0 sc1 lds\n"
    "s_waitcnt lgkmcnt(6)\n"
    "v_mfma_f32_16x16x128_f8f6f4 v[200:203], v[136:143], a[80:87], v[200:203] cbsz:1 blgp:1\n"
    "ds_read_b128 v[112:115], v30 offset:18432\n"
    "ds_read_b128 v[116:119], v31 offset:18432\n"
    "global_store_dword v69, v197, s[22:23] sc0 sc1\n"
    "v_mfma_f32_16x16x128_f8f6f4 v[204:207], v[136:143], a[88:95], v[204:207] cbsz:1 blgp:1\n"
    "buffer_load_dwordx4 v25, s[44:47], s95 offen offset:2048 sc0 sc1 lds\n"
    "s_waitcnt lgkmcnt(6)\n"
    "v_mfma_f32_16x16x128_f8f6f4 v[200:203], v[144:151], a[96:103], v[200:203] cbsz:1 blgp:1\n"
    "ds_read_b128 v[120:123], v30 offset:20480\n"
    "ds_read_b128 v[124:127], v31 offset:20480\n"
    "global_store_dword v66, v194, s[22:23] sc0 sc1\n"
    "v_mfma_f32_16x16x128_f8f6f4 v[204:207], v[144:151], a[104:111], v[204:207] cbsz:1 blgp:1\n"
    "buffer_load_dwordx4 v25, s[44:47], s95 offen offset:3072 sc0 sc1 lds\n"
    "s_lshl1_add_u32 s95, s73, s95\n"
    "s_add_u32 m0, s87, 0x1000\n"
    "s_waitcnt lgkmcnt(6)\n"
    "v_mfma_f32_16x16x128_f8f6f4 v[200:203], v[96:103], a[112:119], v[200:203] cbsz:1 blgp:1\n"
    "ds_read_b128 v[128:131], v30 offset:22528\n"
    "ds_read_b128 v[132:135], v31 offset:22528\n"
    "global_store_dword v70, v198, s[22:23] sc0 sc1\n"
    "s_waitcnt vmcnt(20)\n"
    "s_barrier\n"
    "v_add_u32 v46, 1, v46\n"
    "buffer_store_dword v46, v40, s[56:59], 0 offen sc0 sc1\n"
    "v_mfma_f32_16x16x128_f8f6f4 v[204:207], v[96:103], a[120:127], v[204:207] cbsz:1 blgp:1\n"
    "buffer_load_dwordx4 v25, s[44:47], s82 offen offset:0 sc0 sc1 lds\n"
    "s_waitcnt lgkmcnt(6)\n"
    "v_mfma_f32_16x16x128_f8f6f4 v[200:203], v[104:111], a[128:135], v[200:203] cbsz:1 blgp:1\n"
    "ds_read_b128 v[136:139], v30 offset:24576\n"
    "ds_read_b128 v[140:143], v31 offset:24576\n"
    "global_store_dword v67, v195, s[22:23] sc0 sc1\n"
    "v_mfma_f32_16x16x128_f8f6f4 v[204:207], v[104:111], a[136:143], v[204:207] cbsz:1 blgp:1\n"
    "buffer_load_dwordx4 v25, s[44:47], s82 offen offset:1024 sc0 sc1 lds\n"
    "s_waitcnt lgkmcnt(6)\n"
    "v_mfma_f32_16x16x128_f8f6f4 v[200:203], v[112:119], a[144:151], v[200:203] cbsz:1 blgp:1\n"
    "ds_read_b128 v[144:147], v30 offset:26624\n"
    "ds_read_b128 v[148:151], v31 offset:26624\n"
    "global_store_dword v71, v199, s[22:23] sc0 sc1\n"
    "s_cmp_ge_u32 s62, s89\n"
    "s_cbranch_scc0 L_p6%=\n"
    "L_q6%=:\n"
    "s_add_u32 s22, s22, 0x40000\n"
    "s_addc_u32 s23, s23, 0\n"
    "v_mfma_f32_16x16x128_f8f6f4 v[204:207], v[112:119], a[152:159], v[204:207] cbsz:1 blgp:1\n"
    "buffer_load_dwordx4 v25, s[44:47], s82 offen offset:2048 sc0 sc1 lds\n"
    "s_waitcnt vmcnt(16)\n"
    "s_cmp_lt_u32 s64, s68\n"
    "s_cbranch_scc0 L_c7%=\n"
    "L_r7%=:\n"
    "s_waitcnt lgkmcnt(6)\n"
    "v_mfma_f32_16x16x128_f8f6f4 v[200:203], v[120:127], a[160:167], v[200:203] cbsz:1 blgp:1\n"
    "s_barrier\n"
    "ds_read_b128 v[96:99], v30 offset:32768\n"
    "ds_read_b128 v[100:103], v31 offset:32768\n"
    "s_add_u32 s64, s64, 1\n"
    "v_mfma_f32_16x16x128_f8f6f4 v[204:207], v[120:127], a[168:175], v[204:207] cbsz:1 blgp:1\n"
    "buffer_load_dwordx4 v[168:171], v40, s[48:51], s93 offen offset:0 sc0 sc1\n"
    "s_cmp_le_u32 s67, s64\n"
    "s_cbranch_scc1 L_u7_0%=\n"
    "L_v7_0%=:\n"
    "s_waitcnt lgkmcnt(6)\n"
    "v_mfma_f32_16x16x128_f8f6f4 v[200:203], v[128:135], a[176:183], v[200:203] cbsz:1 blgp:1\n"
    "ds_read_b128 v[104:107], v30 offset:34816\n"
    "ds_read_b128 v[108:111], v31 offset:34816\n"
    "buffer_load_dwordx4 v[172:175], v40, s[48:51], s93 offen offset:1024 sc0 sc1\n"
    "s_sub_u32 s89, s89, s94\n"
    "s_add_u32 s93, s93, 0x800\n"
    "v_mfma_f32_16x16x128_f8f6f4 v[204:207], v[128:135], a[184:191], v[204:207] cbsz:1 blgp:1\n"
    "s_add_u32 s74, s70, 10\n"
    "s_cmp_lt_u32 s96, s74\n"
    "s_cbranch_scc0 L_n7%=\n"
    "global_load_dword v42, v38, s[20:21] sc0 sc1\n"
    "L_n7%=:\n"
    "s_waitcnt lgkmcnt(6)\n"
    "v_mfma_f32_16x16x128_f8f6f4 v[200:203], v[136:143], a[192:199], v[200:203] cbsz:1 blgp:1\n"
    "ds_read_b128 v[112:115], v30 offset:36864\n"
    "ds_read_b128 v[116:119], v31 offset:36864\n"
    "v_mfma_f32_16x16x128_f8f6f4 v[204:207], v[136:143], a[200:207], v[204:207] cbsz:1 blgp:1\n"
    "s_cmp_eq_u32 s12, 0\n"
    "s_cbranch_scc0 L_s7%=\n"
    "s_load_dword s62, s[20:21], s71 glc\n"
    "L_s7%=:\n"
    "s_waitcnt lgkmcnt(6)\n"
    "v_mfma_f32_16x16x128_f8f6f4 v[200:203], v[144:151], a[208:215], v[200:203] cbsz:1 blgp:1\n"
    "ds_read_b128 v[120:123], v30 offset:38912\n"
    "ds_read_b128 v[124:127], v31 offset:38912\n"
    "v_readlane_b32 s74, v42, 0\n"
    "v_readlane_b32 s75, v42, 16\n"
    "v_readlane_b32 s76, v42, 32\n"
    "v_mfma_f32_16x16x128_f8f6f4 v[204:207], v[144:151], a[216:223], v[204:207] cbsz:1 blgp:1\n"
    "s_min_u32 s74, s74, s75\n"
    "s_min_u32 s96, s76, s74\n"
    "s_min_u32 s79, s96, s69\n"
    "s_add_u32 s74, s64, 1\n"
    "s_min_u32 s74, s74, s79\n"
    "s_waitcnt lgkmcnt(6)\n"
    "v_mfma_f32_16x16x128_f8f6f4 v[208:211], v[96:103], a[0:7], v[208:211] cbsz:1 blgp:1\n"
    "ds_read_b128 v[128:131], v30 offset:40960\n"
    "ds_read_b128 v[132:135], v31 offset:40960\n"
    "s_cmp_lt_u32 s67, s74\n"
    "s_cbranch_scc1 L_e7%=\n"
    "L_f7%=:\n"
    "s_mov_b32 s68, s67\n"
    "v_mfma_f32_16x16x128_f8f6f4 v[212:215], v[96:103], a[8:15], v[212:215] cbsz:1 blgp:1\n"
    "s_cmp_lt_u32 s67, s79\n"
    "s_cselect_b32 s46, s97, 0\n"
    "s_cselect_b32 s87, s72, 155648\n"
    "s_cselect_b32 s73, 0x8000, 0\n"
    "s_addc_u32 s67, s67, 0\n"
    "s_waitcnt lgkmcnt(6)\n"
    "v_mfma_f32_16x16x128_f8f6f4 v[208:211], v[104:111], a[16:23], v[208:211] cbsz:1 blgp:1\n"
    "ds_read_b128 v[136:139], v30 offset:43008\n"
    "ds_read_b128 v[140:143], v31 offset:43008\n"
    "v_fma_f32 v200, v200, v43, v160\n"
    "v_fma_f32 v201, v201, v43, v161\n"
    "v_fma_f32 v202, v202, v43, v162\n"
    "v_fma_f32 v203, v203, v43, v163\n"
    "v_mfma_f32_16x16x128_f8f6f4 v[212:215], v[104:111], a[24:31], v[212:215] cbsz:1 blgp:1\n"
    "v_fma_f32 v204, v204, v43, v164\n"
    "v_fma_f32 v205, v205, v43, v165\n"
    "v_fma_f32 v206, v206, v43, v166\n"
    "v_fma_f32 v207, v207, v43, v167\n"
    "s_waitcnt lgkmcnt(6)\n"
    "v_mfma_f32_16x16x128_f8f6f4 v[208:211], v[112:119], a[32:39], v[208:211] cbsz:1 blgp:1\n"
    "ds_read_b128 v[144:147], v30 offset:45056\n"
    "ds_read_b128 v[148:151], v31 offset:45056\n"
    "v_permlane16_swap_b32 v200, v204\n"
    "v_permlane16_swap_b32 v201, v205\n"
    "v_permlane16_swap_b32 v202, v206\n"
    "v_permlane16_swap_b32 v203, v207\n"
    "v_mfma_f32_16x16x128_f8f6f4 v[212:215], v[112:119], a[40:47], v[212:215] cbsz:1 blgp:1\n"
    "global_store_dword v64, v200, s[22:23] sc0 sc1\n"
    "s_waitcnt lgkmcnt(6)\n"
    "v_mfma_f32_16x16x128_f8f6f4 v[208:211], v[120:127], a[48:55], v[208:211] cbsz:1 blgp:1\n"
    "ds_read_b128 v[96:99], v30 offset:47104\n"
    "ds_read_b128 v[100:103], v31 offset:47104\n"
    "global_store_dword v68, v204, s[22:23] sc0 sc1\n"
    "s_mov_b32 m0, s87\n"
    "s_add_u32 s82, s95, 0x1000\n"
    "v_mfma_f32_16x16x128_f8f6f4 v[212:215], v[120:127], a[56:63], v[212:215] cbsz:1 blgp:1\n"
    "buffer_load_dwordx4 v25, s[44:47], s95 offen offset:0 sc0 sc1 lds\n"
    "s_add_u32 s72, s72, s73\n"
    "s_and_b32 s72, s72, 0x1ffff\n"
    "s_waitcnt lgkmcnt(6)\n"
    "v_mfma_f32_16x16x128_f8f6f4 v[208:211], v[128:135], a[64:71], v[208:211] cbsz:1 blgp:1\n"
    "ds_read_b128 v[104:107], v30 offset:49152\n"
    "ds_read_b128 v[108:111], v31 offset:49152\n"
    "global_store_dword v65, v201, s[22:23] sc0 sc1\n"
    "v_mfma_f32_16x16x128_f8f6f4 v[212:215], v[128:135], a[72:79], v[212:215] cbsz:1 blgp:1\n"
    "buffer_load_dwordx4 v25, s[44:47], s95 offen offset:1024 sc0 sc1 lds\n"
    "s_waitcnt lgkmcnt(6)\n"
    "v_mfma_f32_16x16x128_f8f6f4 v[208:211], v[136:143], a[80:87], v[208:211] cbsz:1 blgp:1\n"
    "ds_read_b128 v[112:115], v30 offset:51200\n"
    "ds_read_b128 v[116:119], v31 offset:51200\n"
    "global_store_dword v69, v205, s[22:23] sc0 sc1\n"
    "v_mfma_f32_16x16x128_f8f6f4 v[212:215], v[136:143], a[88:95], v[212:215] cbsz:1 blgp:1\n"
    "buffer_load_dwordx4 v25, s[44:47], s95 offen offset:2048 sc0 sc1 lds\n"
    "s_waitcnt lgkmcnt(6)\n"
    "v_mfma_f32_16x16x128_f8f6f4 v[208:211], v[144:151], a[96:103], v[208:211] cbsz:1 blgp:1\n"
    "ds_read_b128 v[120:123], v30 offset:53248\n"
    "ds_read_b128 v[124:127], v31 offset:53248\n"
    "global_store_dword v66, v202, s[22:23] sc0 sc1\n"
    "v_mfma_f32_16x16x128_f8f6f4 v[212:215], v[144:151], a[104:111], v[212:215] cbsz:1 blgp:1\n"
    "buffer_load_dwordx4 v25, s[44:47], s95 offen offset:3072 sc0 sc1 lds\n"
    "s_lshl1_add_u32 s95, s73, s95\n"
    "s_add_u32 m0, s87, 0x1000\n"
    "s_waitcnt lgkmcnt(6)\n"
    "v_mfma_f32_16x16x128_f8f6f4 v[208:211], v[96:103], a[112:119], v[208:211] cbsz:1 blgp:1\n"
    "ds_read_b128 v[128:131], v30 offset:55296\n"
    "ds_read_b128 v[132:135], v31 offset:55296\n"
    "global_store_dword v70, v206, s[22:23] sc0 sc1\n"
    "v_mfma_f32_16x16x128_f8f6f4 v[212:215], v[96:103], a[120:127], v[212:215] cbsz:1 blgp:1\n"
    "buffer_load_dwordx4 v25, s[44:47], s82 offen offset:0 sc0 sc1 lds\n"
    "s_waitcnt lgkmcnt(6)\n"
    "v_mfma_f32_16x16x128_f8f6f4 v[208:211], v[104:111], a[128:135], v[208:211] cbsz:1 blgp:1\n"
    "ds_read_b128 v[136:139], v30 offset:57344\n"
    "ds_read_b128 v[140:143], v31 offset:57344\n"
    "global_store_dword v67, v203, s[22:23] sc0 sc1\n"
    "v_mfma_f32_16x16x128_f8f6f4 v[212:215], v[104:111], a[136:143], v[212:215] cbsz:1 blgp:1\n"
    "buffer_load_dwordx4 v25, s[44:47], s82 offen offset:1024 sc0 sc1 lds\n"
    "s_waitcnt lgkmcnt(6)\n"
    "v_mfma_f32_16x16x128_f8f6f4 v[208:211], v[112:119], a[144:151], v[208:211] cbsz:1 blgp:1\n"
    "ds_read_b128 v[144:147], v30 offset:59392\n"
    "ds_read_b128 v[148:151], v31 offset:59392\n"
    "global_store_dword v71, v207, s[22:23] sc0 sc1\n"
    "s_cmp_ge_u32 s62, s89\n"
    "s_cbranch_scc0 L_p7%=\n"
    "L_q7%=:\n"
    "s_add_u32 s22, s22, 0x40000\n"
    "s_addc_u32 s23, s23, 0\n"
    "v_mfma_f32_16x16x128_f8f6f4 v[212:215], v[112:119], a[152:159], v[212:215] cbsz:1 blgp:1\n"
    "buffer_load_dwordx4 v25, s[44:47], s82 offen offset:2048 sc0 sc1 lds\n"
    "s_waitcnt vmcnt(15)\n"
    "s_cmp_lt_u32 s64, s68\n"
    "s_cbranch_scc0 L_c8%=\n"
    "L_r8%=:\n"
    "s_waitcnt lgkmcnt(6)\n"
    "v_mfma_f32_16x16x128_f8f6f4 v[208:211], v[120:127], a[160:167], v[208:211] cbsz:1 blgp:1\n"
    "s_barrier\n"
    "ds_read_b128 v[96:99], v26 offset:0\n"
    "ds_read_b128 v[100:103], v27 offset:0\n"
    "s_add_u32 s64, s64, 1\n"
    "v_mfma_f32_16x16x128_f8f6f4 v[212:215], v[120:127], a[168:175], v[212:215] cbsz:1 blgp:1\n"
    "buffer_load_dwordx4 v[176:179], v40, s[48:51], s93 offen offset:0 sc0 sc1\n"
    "s_cmp_le_u32 s67, s64\n"
    "s_cbranch_scc1 L_u8_0%=\n"
    "L_v8_0%=:\n"
    "s_waitcnt lgkmcnt(6)\n"
    "v_mfma_f32_16x16x128_f8f6f4 v[208:211], v[128:135], a[176:183], v[208:211] cbsz:1 blgp:1\n"
    "ds_read_b128 v[104:107], v26 offset:2048\n"
    "ds_read_b128 v[108:111], v27 offset:2048\n"
    "buffer_load_dwordx4 v[180:183], v40, s[48:51], s93 offen offset:1024 sc0 sc1\n"
    "s_sub_u32 s89, s89, s94\n"
    "s_add_u32 s93, s93, 0x800\n"
    "v_mfma_f32_16x16x128_f8f6f4 v[212:215], v[128:135], a[184:191], v[212:215] cbsz:1 blgp:1\n"
    "s_add_u32 s74, s70, 10\n"
    "s_cmp_lt_u32 s96, s74\n"
    "s_cbranch_scc0 L_n8%=\n"
    "global_load_dword v42, v38, s[20:21] sc0 sc1\n"
    "L_n8%=:\n"
    "s_waitcnt lgkmcnt(6)\n"
    "v_mfma_f32_16x16x128_f8f6f4 v[208:211], v[136:143], a[192:199], v[208:211] cbsz:1 blgp:1\n"
    "ds_read_b128 v[112:115], v26 offset:4096\n"
    "ds_read_b128 v[116:119], v27 offset:4096\n"
    "v_mfma_f32_16x16x128_f8f6f4 v[212:215], v[136:143], a[200:207], v[212:215] cbsz:1 blgp:1\n"
    "s_cmp_eq_u32 s12, 0\n"
    "s_cbranch_scc0 L_s8%=\n"
    "s_load_dword s62, s[20:21], s71 glc\n"
    "L_s8%=:\n"
    "s_waitcnt lgkmcnt(6)\n"
    "v_mfma_f32_16x16x128_f8f6f4 v[208:211], v[144:151], a[208:215], v[208:211] cbsz:1 blgp:1\n"
    "ds_read_b128 v[120:123], v26 offset:6144\n"
    "ds_read_b128 v[124:127], v27 offset:6144\n"
    "v_readlane_b32 s74, v42, 0\n"
    "v_readlane_b32 s75, v42, 16\n"
    "v_readlane_b32 s76, v42, 32\n"
    "v_mfma_f32_16x16x128_f8f6f4 v[212:215], v[144:151], a[216:223], v[212:215] cbsz:1 blgp:1\n"
    "s_min_u32 s74, s74, s75\n"
    "s_min_u32 s96, s76, s74\n"
    "s_min_u32 s79, s96, s69\n"
    "s_add_u32 s74, s64, 1\n"
    "s_min_u32 s74, s74, s79\n"
    "s_waitcnt lgkmcnt(6)\n"
    "v_mfma_f32_16x16x128_f8f6f4 v[216:219], v[96:103], a[0:7], v[216:219] cbsz:1 blgp:1\n"
    "ds_read_b128 v[128:131], v26 offset:8192\n"
    "ds_read_b128 v[132:135], v27 offset:8192\n"
    "s_cmp_lt_u32 s67, s74\n"
    "s_cbranch_scc1 L_e8%=\n"
    "L_f8%=:\n"
    "s_mov_b32 s68, s67\n"
    "v_mfma_f32_16x16x128_f8f6f4 v[220:223], v[96:103], a[8:15], v[220:223] cbsz:1 blgp:1\n"
    "s_waitcnt lgkmcnt(6)\n"
    "v_mfma_f32_16x16x128_f8f6f4 v[216:219], v[104:111], a[16:23], v[216:219] cbsz:1 blgp:1\n"
    "ds_read_b128 v[136:139], v26 offset:10240\n"
    "ds_read_b128 v[140:143], v27 offset:10240\n"
    "v_fma_f32 v208, v208, v43, v168\n"
    "v_fma_f32 v209, v209, v43, v169\n"
    "v_fma_f32 v210, v210, v43, v170\n"
    "v_fma_f32 v211, v211, v43, v171\n"
    "v_mfma_f32_16x16x128_f8f6f4 v[220:223], v[104:111], a[24:31], v[220:223] cbsz:1 blgp:1\n"
    "v_fma_f32 v212, v212, v43, v172\n"
    "v_fma_f32 v213, v213, v43, v173\n"
    "v_fma_f32 v214, v214, v43, v174\n"
    "v_fma_f32 v215, v215, v43, v175\n"
    "s_waitcnt lgkmcnt(6)\n"
    "v_mfma_f32_16x16x128_f8f6f4 v[216:219], v[112:119], a[32:39], v[216:219] cbsz:1 blgp:1\n"
    "ds_read_b128 v[144:147], v26 offset:12288\n"
    "ds_read_b128 v[148:151], v27 offset:12288\n"
    "v_permlane16_swap_b32 v208, v212\n"
    "v_permlane16_swap_b32 v209, v213\n"
    "v_permlane16_swap_b32 v210, v214\n"
    "v_permlane16_swap_b32 v211, v215\n"
    "v_mfma_f32_16x16x128_f8f6f4 v[220:223], v[112:119], a[40:47], v[220:223] cbsz:1 blgp:1\n"
    "global_store_dword v64, v208, s[22:23] sc0 sc1\n"
    "s_waitcnt lgkmcnt(6)\n"
    "v_mfma_f32_16x16x128_f8f6f4 v[216:219], v[120:127], a[48:55], v[216:219] cbsz:1 blgp:1\n"
    "ds_read_b128 v[96:99], v26 offset:14336\n"
    "ds_read_b128 v[100:103], v27 offset:14336\n"
    "global_store_dword v68, v212, s[22:23] sc0 sc1\n"
    "v_mfma_f32_16x16x128_f8f6f4 v[220:223], v[120:127], a[56:63], v[220:223] cbsz:1 blgp:1\n"
    "global_store_dword v65, v209, s[22:23] sc0 sc1\n"
    "s_waitcnt lgkmcnt(6)\n"
    "v_mfma_f32_16x16x128_f8f6f4 v[216:219], v[128:135], a[64:71], v[216:219] cbsz:1 blgp:1\n"
    "ds_read_b128 v[104:107], v26 offset:16384\n"
    "ds_read_b128 v[108:111], v27 offset:16384\n"
    "global_store_dword v69, v213, s[22:23] sc0 sc1\n"
    "v_mfma_f32_16x16x128_f8f6f4 v[220:223], v[128:135], a[72:79], v[220:223] cbsz:1 blgp:1\n"
    "global_store_dword v66, v210, s[22:23] sc0 sc1\n"
    "s_waitcnt lgkmcnt(6)\n"
    "v_mfma_f32_16x16x128_f8f6f4 v[216:219], v[136:143], a[80:87], v[216:219] cbsz:1 blgp:1\n"
    "ds_read_b128 v[112:115], v26 offset:18432\n"
    "ds_read_b128 v[116:119], v27 offset:18432\n"
    "global_store_dword v70, v214, s[22:23] sc0 sc1\n"
    "v_mfma_f32_16x16x128_f8f6f4 v[220:223], v[136:143], a[88:95], v[220:223] cbsz:1 blgp:1\n"
    "global_store_dword v67, v211, s[22:23] sc0 sc1\n"
    "s_waitcnt lgkmcnt(6)\n"
    "v_mfma_f32_16x16x128_f8f6f4 v[216:219], v[144:151], a[96:103], v[216:219] cbsz:1 blgp:1\n"
    "ds_read_b128 v[120:123], v26 offset:20480\n"
    "ds_read_b128 v[124:127], v27 offset:20480\n"
    "global_store_dword v71, v215, s[22:23] sc0 sc1\n"
    "v_mfma_f32_16x16x128_f8f6f4 v[220:223], v[144:151], a[104:111], v[220:223] cbsz:1 blgp:1\n"
    "s_waitcnt lgkmcnt(6)\n"
    "v_mfma_f32_16x16x128_f8f6f4 v[216:219], v[96:103], a[112:119], v[216:219] cbsz:1 blgp:1\n"
    "ds_read_b128 v[128:131], v26 offset:22528\n"
    "ds_read_b128 v[132:135], v27 offset:22528\n"
    "v_mfma_f32_16x16x128_f8f6f4 v[220:223], v[96:103], a[120:127], v[220:223] cbsz:1 blgp:1\n"
    "s_waitcnt lgkmcnt(6)\n"
    "v_mfma_f32_16x16x128_f8f6f4 v[216:219], v[104:111], a[128:135], v[216:219] cbsz:1 blgp:1\n"
    "ds_read_b128 v[136:139], v26 offset:24576\n"
    "ds_read_b128 v[140:143], v27 offset:24576\n"
    "v_mfma_f32_16x16x128_f8f6f4 v[220:223], v[104:111], a[136:143], v[220:223] cbsz:1 blgp:1\n"
    "s_waitcnt lgkmcnt(6)\n"
    "v_mfma_f32_16x16x128_f8f6f4 v[216:219], v[112:119], a[144:151], v[216:219] cbsz:1 blgp:1\n"
    "ds_read_b128 v[144:147], v26 offset:26624\n"
    "ds_read_b128 v[148:151], v27 offset:26624\n"
    "s_cmp_ge_u32 s62, s89\n"
    "s_cbranch_scc0 L_p8%=\n"
    "L_q8%=:\n"
    "s_add_u32 s22, s22, 0x40000\n"
    "s_addc_u32 s23, s23, 0\n"
    "v_mfma_f32_16x16x128_f8f6f4 v[220:223], v[112:119], a[152:159], v[220:223] cbsz:1 blgp:1\n"
    "s_waitcnt vmcnt(8)\n"
    "s_cmp_lt_u32 s64, s68\n"
    "s_cbranch_scc0 L_c9%=\n"
    "L_r9%=:\n"
    "s_waitcnt lgkmcnt(6)\n"
    "v_mfma_f32_16x16x128_f8f6f4 v[216:219], v[120:127], a[160:167], v[216:219] cbsz:1 blgp:1\n"
    "s_barrier\n"
    "ds_read_b128 v[96:99], v26 offset:32768\n"
    "ds_read_b128 v[100:103], v27 offset:32768\n"
    "v_mfma_f32_16x16x128_f8f6f4 v[220:223], v[120:127], a[168:175], v[220:223] cbsz:1 blgp:1\n"
    "buffer_load_dwordx4 v[184:187], v40, s[48:51], s93 offen offset:0 sc0 sc1\n"
    "s_waitcnt lgkmcnt(6)\n"
    "v_mfma_f32_16x16x128_f8f6f4 v[216:219], v[128:135], a[176:183], v[216:219] cbsz:1 blgp:1\n"
    "ds_read_b128 v[104:107], v26 offset:34816\n"
    "ds_read_b128 v[108:111], v27 offset:34816\n"
    "buffer_load_dwordx4 v[188:191], v40, s[48:51], s93 offen offset:1024 sc0 sc1\n"
    "s_sub_u32 s89, s89, s94\n"
    "s_add_u32 s93, s93, 0x800\n"
    "v_mfma_f32_16x16x128_f8f6f4 v[220:223], v[128:135], a[184:191], v[220:223] cbsz:1 blgp:1\n"
    "s_waitcnt lgkmcnt(6)\n"
    "v_mfma_f32_16x16x128_f8f6f4 v[216:219], v[136:143], a[192:199], v[216:219] cbsz:1 blgp:1\n"
    "ds_read_b128 v[112:115], v26 offset:36864\n"
    "ds_read_b128 v[116:119], v27 offset:36864\n"
    "v_mfma_f32_16x16x128_f8f6f4 v[220:223], v[136:143], a[200:207], v[220:223] cbsz:1 blgp:1\n"
    "s_waitcnt lgkmcnt(6)\n"
    "v_mfma_f32_16x16x128_f8f6f4 v[216:219], v[144:151], a[208:215], v[216:219] cbsz:1 blgp:1\n"
    "ds_read_b128 v[120:123], v26 offset:38912\n"
    "ds_read_b128 v[124:127], v27 offset:38912\n"
    "v_mfma_f32_16x16x128_f8f6f4 v[220:223], v[144:151], a[216:223], v[220:223] cbsz:1 blgp:1\n"
    "s_waitcnt lgkmcnt(6)\n"
    "v_mfma_f32_16x16x128_f8f6f4 v[224:227], v[96:103], a[0:7], v[224:227] cbsz:1 blgp:1\n"
    "ds_read_b128 v[128:131], v26 offset:40960\n"
    "ds_read_b128 v[132:135], v27 offset:40960\n"
    "v_mfma_f32_16x16x128_f8f6f4 v[228:231], v[96:103], a[8:15], v[228:231] cbsz:1 blgp:1\n"
    "s_waitcnt lgkmcnt(6)\n"
    "v_mfma_f32_16x16x128_f8f6f4 v[224:227], v[104:111], a[16:23], v[224:227] cbsz:1 blgp:1\n"
    "ds_read_b128 v[136:139], v26 offset:43008\n"
    "ds_read_b128 v[140:143], v27 offset:43008\n"
    "v_fma_f32 v216, v216, v43, v176\n"
    "v_fma_f32 v217, v217, v43, v177\n"
    "v_fma_f32 v218, v218, v43, v178\n"
    "v_fma_f32 v219, v219, v43, v179\n"
    "v_mfma_f32_16x16x128_f8f6f4 v[228:231], v[104:111], a[24:31], v[228:231] cbsz:1 blgp:1\n"
    "v_fma_f32 v220, v220, v43, v180\n"
    "v_fma_f32 v221, v221, v43, v181\n"
    "v_fma_f32 v222, v222, v43, v182\n"
    "v_fma_f32 v223, v223, v43, v183\n"
    "s_waitcnt lgkmcnt(6)\n"
    "v_mfma_f32_16x16x128_f8f6f4 v[224:227], v[112:119], a[32:39], v[224:227] cbsz:1 blgp:1\n"
    "ds_read_b128 v[144:147], v26 offset:45056\n"
    "ds_read_b128 v[148:151], v27 offset:45056\n"
    "v_permlane16_swap_b32 v216, v220\n"
    "v_permlane16_swap_b32 v217, v221\n"
    "v_permlane16_swap_b32 v218, v222\n"
    "v_permlane16_swap_b32 v219, v223\n"
    "v_mfma_f32_16x16x128_f8f6f4 v[228:231], v[112:119], a[40:47], v[228:231] cbsz:1 blgp:1\n"
    "global_store_dword v64, v216, s[22:23] sc0 sc1\n"
    "s_waitcnt lgkmcnt(6)\n"
    "v_mfma_f32_16x16x128_f8f6f4 v[224:227], v[120:127], a[48:55], v[224:227] cbsz:1 blgp:1\n"
    "ds_read_b128 v[96:99], v26 offset:47104\n"
    "ds_read_b128 v[100:103], v27 offset:47104\n"
    "global_store_dword v68, v220, s[22:23] sc0 sc1\n"
    "v_mfma_f32_16x16x128_f8f6f4 v[228:231], v[120:127], a[56:63], v[228:231] cbsz:1 blgp:1\n"
    "global_store_dword v65, v217, s[22:23] sc0 sc1\n"
    "s_waitcnt lgkmcnt(6)\n"
    "v_mfma_f32_16x16x128_f8f6f4 v[224:227], v[128:135], a[64:71], v[224:227] cbsz:1 blgp:1\n"
    "ds_read_b128 v[104:107], v26 offset:49152\n"
    "ds_read_b128 v[108:111], v27 offset:49152\n"
    "global_store_dword v69, v221, s[22:23] sc0 sc1\n"
    "v_mfma_f32_16x16x128_f8f6f4 v[228:231], v[128:135], a[72:79], v[228:231] cbsz:1 blgp:1\n"
    "global_store_dword v66, v218, s[22:23] sc0 sc1\n"
    "s_waitcnt lgkmcnt(6)\n"
    "v_mfma_f32_16x16x128_f8f6f4 v[224:227], v[136:143], a[80:87], v[224:227] cbsz:1 blgp:1\n"
    "ds_read_b128 v[112:115], v26 offset:51200\n"
    "ds_read_b128 v[116:119], v27 offset:51200\n"
    "global_store_dword v70, v222, s[22:23] sc0 sc1\n"
    "v_mfma_f32_16x16x128_f8f6f4 v[228:231], v[136:143], a[88:95], v[228:231] cbsz:1 blgp:1\n"
    "global_store_dword v67, v219, s[22:23] sc0 sc1\n"
    "s_waitcnt lgkmcnt(6)\n"
    "v_mfma_f32_16x16x128_f8f6f4 v[224:227], v[144:151], a[96:103], v[224:227] cbsz:1 blgp:1\n"
    "ds_read_b128 v[120:123], v26 offset:53248\n"
    "ds_read_b128 v[124:127], v27 offset:53248\n"
    "global_store_dword v71, v223, s[22:23] sc0 sc1\n"
    "v_mfma_f32_16x16x128_f8f6f4 v[228:231], v[144:151], a[104:111], v[228:231] cbsz:1 blgp:1\n"
    "s_waitcnt lgkmcnt(6)\n"
    "v_mfma_f32_16x16x128_f8f6f4 v[224:227], v[96:103], a[112:119], v[224:227] cbsz:1 blgp:1\n"
    "ds_read_b128 v[128:131], v26 offset:55296\n"
    "ds_read_b128 v[132:135], v27 offset:55296\n"
    "v_mfma_f32_16x16x128_f8f6f4 v[228:231], v[96:103], a[120:127], v[228:231] cbsz:1 blgp:1\n"
    "s_waitcnt lgkmcnt(6)\n"
    "v_mfma_f32_16x16x128_f8f6f4 v[224:227], v[104:111], a[128:135], v[224:227] cbsz:1 blgp:1\n"
    "ds_read_b128 v[136:139], v26 offset:57344\n"
    "ds_read_b128 v[140:143], v27 offset:57344\n"
    "v_mfma_f32_16x16x128_f8f6f4 v[228:231], v[104:111], a[136:143], v[228:231] cbsz:1 blgp:1\n"
    "s_waitcnt lgkmcnt(6)\n"
    "v_mfma_f32_16x16x128_f8f6f4 v[224:227], v[112:119], a[144:151], v[224:227] cbsz:1 blgp:1\n"
    "ds_read_b128 v[144:147], v26 offset:59392\n"
    "ds_read_b128 v[148:151], v27 offset:59392\n"
    "s_add_u32 s22, s22, 0x40000\n"
    "s_addc_u32 s23, s23, 0\n"
    "v_mfma_f32_16x16x128_f8f6f4 v[228:231], v[112:119], a[152:159], v[228:231] cbsz:1 blgp:1\n"
    "s_waitcnt lgkmcnt(6)\n"
    "v_mfma_f32_16x16x128_f8f6f4 v[224:227], v[120:127], a[160:167], v[224:227] cbsz:1 blgp:1\n"
    "v_mfma_f32_16x16x128_f8f6f4 v[228:231], v[120:127], a[168:175], v[228:231] cbsz:1 blgp:1\n"
    "s_waitcnt lgkmcnt(4)\n"
    "v_mfma_f32_16x16x128_f8f6f4 v[224:227], v[128:135], a[176:183], v[224:227] cbsz:1 blgp:1\n"
    "v_mfma_f32_16x16x128_f8f6f4 v[228:231], v[128:135], a[184:191], v[228:231] cbsz:1 blgp:1\n"
    "s_waitcnt lgkmcnt(2)\n"
    "v_mfma_f32_16x16x128_f8f6f4 v[224:227], v[136:143], a[192:199], v[224:227] cbsz:1 blgp:1\n"
    "v_mfma_f32_16x16x128_f8f6f4 v[228:231], v[136:143], a[200:207], v[228:231] cbsz:1 blgp:1\n"
    "s_waitcnt lgkmcnt(0)\n"
    "v_mfma_f32_16x16x128_f8f6f4 v[224:227], v[144:151], a[208:215], v[224:227] cbsz:1 blgp:1\n"
    "v_mfma_f32_16x16x128_f8f6f4 v[228:231], v[144:151], a[216:223], v[228:231] cbsz:1 blgp:1\n"
    "s_waitcnt vmcnt(8)\n"
    "s_nop 10\n"
    "v_fma_f32 v224, v224, v43, v184\n"
    "v_fma_f32 v228, v228, v43, v188\n"
    "v_fma_f32 v225, v225, v43, v185\n"
    "v_fma_f32 v229, v229, v43, v189\n"
    "v_fma_f32 v226, v226, v43, v186\n"
    "v_fma_f32 v230, v230, v43, v190\n"
    "v_fma_f32 v227, v227, v43, v187\n"
    "v_fma_f32 v231, v231, v43, v191\n"
    "v_permlane16_swap_b32 v224, v228\n"
    "v_permlane16_swap_b32 v225, v229\n"
    "v_permlane16_swap_b32 v226, v230\n"
    "v_permlane16_swap_b32 v227, v231\n"
    "global_store_dword v64, v224, s[22:23] sc0 sc1\n"
    "global_store_dword v68, v228, s[22:23] sc0 sc1\n"
    "global_store_dword v65, v225, s[22:23] sc0 sc1\n"
    "global_store_dword v69, v229, s[22:23] sc0 sc1\n"
    "global_store_dword v66, v226, s[22:23] sc0 sc1\n"
    "global_store_dword v70, v230, s[22:23] sc0 sc1\n"
    "global_store_dword v67, v227, s[22:23] sc0 sc1\n"
    "global_store_dword v71, v231, s[22:23] sc0 sc1\n"
    "s_endpgm\n"
    "L_ext%=:\n"
    "v_readlane_b32 s98, v44, 15\n"
    "v_mov_b32 v28, v48\n"
    "v_mov_b32 v29, v49\n"
    "v_mov_b32 v30, v50\n"
    "v_mov_b32 v31, v51\n"
    "s_mov_b32 exec_lo, 0xff00ff00\n"
    "s_mov_b32 exec_hi, 0xff00\n"
    "v_add_u32 v41, 128, v41\n"
    "s_mov_b64 exec, -1\n"
    "v_mov_b32 v42, 0\n"
    "s_add_u32 m0, s85, 32768\n"
    "v_readlane_b32 s99, v44, 15\n"
    "buffer_load_dwordx4 v16, s[40:43], s98 offen offset:0 lds\n"
    "buffer_load_dwordx4 v17, s[40:43], s98 offen offset:1024 lds\n"
    "buffer_load_dwordx4 v18, s[40:43], s98 offen offset:2048 lds\n"
    "buffer_load_dwordx4 v19, s[40:43], s98 offen offset:3072 lds\n"
    "s_waitcnt lgkmcnt(0)\n"
    "v_mov_b32 v43, s83\n"
    "v_mul_f32 v43, s84, v43\n"
    "s_waitcnt vmcnt(9)\n"
    "s_barrier\n"
    "s_add_u32 m0, s85, 49152\n"
    "ds_read_b64_tr_b8 a[224:225], v20 offset:0\n"
    "v_readlane_b32 s98, v44, 0\n"
    "ds_read_b64_tr_b8 a[226:227], v20 offset:1024\n"
    "buffer_load_dwordx4 v28, s[40:43], s99 offen offset:0 lds\n"
    "ds_read_b64_tr_b8 a[228:229], v20 offset:8192\n"
    "buffer_load_dwordx4 v29, s[40:43], s99 offen offset:1024 lds\n"
    "ds_read_b64_tr_b8 a[230:231], v20 offset:9216\n"
    "ds_read_b64_tr_b8 a[232:233], v21 offset:0\n"
    "ds_read_b64_tr_b8 a[234:235], v21 offset:1024\n"
    "ds_read_b64_tr_b8 a[236:237], v21 offset:8192\n"
    "ds_read_b64_tr_b8 a[238:239], v21 offset:9216\n"
    "buffer_load_dwordx4 v30, s[40:43], s99 offen offset:2048 lds\n"
    "buffer_load_dwordx4 v31, s[40:43], s99 offen offset:3072 lds\n"
    "s_waitcnt vmcnt(9)\n"
    "s_barrier\n"
    "s_add_u32 m0, s85, 65536\n"
    "ds_read_b64_tr_b8 v[56:57], v20 offset:16384\n"
    "v_readlane_b32 s99, v44, 4\n"
    "ds_read_b64_tr_b8 v[58:59], v20 offset:17408\n"
    "buffer_load_dwordx4 v16, s[40:43], s98 offen offset:0 lds\n"
    "ds_read_b64_tr_b8 v[60:61], v20 offset:24576\n"
    "buffer_load_dwordx4 v17, s[40:43], s98 offen offset:1024 lds\n"
    "ds_read_b64_tr_b8 v[62:63], v20 offset:25600\n"
    "ds_read_b64_tr_b8 v[72:73], v21 offset:16384\n"
    "ds_read_b64_tr_b8 v[74:75], v21 offset:17408\n"
    "ds_read_b64_tr_b8 v[76:77], v21 offset:24576\n"
    "s_waitcnt lgkmcnt(11)\n"
    "ds_read_b64_tr_b8 v[78:79], v21 offset:25600\n"
    "buffer_load_dwordx4 v18, s[40:43], s98 offen offset:2048 lds\n"
    "buffer_load_dwordx4 v19, s[40:43], s98 offen offset:3072 lds\n"
    "s_waitcnt vmcnt(8) lgkmcnt(8)\n"
    "s_barrier\n"
    "s_add_u32 m0, s85, 0\n"
    "ds_read_b64_tr_b8 a[240:241], v20 offset:32768\n"
    "v_readlane_b32 s98, v44, 8\n"
    "ds_read_b64_tr_b8 a[242:243], v20 offset:33792\n"
    "buffer_load_dwordx4 v16, s[40:43], s99 offen offset:0 lds\n"
    "ds_read_b64_tr_b8 a[244:245], v20 offset:40960\n"
    "buffer_load_dwordx4 v17, s[40:43], s99 offen offset:1024 lds\n"
    "ds_read_b64_tr_b8 a[246:247], v20 offset:41984\n"
    "ds_read_b64_tr_b8 a[248:249], v21 offset:32768\n"
    "ds_read_b64_tr_b8 a[250:251], v21 offset:33792\n"
    "ds_read_b64_tr_b8 a[252:253], v21 offset:40960\n"
    "s_waitcnt lgkmcnt(11)\n"
    "ds_read_b64_tr_b8 a[254:255], v21 offset:41984\n"
    "buffer_load_dwordx4 v18, s[40:43], s99 offen offset:2048 lds\n"
    "buffer_load_dwordx4 v19, s[40:43], s99 offen offset:3072 lds\n"
    "s_waitcnt vmcnt(8) lgkmcnt(8)\n"
    "s_barrier\n"
    "s_add_u32 m0, s85, 16384\n"
    "ds_read_b64_tr_b8 v[80:81], v20 offset:49152\n"
    "v_readlane_b32 s99, v44, 1\n"
    "ds_read_b64_tr_b8 v[82:83], v20 offset:50176\n"
    "buffer_load_dwordx4 v16, s[40:43], s98 offen offset:0 lds\n"
    "ds_read_b64_tr_b8 v[84:85], v20 offset:57344\n"
    "buffer_load_dwordx4 v17, s[40:43], s98 offen offset:1024 lds\n"
    "ds_read_b64_tr_b8 v[86:87], v20 offset:58368\n"
    "ds_read_b64_tr_b8 v[88:89], v21 offset:49152\n"
    "ds_read_b64_tr_b8 v[90:91], v21 offset:50176\n"
    "ds_read_b64_tr_b8 v[92:93], v21 offset:57344\n"
    "s_waitcnt lgkmcnt(11)\n"
    "ds_read_b64_tr_b8 v[94:95], v21 offset:58368\n"
    "buffer_load_dwordx4 v18, s[40:43], s98 offen offset:2048 lds\n"
    "buffer_load_dwordx4 v19, s[40:43], s98 offen offset:3072 lds\n"
    "v_readfirstlane_b32 s74, v45\n"
    "s_add_u32 s74, s74, 1\n"
    "s_lshl_b32 s70, s74, 4\n"
    "s_and_b32 s74, s74, 1\n"
    "s_lshl_b32 s74, s74, 21\n"
    "s_lshl1_add_u32 s75, s86, s15\n"
    "s_lshl_b32 s75, s75, 15\n"
    "s_add_u32 s74, s74, s75\n"
    "s_add_u32 s44, s20, s74\n"
    "s_addc_u32 s45, s21, 0\n"
    "s_and_b32 s45, s45, 0xffff\n"
    "s_mov_b32 s46, 0\n"
    "s_mov_b32 s47, 0x20000\n"
    "s_mov_b32 s64, s70\n"
    "s_mov_b32 s67, s70\n"
    "s_add_u32 s69, s70, 1\n"
    "s_cmp_eq_u32 s12, 0\n"
    "s_cselect_b32 s94, -1, 0\n"
    "s_and_b32 s89, s69, s94\n"
    "v_mov_b32 v46, s70\n"
    "s_waitcnt vmcnt(8) lgkmcnt(8)\n"
    "s_barrier\n"
    "s_add_u32 m0, s85, 32768\n"
    "ds_read_b64_tr_b8 a[0:1], v22 offset:0\n"
    "v_readlane_b32 s98, v44, 5\n"
    "ds_read_b64_tr_b8 a[2:3], v22 offset:1024\n"
    "buffer_load_dwordx4 v16, s[40:43], s99 offen offset:0 lds\n"
    "ds_read_b64_tr_b8 a[4:5], v22 offset:8192\n"
    "buffer_load_dwordx4 v17, s[40:43], s99 offen offset:1024 lds\n"
    "ds_read_b64_tr_b8 a[6:7], v22 offset:9216\n"
    "ds_read_b64_tr_b8 a[8:9], v23 offset:0\n"
    "ds_read_b64_tr_b8 a[10:11], v23 offset:1024\n"
    "ds_read_b64_tr_b8 a[12:13], v23 offset:8192\n"
    "s_waitcnt lgkmcnt(11)\n"
    "ds_read_b64_tr_b8 a[14:15], v23 offset:9216\n"
    "buffer_load_dwordx4 v18, s[40:43], s99 offen offset:2048 lds\n"
    "buffer_load_dwordx4 v19, s[40:43], s99 offen offset:3072 lds\n"
    "s_waitcnt vmcnt(8) lgkmcnt(8)\n"
    "s_barrier\n"
    "s_add_u32 m0, s85, 49152\n"
    "ds_read_b64_tr_b8 a[64:65], v20 offset:0\n"
    "v_readlane_b32 s99, v44, 9\n"
    "ds_read_b64_tr_b8 a[66:67], v20 offset:1024\n"
    "buffer_load_dwordx4 v16, s[40:43], s98 offen offset:0 lds\n"
    "ds_read_b64_tr_b8 a[68:69], v20 offset:8192\n"
    "buffer_load_dwordx4 v17, s[40:43], s98 offen offset:1024 lds\n"
    "ds_read_b64_tr_b8 a[70:71], v20 offset:9216\n"
    "ds_read_b64_tr_b8 a[72:73], v21 offset:0\n"
    "ds_read_b64_tr_b8 a[74:75], v21 offset:1024\n"
    "ds_read_b64_tr_b8 a[76:77], v21 offset:8192\n"
    "s_waitcnt lgkmcnt(11)\n"
    "ds_read_b64_tr_b8 a[78:79], v21 offset:9216\n"
    "buffer_load_dwordx4 v18, s[40:43], s98 offen offset:2048 lds\n"
    "buffer_load_dwordx4 v19, s[40:43], s98 offen offset:3072 lds\n"
    "s_waitcnt vmcnt(8) lgkmcnt(8)\n"
    "s_barrier\n"
    "s_add_u32 m0, s85, 65536\n"
    "ds_read_b64_tr_b8 a[128:129], v20 offset:16384\n"
    "v_readlane_b32 s98, v44, 12\n"
    "ds_read_b64_tr_b8 a[130:131], v20 offset:17408\n"
    "buffer_load_dwordx4 v16, s[40:43], s99 offen offset:0 lds\n"
    "ds_read_b64_tr_b8 a[132:133], v20 offset:24576\n"
    "buffer_load_dwordx4 v17, s[40:43], s99 offen offset:1024 lds\n"
    "ds_read_b64_tr_b8 a[134:135], v20 offset:25600\n"
    "ds_read_b64_tr_b8 a[136:137], v21 offset:16384\n"
    "ds_read_b64_tr_b8 a[138:139], v21 offset:17408\n"
    "ds_read_b64_tr_b8 a[140:141], v21 offset:24576\n"
    "s_waitcnt lgkmcnt(11)\n"
    "ds_read_b64_tr_b8 a[142:143], v21 offset:25600\n"
    "buffer_load_dwordx4 v18, s[40:43], s99 offen offset:2048 lds\n"
    "buffer_load_dwordx4 v19, s[40:43], s99 offen offset:3072 lds\n"
    "s_waitcnt vmcnt(8) lgkmcnt(8)\n"
    "s_barrier\n"
    "s_add_u32 m0, s85, 0\n"
    "ds_read_b64_tr_b8 a[16:17], v20 offset:32768\n"
    "v_readlane_b32 s99, v44, 12\n"
    "ds_read_b64_tr_b8 a[18:19], v20 offset:33792\n"
    "buffer_load_dwordx4 v16, s[40:43], s98 offen offset:0 lds\n"
    "ds_read_b64_tr_b8 a[20:21], v20 offset:40960\n"
    "buffer_load_dwordx4 v17, s[40:43], s98 offen offset:1024 lds\n"
    "ds_read_b64_tr_b8 a[22:23], v20 offset:41984\n"
    "ds_read_b64_tr_b8 a[24:25], v21 offset:32768\n"
    "ds_read_b64_tr_b8 a[26:27], v21 offset:33792\n"
    "ds_read_b64_tr_b8 a[28:29], v21 offset:40960\n"
    "s_waitcnt lgkmcnt(11)\n"
    "ds_read_b64_tr_b8 a[30:31], v21 offset:41984\n"
    "buffer_load_dwordx4 v18, s[40:43], s98 offen offset:2048 lds\n"
    "buffer_load_dwordx4 v19, s[40:43], s98 offen offset:3072 lds\n"
    "s_waitcnt vmcnt(8) lgkmcnt(8)\n"
    "s_barrier\n"
    "s_add_u32 m0, s85, 16384\n"
    "ds_read_b64_tr_b8 a[80:81], v20 offset:49152\n"
    "v_readlane_b32 s98, v44, 13\n"
    "ds_read_b64_tr_b8 a[82:83], v20 offset:50176\n"
    "buffer_load_dwordx4 v28, s[40:43], s99 offen offset:0 lds\n"
    "ds_read_b64_tr_b8 a[84:85], v20 offset:57344\n"
    "buffer_load_dwordx4 v29, s[40:43], s99 offen offset:1024 lds\n"
    "ds_read_b64_tr_b8 a[86:87], v20 offset:58368\n"
    "ds_read_b64_tr_b8 a[88:89], v21 offset:49152\n"
    "ds_read_b64_tr_b8 a[90:91], v21 offset:50176\n"
    "ds_read_b64_tr_b8 a[92:93], v21 offset:57344\n"
    "s_waitcnt lgkmcnt(11)\n"
    "ds_read_b64_tr_b8 a[94:95], v21 offset:58368\n"
    "buffer_load_dwordx4 v30, s[40:43], s99 offen offset:2048 lds\n"
    "buffer_load_dwordx4 v31, s[40:43], s99 offen offset:3072 lds\n"
    "s_waitcnt vmcnt(8) lgkmcnt(8)\n"
    "s_barrier\n"
    "s_add_u32 m0, s85, 32768\n"
    "ds_read_b64_tr_b8 a[144:145], v22 offset:0\n"
    "v_readlane_b32 s99, v44, 13\n"
    "ds_read_b64_tr_b8 a[146:147], v22 offset:1024\n"
    "buffer_load_dwordx4 v16, s[40:43], s98 offen offset:0 lds\n"
    "ds_read_b64_tr_b8 a[148:149], v22 offset:8192\n"
    "buffer_load_dwordx4 v17, s[40:43], s98 offen offset:1024 lds\n"
    "ds_read_b64_tr_b8 a[150:151], v22 offset:9216\n"
    "ds_read_b64_tr_b8 a[152:153], v23 offset:0\n"
    "ds_read_b64_tr_b8 a[154:155], v23 offset:1024\n"
    "ds_read_b64_tr_b8 a[156:157], v23 offset:8192\n"
    "s_waitcnt lgkmcnt(11)\n"
    "ds_read_b64_tr_b8 a[158:159], v23 offset:9216\n"
    "buffer_load_dwordx4 v18, s[40:43], s98 offen offset:2048 lds\n"
    "buffer_load_dwordx4 v19, s[40:43], s98 offen offset:3072 lds\n"
    "s_waitcnt vmcnt(8) lgkmcnt(8)\n"
    "s_barrier\n"
    "s_add_u32 m0, s85, 49152\n"
    "ds_read_b64_tr_b8 a[192:193], v20 offset:0\n"
    "v_readlane_b32 s98, v44, 2\n"
    "ds_read_b64_tr_b8 a[194:195], v20 offset:1024\n"
    "buffer_load_dwordx4 v28, s[40:43], s99 offen offset:0 lds\n"
    "ds_read_b64_tr_b8 a[196:197], v20 offset:8192\n"
    "buffer_load_dwordx4 v29, s[40:43], s99 offen offset:1024 lds\n"
    "ds_read_b64_tr_b8 a[198:199], v20 offset:9216\n"
    "ds_read_b64_tr_b8 a[200:201], v21 offset:0\n"
    "ds_read_b64_tr_b8 a[202:203], v21 offset:1024\n"
    "ds_read_b64_tr_b8 a[204:205], v21 offset:8192\n"
    "s_waitcnt lgkmcnt(11)\n"
    "ds_read_b64_tr_b8 a[206:207], v21 offset:9216\n"
    "buffer_load_dwordx4 v30, s[40:43], s99 offen offset:2048 lds\n"
    "buffer_load_dwordx4 v31, s[40:43], s99 offen offset:3072 lds\n"
    "s_waitcnt vmcnt(8) lgkmcnt(8)\n"
    "s_barrier\n"
    "s_add_u32 m0, s85, 65536\n"
    "ds_read_b64_tr_b8 v[128:129], v20 offset:16384\n"
    "v_readlane_b32 s99, v44, 6\n"
    "ds_read_b64_tr_b8 v[130:131], v20 offset:17408\n"
    "buffer_load_dwordx4 v16, s[40:43], s98 offen offset:0 lds\n"
    "ds_read_b64_tr_b8 v[132:133], v20 offset:24576\n"
    "buffer_load_dwordx4 v17, s[40:43], s98 offen offset:1024 lds\n"
    "ds_read_b64_tr_b8 v[134:135], v20 offset:25600\n"
    "ds_read_b64_tr_b8 v[136:137], v21 offset:16384\n"
    "ds_read_b64_tr_b8 v[138:139], v21 offset:17408\n"
    "ds_read_b64_tr_b8 v[140:141], v21 offset:24576\n"
    "s_waitcnt lgkmcnt(11)\n"
    "ds_read_b64_tr_b8 v[142:143], v21 offset:25600\n"
    "buffer_load_dwordx4 v18, s[40:43], s98 offen offset:2048 lds\n"
    "buffer_load_dwordx4 v19, s[40:43], s98 offen offset:3072 lds\n"
    "s_waitcnt vmcnt(8) lgkmcnt(8)\n"
    "s_barrier\n"
    "s_add_u32 m0, s85, 0\n"
    "ds_read_b64_tr_b8 a[208:209], v20 offset:32768\n"
    "v_readlane_b32 s98, v44, 10\n"
    "ds_read_b64_tr_b8 a[210:211], v20 offset:33792\n"
    "buffer_load_dwordx4 v16, s[40:43], s99 offen offset:0 lds\n"
    "ds_read_b64_tr_b8 a[212:213], v20 offset:40960\n"
    "buffer_load_dwordx4 v17, s[40:43], s99 offen offset:1024 lds\n"
    "ds_read_b64_tr_b8 a[214:215], v20 offset:41984\n"
    "ds_read_b64_tr_b8 a[216:217], v21 offset:32768\n"
    "ds_read_b64_tr_b8 a[218:219], v21 offset:33792\n"
    "ds_read_b64_tr_b8 a[220:221], v21 offset:40960\n"
    "s_waitcnt lgkmcnt(11)\n"
    "ds_read_b64_tr_b8 a[222:223], v21 offset:41984\n"
    "buffer_load_dwordx4 v18, s[40:43], s99 offen offset:2048 lds\n"
    "buffer_load_dwordx4 v19, s[40:43], s99 offen offset:3072 lds\n"
    "s_lshl_b32 s74, s28, 5\n"
    "s_add_u32 s74, s74, 30\n"
    "s_lshl_b32 s74, s74, 13\n"
    "s_add_u32 s75, s74, 0x2000\n"
    "s_mov_b32 m0, s87\n"
    "s_nop 0\n"
    "buffer_load_dwordx4 v24, s[52:55], s74 offen lds\n"
    "s_add_u32 m0, s87, 0x8000\n"
    "s_nop 0\n"
    "buffer_load_dwordx4 v24, s[52:55], s75 offen lds\n"
    "s_waitcnt vmcnt(10) lgkmcnt(8)\n"
    "s_barrier\n"
    "s_add_u32 m0, s85, 16384\n"
    "ds_read_b64_tr_b8 v[144:145], v20 offset:49152\n"
    "v_readlane_b32 s99, v44, 3\n"
    "ds_read_b64_tr_b8 v[146:147], v20 offset:50176\n"
    "buffer_load_dwordx4 v16, s[40:43], s98 offen offset:0 lds\n"
    "ds_read_b64_tr_b8 v[148:149], v20 offset:57344\n"
    "buffer_load_dwordx4 v17, s[40:43], s98 offen offset:1024 lds\n"
    "ds_read_b64_tr_b8 v[150:151], v20 offset:58368\n"
    "ds_read_b64_tr_b8 v[168:169], v21 offset:49152\n"
    "ds_read_b64_tr_b8 v[170:171], v21 offset:50176\n"
    "ds_read_b64_tr_b8 v[172:173], v21 offset:57344\n"
    "s_waitcnt lgkmcnt(11)\n"
    "ds_read_b64_tr_b8 v[174:175], v21 offset:58368\n"
    "buffer_load_dwordx4 v18, s[40:43], s98 offen offset:2048 lds\n"
    "buffer_load_dwordx4 v19, s[40:43], s98 offen offset:3072 lds\n"
    "s_waitcnt vmcnt(10) lgkmcnt(8)\n"
    "s_barrier\n"
    "s_add_u32 m0, s85, 32768\n"
    "ds_read_b64_tr_b8 a[32:33], v22 offset:0\n"
    "v_readlane_b32 s98, v44, 7\n"
    "ds_read_b64_tr_b8 a[34:35], v22 offset:1024\n"
    "buffer_load_dwordx4 v16, s[40:43], s99 offen offset:0 lds\n"
    "ds_read_b64_tr_b8 a[36:37], v22 offset:8192\n"
    "buffer_load_dwordx4 v17, s[40:43], s99 offen offset:1024 lds\n"
    "ds_read_b64_tr_b8 a[38:39], v22 offset:9216\n"
    "ds_read_b64_tr_b8 a[40:41], v23 offset:0\n"
    "ds_read_b64_tr_b8 a[42:43], v23 offset:1024\n"
    "ds_read_b64_tr_b8 a[44:45], v23 offset:8192\n"
    "s_waitcnt lgkmcnt(11)\n"
    "ds_read_b64_tr_b8 a[46:47], v23 offset:9216\n"
    "buffer_load_dwordx4 v18, s[40:43], s99 offen offset:2048 lds\n"
    "buffer_load_dwordx4 v19, s[40:43], s99 offen offset:3072 lds\n"
    "s_waitcnt vmcnt(10) lgkmcnt(8)\n"
    "s_barrier\n"
    "s_add_u32 m0, s85, 49152\n"
    "ds_read_b64_tr_b8 a[96:97], v20 offset:0\n"
    "v_readlane_b32 s99, v44, 11\n"
    "ds_read_b64_tr_b8 a[98:99], v20 offset:1024\n"
    "buffer_load_dwordx4 v16, s[40:43], s98 offen offset:0 lds\n"
    "ds_read_b64_tr_b8 a[100:101], v20 offset:8192\n"
    "buffer_load_dwordx4 v17, s[40:43], s98 offen offset:1024 lds\n"
    "ds_read_b64_tr_b8 a[102:103], v20 offset:9216\n"
    "ds_read_b64_tr_b8 a[104:105], v21 offset:0\n"
    "ds_read_b64_tr_b8 a[106:107], v21 offset:1024\n"
    "ds_read_b64_tr_b8 a[108:109], v21 offset:8192\n"
    "s_waitcnt lgkmcnt(11)\n"
    "ds_read_b64_tr_b8 a[110:111], v21 offset:9216\n"
    "buffer_load_dwordx4 v18, s[40:43], s98 offen offset:2048 lds\n"
    "buffer_load_dwordx4 v19, s[40:43], s98 offen offset:3072 lds\n"
    "s_waitcnt vmcnt(8) lgkmcnt(8)\n"
    "s_barrier\n"
    "global_load_dword v42, v41, s[20:21] sc0 sc1\n"
    "ds_read_b64_tr_b8 a[160:161], v20 offset:16384\n"
    "s_add_u32 m0, s85, 65536\n"
    "ds_read_b64_tr_b8 a[162:163], v20 offset:17408\n"
    "v_readlane_b32 s98, v44, 0\n"
    "ds_read_b64_tr_b8 a[164:165], v20 offset:24576\n"
    "buffer_load_dwordx4 v16, s[40:43], s99 offen offset:0 lds\n"
    "ds_read_b64_tr_b8 a[166:167], v20 offset:25600\n"
    "ds_read_b64_tr_b8 a[168:169], v21 offset:16384\n"
    "ds_read_b64_tr_b8 a[170:171], v21 offset:17408\n"
    "ds_read_b64_tr_b8 a[172:173], v21 offset:24576\n"
    "s_waitcnt lgkmcnt(11)\n"
    "ds_read_b64_tr_b8 a[174:175], v21 offset:25600\n"
    "buffer_load_dwordx4 v17, s[40:43], s99 offen offset:1024 lds\n"
    "buffer_load_dwordx4 v18, s[40:43], s99 offen offset:2048 lds\n"
    "buffer_load_dwordx4 v19, s[40:43], s99 offen offset:3072 lds\n"
    "s_waitcnt vmcnt(9) lgkmcnt(8)\n"
    "s_barrier\n"
    "ds_read_b128 v[96:99], v26 offset:28672\n"
    "ds_read_b128 v[100:103], v27 offset:28672\n"
    "ds_read_b128 v[104:107], v26 offset:61440\n"
    "ds_read_b128 v[108:111], v27 offset:61440\n"
    "ds_read_b128 v[112:115], v26 offset:30720\n"
    "ds_read_b128 v[116:119], v27 offset:30720\n"
    "s_add_u32 m0, s85, 0\n"
    "ds_read_b64_tr_b8 a[48:49], v20 offset:32768\n"
    "v_readlane_b32 s99, v44, 4\n"
    "s_waitcnt lgkmcnt(11)\n"
    "ds_read_b64_tr_b8 a[50:51], v20 offset:33792\n"
    "buffer_load_dwordx4 v28, s[40:43], s98 offen offset:0 lds\n"
    "ds_read_b64_tr_b8 a[52:53], v20 offset:40960\n"
    "buffer_load_dwordx4 v29, s[40:43], s98 offen offset:1024 lds\n"
    "ds_read_b64_tr_b8 a[54:55], v20 offset:41984\n"
    "ds_read_b64_tr_b8 a[56:57], v21 offset:32768\n"
    "s_waitcnt lgkmcnt(11)\n"
    "ds_read_b64_tr_b8 a[58:59], v21 offset:33792\n"
    "ds_read_b64_tr_b8 a[60:61], v21 offset:40960\n"
    "ds_read_b64_tr_b8 a[62:63], v21 offset:41984\n"
    "buffer_load_dwordx4 v30, s[40:43], s98 offen offset:2048 lds\n"
    "buffer_load_dwordx4 v31, s[40:43], s98 offen offset:3072 lds\n"
    "s_waitcnt vmcnt(9)\n"
    "s_barrier\n"
    "s_waitcnt lgkmcnt(12)\n"
    "v_mfma_f32_16x16x128_f8f6f4 v[152:155], v[96:103], a[224:231], 0 cbsz:1 blgp:1\n"
    "ds_read_b128 v[120:123], v26 offset:63488\n"
    "ds_read_b128 v[124:127], v27 offset:63488\n"
    "s_add_u32 m0, s85, 16384\n"
    "ds_read_b64_tr_b8 a[112:113], v20 offset:49152\n"
    "v_mfma_f32_16x16x128_f8f6f4 v[156:159], v[96:103], a[232:239], 0 cbsz:1 blgp:1\n"
    "v_readlane_b32 s98, v44, 8\n"
    "s_waitcnt lgkmcnt(11)\n"
    "ds_read_b64_tr_b8 a[114:115], v20 offset:50176\n"
    "buffer_load_dwordx4 v28, s[40:43], s99 offen offset:0 lds\n"
    "ds_read_b64_tr_b8 a[116:117], v20 offset:57344\n"
    "v_mfma_f32_16x16x128_f8f6f4 v[160:163], v[104:111], a[224:231], 0 cbsz:1 blgp:1\n"
    "ds_read_b128 v[96:99], v26 offset:28672\n"
    "ds_read_b128 v[100:103], v27 offset:28672\n"
    "buffer_load_dwordx4 v29, s[40:43], s99 offen offset:1024 lds\n"
    "s_waitcnt lgkmcnt(11)\n"
    "ds_read_b64_tr_b8 a[118:119], v20 offset:58368\n"
    "v_mfma_f32_16x16x128_f8f6f4 v[164:167], v[104:111], a[232:239], 0 cbsz:1 blgp:1\n"
    "buffer_load_dwordx4 v30, s[40:43], s99 offen offset:2048 lds\n"
    "ds_read_b64_tr_b8 a[120:121], v21 offset:49152\n"
    "ds_read_b64_tr_b8 a[122:123], v21 offset:50176\n"
    "ds_read_b64_tr_b8 a[124:125], v21 offset:57344\n"
    "buffer_load_dwordx4 v31, s[40:43], s99 offen offset:3072 lds\n"
    "s_waitcnt lgkmcnt(11)\n"
    "ds_read_b64_tr_b8 a[126:127], v21 offset:58368\n"
    "s_waitcnt vmcnt(8)\n"
    "s_barrier\n"
    "v_mfma_f32_16x16x128_f8f6f4 v[152:155], v[112:119], a[240:247], v[152:155] cbsz:1 blgp:1\n"
    "ds_read_b128 v[104:107], v26 offset:61440\n"
    "ds_read_b128 v[108:111], v27 offset:61440\n"
    "global_load_dword v42, v41, s[20:21] sc0 sc1\n"
    "ds_read_b64_tr_b8 a[176:177], v22 offset:0\n"
    "v_mfma_f32_16x16x128_f8f6f4 v[156:159], v[112:119], a[248:255], v[156:159] cbsz:1 blgp:1\n"
    "s_add_u32 m0, s85, 32768\n"
    "s_waitcnt lgkmcnt(11)\n"
    "ds_read_b64_tr_b8 a[178:179], v22 offset:1024\n"
    "v_readlane_b32 s99, v44, 1\n"
    "ds_read_b64_tr_b8 a[180:181], v22 offset:8192\n"
    "v_mfma_f32_16x16x128_f8f6f4 v[160:163], v[120:127], a[240:247], v[160:163] cbsz:1 blgp:1\n"
    "ds_read_b128 v[112:115], v26 offset:30720\n"
    "ds_read_b128 v[116:119], v27 offset:30720\n"
    "buffer_load_dwordx4 v28, s[40:43], s98 offen offset:0 lds\n"
    "s_waitcnt lgkmcnt(11)\n"
    "ds_read_b64_tr_b8 a[182:183], v22 offset:9216\n"
    "v_mfma_f32_16x16x128_f8f6f4 v[164:167], v[120:127], a[248:255], v[164:167] cbsz:1 blgp:1\n"
    "buffer_load_dwordx4 v29, s[40:43], s98 offen offset:1024 lds\n"
    "ds_read_b64_tr_b8 a[184:185], v23 offset:0\n"
    "ds_read_b64_tr_b8 a[186:187], v23 offset:1024\n"
    "ds_read_b64_tr_b8 a[188:189], v23 offset:8192\n"
    "buffer_load_dwordx4 v30, s[40:43], s98 offen offset:2048 lds\n"
    "s_waitcnt lgkmcnt(11)\n"
    "ds_read_b64_tr_b8 a[190:191], v23 offset:9216\n"
    "buffer_load_dwordx4 v31, s[40:43], s98 offen offset:3072 lds\n"
    "v_cmp_lt_u32 vcc, s70, v42\n"
    "s_or_b32 vcc_hi, vcc_hi, 0xffff0000\n"
    "s_cmp_eq_u64 vcc, -1\n"
    "s_cbranch_scc0 L_ey19%=\n"
    "s_cmp_lt_u32 s64, s67\n"
    "s_cbranch_scc1 L_ey19%=\n"
    "s_call_b64 s[10:11], L_se_o%=\n"
    "L_ey19%=:\n"
    "s_waitcnt vmcnt(9)\n"
    "s_barrier\n"
    "v_mfma_f32_16x16x128_f8f6f4 v[192:195], v[96:103], v[56:63], 0 cbsz:1 blgp:1\n"
    "ds_read_b128 v[120:123], v26 offset:63488\n"
    "ds_read_b128 v[124:127], v27 offset:63488\n"
    "s_add_u32 m0, s85, 49152\n"
    "ds_read_b64_tr_b8 v[176:177], v20 offset:0\n"
    "v_mfma_f32_16x16x128_f8f6f4 v[196:199], v[96:103], v[72:79], 0 cbsz:1 blgp:1\n"
    "v_readlane_b32 s98, v44, 5\n"
    "s_waitcnt lgkmcnt(11)\n"
    "ds_read_b64_tr_b8 v[178:179], v20 offset:1024\n"
    "buffer_load_dwordx4 v28, s[40:43], s99 offen offset:0 lds\n"
    "ds_read_b64_tr_b8 v[180:181], v20 offset:8192\n"
    "v_mfma_f32_16x16x128_f8f6f4 v[200:203], v[104:111], v[56:63], 0 cbsz:1 blgp:1\n"
    "buffer_load_dwordx4 v29, s[40:43], s99 offen offset:1024 lds\n"
    "ds_read_b64_tr_b8 v[182:183], v20 offset:9216\n"
    "v_mfma_f32_16x16x128_f8f6f4 v[204:207], v[104:111], v[72:79], 0 cbsz:1 blgp:1\n"
    "buffer_load_dwordx4 v30, s[40:43], s99 offen offset:2048 lds\n"
    "ds_read_b64_tr_b8 v[184:185], v21 offset:0\n"
    "s_waitcnt lgkmcnt(11)\n"
    "ds_read_b64_tr_b8 v[186:187], v21 offset:1024\n"
    "ds_read_b64_tr_b8 v[188:189], v21 offset:8192\n"
    "buffer_load_dwordx4 v31, s[40:43], s99 offen offset:3072 lds\n"
    "ds_read_b64_tr_b8 v[190:191], v21 offset:9216\n"
    "s_waitcnt vmcnt(9) lgkmcnt(10)\n"
    "s_barrier\n"
    "v_mfma_f32_16x16x128_f8f6f4 v[192:195], v[112:119], v[80:87], v[192:195] cbsz:1 blgp:1\n"
    "s_add_u32 m0, s85, 65536\n"
    "ds_read_b64_tr_b8 v[208:209], v20 offset:16384\n"
    "v_mfma_f32_16x16x128_f8f6f4 v[196:199], v[112:119], v[88:95], v[196:199] cbsz:1 blgp:1\n"
    "v_readlane_b32 s99, v44, 9\n"
    "ds_read_b64_tr_b8 v[210:211], v20 offset:17408\n"
    "buffer_load_dwordx4 v28, s[40:43], s98 offen offset:0 lds\n"
    "ds_read_b64_tr_b8 v[212:213], v20 offset:24576\n"
    "s_waitcnt lgkmcnt(11)\n"
    "v_mfma_f32_16x16x128_f8f6f4 v[200:203], v[120:127], v[80:87], v[200:203] cbsz:1 blgp:1\n"
    "buffer_load_dwordx4 v29, s[40:43], s98 offen offset:1024 lds\n"
    "ds_read_b64_tr_b8 v[214:215], v20 offset:25600\n"
    "v_mfma_f32_16x16x128_f8f6f4 v[204:207], v[120:127], v[88:95], v[204:207] cbsz:1 blgp:1\n"
    "buffer_load_dwordx4 v30, s[40:43], s98 offen offset:2048 lds\n"
    "ds_read_b64_tr_b8 v[216:217], v21 offset:16384\n"
    "ds_read_b64_tr_b8 v[218:219], v21 offset:17408\n"
    "ds_read_b64_tr_b8 v[220:221], v21 offset:24576\n"
    "buffer_load_dwordx4 v31, s[40:43], s98 offen offset:3072 lds\n"
    "s_waitcnt lgkmcnt(11)\n"
    "ds_read_b64_tr_b8 v[222:223], v21 offset:25600\n"
    "s_waitcnt vmcnt(8) lgkmcnt(8)\n"
    "s_barrier\n"
    "global_load_dword v42, v41, s[20:21] sc0 sc1\n"
    "ds_read_b64_tr_b8 v[224:225], v20 offset:32768\n"
    "s_add_u32 m0, s85, 0\n"
    "ds_read_b64_tr_b8 v[226:227], v20 offset:33792\n"
    "v_readlane_b32 s98, v44, 2\n"
    "ds_read_b64_tr_b8 v[228:229], v20 offset:40960\n"
    "buffer_load_dwordx4 v28, s[40:43], s99 offen offset:0 lds\n"
    "ds_read_b64_tr_b8 v[230:231], v20 offset:41984\n"
    "ds_read_b64_tr_b8 v[232:233], v21 offset:32768\n"
    "ds_read_b64_tr_b8 v[234:235], v21 offset:33792\n"
    "ds_read_b64_tr_b8 v[236:237], v21 offset:40960\n"
    "s_waitcnt lgkmcnt(11)\n"
    "ds_read_b64_tr_b8 v[238:239], v21 offset:41984\n"
    "buffer_load_dwordx4 v29, s[40:43], s99 offen offset:1024 lds\n"
    "buffer_load_dwordx4 v30, s[40:43], s99 offen offset:2048 lds\n"
    "buffer_load_dwordx4 v31, s[40:43], s99 offen offset:3072 lds\n"
    "v_cmp_lt_u32 vcc, s70, v42\n"
    "s_or_b32 vcc_hi, vcc_hi, 0xffff0000\n"
    "s_cmp_eq_u64 vcc, -1\n"
    "s_cbranch_scc0 L_ey22%=\n"
    "s_cmp_lt_u32 s64, s67\n"
    "s_cbranch_scc1 L_ey22%=\n"
    "s_call_b64 s[10:11], L_se_o%=\n"
    "L_ey22%=:\n"
    "s_waitcnt vmcnt(9) lgkmcnt(0)\n"
    "s_barrier\n"
    "s_add_u32 m0, s85, 16384\n"
    "ds_read_b64_tr_b8 v[248:249], v20 offset:49152\n"
    "v_readlane_b32 s99, v44, 6\n"
    "ds_read_b64_tr_b8 v[250:251], v20 offset:50176\n"
    "buffer_load_dwordx4 v28, s[40:43], s98 offen offset:0 lds\n"
    "ds_read_b64_tr_b8 v[252:253], v20 offset:57344\n"
    "buffer_load_dwordx4 v29, s[40:43], s98 offen offset:1024 lds\n"
    "ds_read_b64_tr_b8 v[254:255], v20 offset:58368\n"
    "ds_read_b64_tr_b8 v[48:49], v21 offset:49152\n"
    "ds_read_b64_tr_b8 v[50:51], v21 offset:50176\n"
    "ds_read_b64_tr_b8 v[52:53], v21 offset:57344\n"
    "ds_read_b64_tr_b8 v[54:55], v21 offset:58368\n"
    "buffer_load_dwordx4 v30, s[40:43], s98 offen offset:2048 lds\n"
    "buffer_load_dwordx4 v31, s[40:43], s98 offen offset:3072 lds\n"
    "s_add_u32 m0, s85, 32768\n"
    "v_readlane_b32 s98, v44, 10\n"
    "buffer_load_dwordx4 v28, s[40:43], s99 offen offset:0 lds\n"
    "buffer_load_dwordx4 v29, s[40:43], s99 offen offset:1024 lds\n"
    "buffer_load_dwordx4 v30, s[40:43], s99 offen offset:2048 lds\n"
    "buffer_load_dwordx4 v31, s[40:43], s99 offen offset:3072 lds\n"
    "s_waitcnt vmcnt(8)\n"
    "s_cmp_lt_u32 s64, s67\n"
    "s_cbranch_scc1 L_ex1%=\n"
    "s_call_b64 s[10:11], L_slowe%=\n"
    "s_waitcnt vmcnt(0)\n"
    "L_ex1%=:\n"
    "s_barrier\n"
    "ds_read_b128 v[96:99], v26 offset:0\n"
    "ds_read_b128 v[100:103], v27 offset:0\n"
    "ds_read_b128 v[104:107], v26 offset:32768\n"
    "ds_read_b128 v[108:111], v27 offset:32768\n"
    "ds_read_b128 v[112:115], v26 offset:8192\n"
    "ds_read_b128 v[116:119], v27 offset:8192\n"
    "s_waitcnt lgkmcnt(6)\n"
    "s_barrier\n"
    "s_waitcnt lgkmcnt(4)\n"
    "v_mfma_f32_16x16x128_f8f6f4 v[152:155], v[96:103], a[0:7], v[152:155] cbsz:1 blgp:1\n"
    "ds_read_b128 v[120:123], v26 offset:40960\n"
    "ds_read_b128 v[124:127], v27 offset:40960\n"
    "s_add_u32 m0, s85, 49152\n"
    "ds_read_b64_tr_b8 a[224:225], v22 offset:0\n"
    "v_mfma_f32_16x16x128_f8f6f4 v[156:159], v[96:103], a[8:15], v[156:159] cbsz:1 blgp:1\n"
    "v_readlane_b32 s99, v44, 3\n"
    "ds_read_b64_tr_b8 a[226:227], v22 offset:1024\n"
    "buffer_load_dwordx4 v28, s[40:43], s98 offen offset:0 lds\n"
    "ds_read_b64_tr_b8 a[228:229], v22 offset:8192\n"
    "s_waitcnt lgkmcnt(7)\n"
    "v_mfma_f32_16x16x128_f8f6f4 v[160:163], v[104:111], a[0:7], v[160:163] cbsz:1 blgp:1\n"
    "ds_read_b128 v[96:99], v26 offset:16384\n"
    "ds_read_b128 v[100:103], v27 offset:16384\n"
    "buffer_load_dwordx4 v29, s[40:43], s98 offen offset:1024 lds\n"
    "ds_read_b64_tr_b8 a[230:231], v22 offset:9216\n"
    "v_mfma_f32_16x16x128_f8f6f4 v[164:167], v[104:111], a[8:15], v[164:167] cbsz:1 blgp:1\n"
    "buffer_load_dwordx4 v30, s[40:43], s98 offen offset:2048 lds\n"
    "ds_read_b64_tr_b8 a[232:233], v23 offset:0\n"
    "ds_read_b64_tr_b8 a[234:235], v23 offset:1024\n"
    "ds_read_b64_tr_b8 a[236:237], v23 offset:8192\n"
    "s_waitcnt lgkmcnt(11)\n"
    "v_mfma_f32_16x16x128_f8f6f4 v[152:155], v[112:119], a[64:71], v[152:155] cbsz:1 blgp:1\n"
    "ds_read_b128 v[104:107], v26 offset:49152\n"
    "ds_read_b128 v[108:111], v27 offset:49152\n"
    "buffer_load_dwordx4 v31, s[40:43], s98 offen offset:3072 lds\n"
    "ds_read_b64_tr_b8 a[238:239], v23 offset:9216\n"
    "v_mfma_f32_16x16x128_f8f6f4 v[156:159], v[112:119], a[72:79], v[156:159] cbsz:1 blgp:1\n"
    "s_waitcnt lgkmcnt(12)\n"
    "v_mfma_f32_16x16x128_f8f6f4 v[160:163], v[120:127], a[64:71], v[160:163] cbsz:1 blgp:1\n"
    "ds_read_b128 v[112:115], v26 offset:2048\n"
    "ds_read_b128 v[116:119], v27 offset:2048\n"
    "v_mfma_f32_16x16x128_f8f6f4 v[164:167], v[120:127], a[72:79], v[164:167] cbsz:1 blgp:1\n"
    "s_waitcnt lgkmcnt(2)\n"
    "s_barrier\n"
    "v_mfma_f32_16x16x128_f8f6f4 v[152:155], v[96:103], a[128:135], v[152:155] cbsz:1 blgp:1\n"
    "ds_read_b128 v[120:123], v26 offset:34816\n"
    "ds_read_b128 v[124:127], v27 offset:34816\n"
    "s_add_u32 m0, s85, 65536\n"
    "ds_read_b64_tr_b8 a[240:241], v20 offset:0\n"
    "v_mfma_f32_16x16x128_f8f6f4 v[156:159], v[96:103], a[136:143], v[156:159] cbsz:1 blgp:1\n"
    "v_readlane_b32 s98, v44, 7\n"
    "ds_read_b64_tr_b8 a[242:243], v20 offset:1024\n"
    "buffer_load_dwordx4 v28, s[40:43], s99 offen offset:0 lds\n"
    "ds_read_b64_tr_b8 a[244:245], v20 offset:8192\n"
    "v_mfma_f32_16x16x128_f8f6f4 v[160:163], v[104:111], a[128:135], v[160:163] cbsz:1 blgp:1\n"
    "ds_read_b128 v[96:99], v26 offset:10240\n"
    "ds_read_b128 v[100:103], v27 offset:10240\n"
    "buffer_load_dwordx4 v29, s[40:43], s99 offen offset:1024 lds\n"
    "ds_read_b64_tr_b8 a[246:247], v20 offset:9216\n"
    "v_mfma_f32_16x16x128_f8f6f4 v[164:167], v[104:111], a[136:143], v[164:167] cbsz:1 blgp:1\n"
    "buffer_load_dwordx4 v30, s[40:43], s99 offen offset:2048 lds\n"
    "ds_read_b64_tr_b8 a[248:249], v21 offset:0\n"
    "ds_read_b64_tr_b8 a[250:251], v21 offset:1024\n"
    "ds_read_b64_tr_b8 a[252:253], v21 offset:8192\n"
    "s_waitcnt lgkmcnt(11)\n"
    "v_mfma_f32_16x16x128_f8f6f4 v[152:155], v[112:119], a[16:23], v[152:155] cbsz:1 blgp:1\n"
    "ds_read_b128 v[104:107], v26 offset:43008\n"
    "ds_read_b128 v[108:111], v27 offset:43008\n"
    "buffer_load_dwordx4 v31, s[40:43], s99 offen offset:3072 lds\n"
    "ds_read_b64_tr_b8 a[254:255], v21 offset:9216\n"
    "v_mfma_f32_16x16x128_f8f6f4 v[156:159], v[112:119], a[24:31], v[156:159] cbsz:1 blgp:1\n"
    "s_waitcnt lgkmcnt(12)\n"
    "v_mfma_f32_16x16x128_f8f6f4 v[160:163], v[120:127], a[16:23], v[160:163] cbsz:1 blgp:1\n"
    "ds_read_b128 v[112:115], v26 offset:18432\n"
    "ds_read_b128 v[116:119], v27 offset:18432\n"
    "v_mfma_f32_16x16x128_f8f6f4 v[164:167], v[120:127], a[24:31], v[164:167] cbsz:1 blgp:1\n"
    "s_waitcnt vmcnt(12) lgkmcnt(2)\n"
    "s_barrier\n"
    "v_mfma_f32_16x16x128_f8f6f4 v[152:155], v[96:103], a[80:87], v[152:155] cbsz:1 blgp:1\n"
    "ds_read_b128 v[120:123], v26 offset:51200\n"
    "ds_read_b128 v[124:127], v27 offset:51200\n"
    "s_add_u32 m0, s85, 0\n"
    "ds_read_b64_tr_b8 a[0:1], v20 offset:16384\n"
    "v_mfma_f32_16x16x128_f8f6f4 v[156:159], v[96:103], a[88:95], v[156:159] cbsz:1 blgp:1\n"
    "v_readlane_b32 s99, v44, 11\n"
    "ds_read_b64_tr_b8 a[2:3], v20 offset:17408\n"
    "buffer_load_dwordx4 v28, s[40:43], s98 offen offset:0 lds\n"
    "ds_read_b64_tr_b8 a[4:5], v20 offset:24576\n"
    "v_mfma_f32_16x16x128_f8f6f4 v[160:163], v[104:111], a[80:87], v[160:163] cbsz:1 blgp:1\n"
    "ds_read_b128 v[96:99], v26 offset:24576\n"
    "ds_read_b128 v[100:103], v27 offset:24576\n"
    "buffer_load_dwordx4 v29, s[40:43], s98 offen offset:1024 lds\n"
    "ds_read_b64_tr_b8 a[6:7], v20 offset:25600\n"
    "v_mfma_f32_16x16x128_f8f6f4 v[164:167], v[104:111], a[88:95], v[164:167] cbsz:1 blgp:1\n"
    "buffer_load_dwordx4 v30, s[40:43], s98 offen offset:2048 lds\n"
    "ds_read_b64_tr_b8 a[8:9], v21 offset:16384\n"
    "ds_read_b64_tr_b8 a[10:11], v21 offset:17408\n"
    "ds_read_b64_tr_b8 a[12:13], v21 offset:24576\n"
    "s_waitcnt lgkmcnt(11)\n"
    "v_mfma_f32_16x16x128_f8f6f4 v[152:155], v[112:119], a[144:151], v[152:155] cbsz:1 blgp:1\n"
    "ds_read_b128 v[104:107], v26 offset:57344\n"
    "ds_read_b128 v[108:111], v27 offset:57344\n"
    "buffer_load_dwordx4 v31, s[40:43], s98 offen offset:3072 lds\n"
    "ds_read_b64_tr_b8 a[14:15], v21 offset:25600\n"
    "v_mfma_f32_16x16x128_f8f6f4 v[156:159], v[112:119], a[152:159], v[156:159] cbsz:1 blgp:1\n"
    "s_waitcnt lgkmcnt(12)\n"
    "v_mfma_f32_16x16x128_f8f6f4 v[160:163], v[120:127], a[144:151], v[160:163] cbsz:1 blgp:1\n"
    "ds_read_b128 v[112:115], v26 offset:26624\n"
    "ds_read_b128 v[116:119], v27 offset:26624\n"
    "v_mfma_f32_16x16x128_f8f6f4 v[164:167], v[120:127], a[152:159], v[164:167] cbsz:1 blgp:1\n"
    "s_waitcnt vmcnt(12) lgkmcnt(2)\n"
    "s_barrier\n"
    "v_mfma_f32_16x16x128_f8f6f4 v[152:155], v[96:103], a[192:199], v[152:155] cbsz:1 blgp:1\n"
    "ds_read_b128 v[120:123], v26 offset:59392\n"
    "ds_read_b128 v[124:127], v27 offset:59392\n"
    "s_add_u32 m0, s85, 16384\n"
    "ds_read_b64_tr_b8 a[64:65], v20 offset:32768\n"
    "v_mfma_f32_16x16x128_f8f6f4 v[156:159], v[96:103], a[200:207], v[156:159] cbsz:1 blgp:1\n"
    "v_readlane_b32 s98, v44, 14\n"
    "ds_read_b64_tr_b8 a[66:67], v20 offset:33792\n"
    "buffer_load_dwordx4 v28, s[40:43], s99 offen offset:0 lds\n"
    "ds_read_b64_tr_b8 a[68:69], v20 offset:40960\n"
    "v_mfma_f32_16x16x128_f8f6f4 v[160:163], v[104:111], a[192:199], v[160:163] cbsz:1 blgp:1\n"
    "ds_read_b128 v[96:99], v26 offset:4096\n"
    "ds_read_b128 v[100:103], v27 offset:4096\n"
    "buffer_load_dwordx4 v29, s[40:43], s99 offen offset:1024 lds\n"
    "ds_read_b64_tr_b8 a[70:71], v20 offset:41984\n"
    "v_mfma_f32_16x16x128_f8f6f4 v[164:167], v[104:111], a[200:207], v[164:167] cbsz:1 blgp:1\n"
    "buffer_load_dwordx4 v30, s[40:43], s99 offen offset:2048 lds\n"
    "ds_read_b64_tr_b8 a[72:73], v21 offset:32768\n"
    "ds_read_b64_tr_b8 a[74:75], v21 offset:33792\n"
    "ds_read_b64_tr_b8 a[76:77], v21 offset:40960\n"
    "s_waitcnt lgkmcnt(11)\n"
    "v_mfma_f32_16x16x128_f8f6f4 v[152:155], v[112:119], a[208:215], v[152:155] cbsz:1 blgp:1\n"
    "ds_read_b128 v[104:107], v26 offset:36864\n"
    "ds_read_b128 v[108:111], v27 offset:36864\n"
    "buffer_load_dwordx4 v31, s[40:43], s99 offen offset:3072 lds\n"
    "ds_read_b64_tr_b8 a[78:79], v21 offset:41984\n"
    "v_mfma_f32_16x16x128_f8f6f4 v[156:159], v[112:119], a[216:223], v[156:159] cbsz:1 blgp:1\n"
    "s_waitcnt lgkmcnt(12)\n"
    "v_mfma_f32_16x16x128_f8f6f4 v[160:163], v[120:127], a[208:215], v[160:163] cbsz:1 blgp:1\n"
    "ds_read_b128 v[112:115], v26 offset:12288\n"
    "ds_read_b128 v[116:119], v27 offset:12288\n"
    "v_mfma_f32_16x16x128_f8f6f4 v[164:167], v[120:127], a[216:223], v[164:167] cbsz:1 blgp:1\n"
    "s_waitcnt vmcnt(12) lgkmcnt(2)\n"
    "s_barrier\n"
    "v_mfma_f32_16x16x128_f8f6f4 v[152:155], v[96:103], a[32:39], v[152:155] cbsz:1 blgp:1\n"
    "ds_read_b128 v[120:123], v26 offset:45056\n"
    "ds_read_b128 v[124:127], v27 offset:45056\n"
    "ds_read_b64_tr_b8 a[128:129], v20 offset:49152\n"
    "ds_read_b64_tr_b8 a[130:131], v20 offset:50176\n"
    "v_mfma_f32_16x16x128_f8f6f4 v[156:159], v[96:103], a[40:47], v[156:159] cbsz:1 blgp:1\n"
    "ds_read_b64_tr_b8 a[132:133], v20 offset:57344\n"
    "ds_read_b64_tr_b8 a[134:135], v20 offset:58368\n"
    "ds_read_b64_tr_b8 a[136:137], v21 offset:49152\n"
    "ds_read_b64_tr_b8 a[138:139], v21 offset:50176\n"
    "v_mfma_f32_16x16x128_f8f6f4 v[160:163], v[104:111], a[32:39], v[160:163] cbsz:1 blgp:1\n"
    "ds_read_b128 v[96:99], v26 offset:20480\n"
    "ds_read_b128 v[100:103], v27 offset:20480\n"
    "ds_read_b64_tr_b8 a[140:141], v21 offset:57344\n"
    "ds_read_b64_tr_b8 a[142:143], v21 offset:58368\n"
    "v_mfma_f32_16x16x128_f8f6f4 v[164:167], v[104:111], a[40:47], v[164:167] cbsz:1 blgp:1\n"
    "s_waitcnt lgkmcnt(12)\n"
    "v_mfma_f32_16x16x128_f8f6f4 v[152:155], v[112:119], a[96:103], v[152:155] cbsz:1 blgp:1\n"
    "ds_read_b128 v[104:107], v26 offset:53248\n"
    "ds_read_b128 v[108:111], v27 offset:53248\n"
    "v_mfma_f32_16x16x128_f8f6f4 v[156:159], v[112:119], a[104:111], v[156:159] cbsz:1 blgp:1\n"
    "s_waitcnt lgkmcnt(12)\n"
    "v_mfma_f32_16x16x128_f8f6f4 v[160:163], v[120:127], a[96:103], v[160:163] cbsz:1 blgp:1\n"
    "ds_read_b128 v[112:115], v26 offset:6144\n"
    "ds_read_b128 v[116:119], v27 offset:6144\n"
    "v_mfma_f32_16x16x128_f8f6f4 v[164:167], v[120:127], a[104:111], v[164:167] cbsz:1 blgp:1\n"
    "s_waitcnt vmcnt(8) lgkmcnt(4)\n"
    "s_barrier\n"
    "v_mfma_f32_16x16x128_f8f6f4 v[152:155], v[96:103], a[160:167], v[152:155] cbsz:1 blgp:1\n"
    "ds_read_b128 v[120:123], v26 offset:38912\n"
    "ds_read_b128 v[124:127], v27 offset:38912\n"
    "ds_read_b64_tr_b8 a[16:17], v22 offset:0\n"
    "ds_read_b64_tr_b8 a[18:19], v22 offset:1024\n"
    "v_mfma_f32_16x16x128_f8f6f4 v[156:159], v[96:103], a[168:175], v[156:159] cbsz:1 blgp:1\n"
    "ds_read_b64_tr_b8 a[20:21], v22 offset:8192\n"
    "ds_read_b64_tr_b8 a[22:23], v22 offset:9216\n"
    "ds_read_b64_tr_b8 a[24:25], v23 offset:0\n"
    "ds_read_b64_tr_b8 a[26:27], v23 offset:1024\n"
    "s_waitcnt lgkmcnt(10)\n"
    "v_mfma_f32_16x16x128_f8f6f4 v[160:163], v[104:111], a[160:167], v[160:163] cbsz:1 blgp:1\n"
    "ds_read_b128 v[96:99], v26 offset:14336\n"
    "ds_read_b128 v[100:103], v27 offset:14336\n"
    "ds_read_b64_tr_b8 a[28:29], v23 offset:8192\n"
    "ds_read_b64_tr_b8 a[30:31], v23 offset:9216\n"
    "v_mfma_f32_16x16x128_f8f6f4 v[164:167], v[104:111], a[168:175], v[164:167] cbsz:1 blgp:1\n"
    "s_waitcnt lgkmcnt(12)\n"
    "v_mfma_f32_16x16x128_f8f6f4 v[152:155], v[112:119], a[48:55], v[152:155] cbsz:1 blgp:1\n"
    "ds_read_b128 v[104:107], v26 offset:47104\n"
    "ds_read_b128 v[108:111], v27 offset:47104\n"
    "v_mfma_f32_16x16x128_f8f6f4 v[156:159], v[112:119], a[56:63], v[156:159] cbsz:1 blgp:1\n"
    "s_waitcnt lgkmcnt(12)\n"
    "v_mfma_f32_16x16x128_f8f6f4 v[160:163], v[120:127], a[48:55], v[160:163] cbsz:1 blgp:1\n"
    "ds_read_b128 v[112:115], v26 offset:22528\n"
    "ds_read_b128 v[116:119], v27 offset:22528\n"
    "v_mfma_f32_16x16x128_f8f6f4 v[164:167], v[120:127], a[56:63], v[164:167] cbsz:1 blgp:1\n"
    "s_waitcnt vmcnt(4) lgkmcnt(4)\n"
    "s_barrier\n"
    "v_mfma_f32_16x16x128_f8f6f4 v[152:155], v[96:103], a[112:119], v[152:155] cbsz:1 blgp:1\n"
    "ds_read_b128 v[120:123], v26 offset:55296\n"
    "ds_read_b128 v[124:127], v27 offset:55296\n"
    "ds_read_b64_tr_b8 a[80:81], v20 offset:0\n"
    "ds_read_b64_tr_b8 a[82:83], v20 offset:1024\n"
    "v_mfma_f32_16x16x128_f8f6f4 v[156:159], v[96:103], a[120:127], v[156:159] cbsz:1 blgp:1\n"
    "ds_read_b64_tr_b8 a[84:85], v20 offset:8192\n"
    "ds_read_b64_tr_b8 a[86:87], v20 offset:9216\n"
    "ds_read_b64_tr_b8 a[88:89], v21 offset:0\n"
    "ds_read_b64_tr_b8 a[90:91], v21 offset:1024\n"
    "s_waitcnt lgkmcnt(10)\n"
    "v_mfma_f32_16x16x128_f8f6f4 v[160:163], v[104:111], a[112:119], v[160:163] cbsz:1 blgp:1\n"
    "ds_read_b128 v[96:99], v26 offset:24576\n"
    "ds_read_b128 v[100:103], v27 offset:24576\n"
    "ds_read_b64_tr_b8 a[92:93], v21 offset:8192\n"
    "ds_read_b64_tr_b8 a[94:95], v21 offset:9216\n"
    "v_mfma_f32_16x16x128_f8f6f4 v[164:167], v[104:111], a[120:127], v[164:167] cbsz:1 blgp:1\n"
    "s_waitcnt lgkmcnt(12)\n"
    "v_mfma_f32_16x16x128_f8f6f4 v[152:155], v[112:119], a[176:183], v[152:155] cbsz:1 blgp:1\n"
    "ds_read_b128 v[104:107], v26 offset:57344\n"
    "ds_read_b128 v[108:111], v27 offset:57344\n"
    "v_mfma_f32_16x16x128_f8f6f4 v[156:159], v[112:119], a[184:191], v[156:159] cbsz:1 blgp:1\n"
    "s_waitcnt lgkmcnt(12)\n"
    "v_mfma_f32_16x16x128_f8f6f4 v[160:163], v[120:127], a[176:183], v[160:163] cbsz:1 blgp:1\n"
    "ds_read_b128 v[112:115], v26 offset:26624\n"
    "ds_read_b128 v[116:119], v27 offset:26624\n"
    "v_mfma_f32_16x16x128_f8f6f4 v[164:167], v[120:127], a[184:191], v[164:167] cbsz:1 blgp:1\n"
    "s_waitcnt vmcnt(0) lgkmcnt(4)\n"
    "s_barrier\n"
    "ds_read_b64_tr_b8 a[144:145], v20 offset:16384\n"
    "ds_read_b64_tr_b8 a[146:147], v20 offset:17408\n"
    "ds_read_b64_tr_b8 a[148:149], v20 offset:24576\n"
    "ds_read_b64_tr_b8 a[150:151], v20 offset:25600\n"
    "ds_read_b64_tr_b8 a[152:153], v21 offset:16384\n"
    "ds_read_b64_tr_b8 a[154:155], v21 offset:17408\n"
    "ds_read_b64_tr_b8 a[156:157], v21 offset:24576\n"
    "ds_read_b64_tr_b8 a[158:159], v21 offset:25600\n"
    "v_mfma_f32_16x16x128_f8f6f4 v[192:195], v[96:103], v[128:135], v[192:195] cbsz:1 blgp:1\n"
    "ds_read_b128 v[120:123], v26 offset:59392\n"
    "ds_read_b128 v[124:127], v27 offset:59392\n"
    "v_mfma_f32_16x16x128_f8f6f4 v[196:199], v[96:103], v[136:143], v[196:199] cbsz:1 blgp:1\n"
    "s_waitcnt lgkmcnt(12)\n"
    "v_mfma_f32_16x16x128_f8f6f4 v[200:203], v[104:111], v[128:135], v[200:203] cbsz:1 blgp:1\n"
    "ds_read_b128 v[96:99], v26 offset:0\n"
    "ds_read_b128 v[100:103], v27 offset:0\n"
    "v_mfma_f32_16x16x128_f8f6f4 v[204:207], v[104:111], v[136:143], v[204:207] cbsz:1 blgp:1\n"
    "s_waitcnt lgkmcnt(12)\n"
    "v_mfma_f32_16x16x128_f8f6f4 v[192:195], v[112:119], v[144:151], v[192:195] cbsz:1 blgp:1\n"
    "ds_read_b128 v[104:107], v26 offset:32768\n"
    "ds_read_b128 v[108:111], v27 offset:32768\n"
    "v_mfma_f32_16x16x128_f8f6f4 v[196:199], v[112:119], v[168:175], v[196:199] cbsz:1 blgp:1\n"
    "s_waitcnt lgkmcnt(4)\n"
    "v_mfma_f32_16x16x128_f8f6f4 v[200:203], v[120:127], v[144:151], v[200:203] cbsz:1 blgp:1\n"
    "ds_read_b128 v[112:115], v26 offset:8192\n"
    "ds_read_b128 v[116:119], v27 offset:8192\n"
    "v_mul_f32 v152, v43, v152\n"
    "v_mul_f32 v153, v43, v153\n"
    "v_mfma_f32_16x16x128_f8f6f4 v[204:207], v[120:127], v[168:175], v[204:207] cbsz:1 blgp:1\n"
    "v_mul_f32 v154, v43, v154\n"
    "v_mul_f32 v155, v43, v155\n"
    "v_mul_f32 v156, v43, v156\n"
    "v_mul_f32 v157, v43, v157\n"
    "s_waitcnt lgkmcnt(4)\n"
    "v_mfma_f32_16x16x128_f8f6f4 v[192:195], v[96:103], v[176:183], v[192:195] cbsz:1 blgp:1\n"
    "ds_read_b128 v[120:123], v26 offset:40960\n"
    "ds_read_b128 v[124:127], v27 offset:40960\n"
    "v_mul_f32 v158, v43, v158\n"
    "v_mul_f32 v159, v43, v159\n"
    "v_mfma_f32_16x16x128_f8f6f4 v[196:199], v[96:103], v[184:191], v[196:199] cbsz:1 blgp:1\n"
    "global_store_dwordx4 v40, v[152:155], s[60:61] offset:-4096 sc0 sc1\n"
    "s_waitcnt lgkmcnt(4)\n"
    "v_mfma_f32_16x16x128_f8f6f4 v[200:203], v[104:111], v[176:183], v[200:203] cbsz:1 blgp:1\n"
    "ds_read_b128 v[96:99], v26 offset:16384\n"
    "ds_read_b128 v[100:103], v27 offset:16384\n"
    "global_store_dwordx4 v40, v[156:159], s[60:61] offset:-3072 sc0 sc1\n"
    "v_mul_f32 v160, v43, v160\n"
    "v_mfma_f32_16x16x128_f8f6f4 v[204:207], v[104:111], v[184:191], v[204:207] cbsz:1 blgp:1\n"
    "v_mul_f32 v161, v43, v161\n"
    "v_mul_f32 v162, v43, v162\n"
    "v_mul_f32 v163, v43, v163\n"
    "v_mul_f32 v164, v43, v164\n"
    "s_waitcnt lgkmcnt(4)\n"
    "v_mfma_f32_16x16x128_f8f6f4 v[192:195], v[112:119], v[208:215], v[192:195] cbsz:1 blgp:1\n"
    "ds_read_b128 v[104:107], v26 offset:49152\n"
    "ds_read_b128 v[108:111], v27 offset:49152\n"
    "v_mul_f32 v165, v43, v165\n"
    "v_mul_f32 v166, v43, v166\n"
    "v_mfma_f32_16x16x128_f8f6f4 v[196:199], v[112:119], v[216:223], v[196:199] cbsz:1 blgp:1\n"
    "v_mul_f32 v167, v43, v167\n"
    "global_store_dwordx4 v40, v[160:163], s[60:61] offset:-2048 sc0 sc1\n"
    "s_waitcnt lgkmcnt(4)\n"
    "v_mfma_f32_16x16x128_f8f6f4 v[200:203], v[120:127], v[208:215], v[200:203] cbsz:1 blgp:1\n"
    "ds_read_b128 v[112:115], v26 offset:2048\n"
    "ds_read_b128 v[116:119], v27 offset:2048\n"
    "global_store_dwordx4 v40, v[164:167], s[60:61] offset:-1024 sc0 sc1\n"
    "v_mfma_f32_16x16x128_f8f6f4 v[204:207], v[120:127], v[216:223], v[204:207] cbsz:1 blgp:1\n"
    "s_waitcnt lgkmcnt(4)\n"
    "v_mfma_f32_16x16x128_f8f6f4 v[192:195], v[96:103], v[224:231], v[192:195] cbsz:1 blgp:1\n"
    "ds_read_b128 v[120:123], v26 offset:34816\n"
    "ds_read_b128 v[124:127], v27 offset:34816\n"
    "v_mfma_f32_16x16x128_f8f6f4 v[196:199], v[96:103], v[232:239], v[196:199] cbsz:1 blgp:1\n"
    "s_waitcnt lgkmcnt(4)\n"
    "v_mfma_f32_16x16x128_f8f6f4 v[200:203], v[104:111], v[224:231], v[200:203] cbsz:1 blgp:1\n"
    "ds_read_b128 v[96:99], v26 offset:10240\n"
    "ds_read_b128 v[100:103], v27 offset:10240\n"
    "v_mfma_f32_16x16x128_f8f6f4 v[204:207], v[104:111], v[232:239], v[204:207] cbsz:1 blgp:1\n"
    "s_waitcnt lgkmcnt(4)\n"
    "v_mfma_f32_16x16x128_f8f6f4 v[192:195], v[112:119], v[248:255], v[192:195] cbsz:1 blgp:1\n"
    "ds_read_b128 v[104:107], v26 offset:43008\n"
    "ds_read_b128 v[108:111], v27 offset:43008\n"
    "v_mfma_f32_16x16x128_f8f6f4 v[196:199], v[112:119], v[48:55], v[196:199] cbsz:1 blgp:1\n"
    "s_waitcnt lgkmcnt(4)\n"
    "v_mfma_f32_16x16x128_f8f6f4 v[200:203], v[120:127], v[248:255], v[200:203] cbsz:1 blgp:1\n"
    "ds_read_b128 v[112:115], v26 offset:18432\n"
    "ds_read_b128 v[116:119], v27 offset:18432\n"
    "v_mfma_f32_16x16x128_f8f6f4 v[204:207], v[120:127], v[48:55], v[204:207] cbsz:1 blgp:1\n"
    "s_waitcnt lgkmcnt(4)\n"
    "v_mfma_f32_16x16x128_f8f6f4 v[192:195], v[96:103], a[224:231], v[192:195] cbsz:1 blgp:1\n"
    "ds_read_b128 v[120:123], v26 offset:51200\n"
    "ds_read_b128 v[124:127], v27 offset:51200\n"
    "v_mfma_f32_16x16x128_f8f6f4 v[196:199], v[96:103], a[232:239], v[196:199] cbsz:1 blgp:1\n"
    "s_waitcnt lgkmcnt(4)\n"
    "v_mfma_f32_16x16x128_f8f6f4 v[200:203], v[104:111], a[224:231], v[200:203] cbsz:1 blgp:1\n"
    "ds_read_b128 v[96:99], v26 offset:4096\n"
    "ds_read_b128 v[100:103], v27 offset:4096\n"
    "v_mfma_f32_16x16x128_f8f6f4 v[204:207], v[104:111], a[232:239], v[204:207] cbsz:1 blgp:1\n"
    "s_waitcnt vmcnt(0)\n"
    "s_barrier\n"
    "v_add_u32 v46, 1, v46\n"
    "buffer_store_dword v46, v40, s[56:59], 0 offen sc0 sc1\n"
    "s_waitcnt lgkmcnt(4)\n"
    "v_mfma_f32_16x16x128_f8f6f4 v[192:195], v[112:119], a[240:247], v[192:195] cbsz:1 blgp:1\n"
    "ds_read_b128 v[104:107], v26 offset:36864\n"
    "ds_read_b128 v[108:111], v27 offset:36864\n"
    "v_mfma_f32_16x16x128_f8f6f4 v[196:199], v[112:119], a[248:255], v[196:199] cbsz:1 blgp:1\n"
    "s_waitcnt lgkmcnt(4)\n"
    "v_mfma_f32_16x16x128_f8f6f4 v[200:203], v[120:127], a[240:247], v[200:203] cbsz:1 blgp:1\n"
    "ds_read_b128 v[112:115], v26 offset:12288\n"
    "ds_read_b128 v[116:119], v27 offset:12288\n"
    "v_mfma_f32_16x16x128_f8f6f4 v[204:207], v[120:127], a[248:255], v[204:207] cbsz:1 blgp:1\n"
    "s_waitcnt lgkmcnt(4)\n"
    "v_mfma_f32_16x16x128_f8f6f4 v[192:195], v[96:103], a[0:7], v[192:195] cbsz:1 blgp:1\n"
    "ds_read_b128 v[120:123], v26 offset:45056\n"
    "ds_read_b128 v[124:127], v27 offset:45056\n"
    "v_mfma_f32_16x16x128_f8f6f4 v[196:199], v[96:103], a[8:15], v[196:199] cbsz:1 blgp:1\n"
    "s_waitcnt lgkmcnt(4)\n"
    "v_mfma_f32_16x16x128_f8f6f4 v[200:203], v[104:111], a[0:7], v[200:203] cbsz:1 blgp:1\n"
    "ds_read_b128 v[96:99], v26 offset:20480\n"
    "ds_read_b128 v[100:103], v27 offset:20480\n"
    "v_mfma_f32_16x16x128_f8f6f4 v[204:207], v[104:111], a[8:15], v[204:207] cbsz:1 blgp:1\n"
    "s_waitcnt lgkmcnt(4)\n"
    "v_mfma_f32_16x16x128_f8f6f4 v[192:195], v[112:119], a[64:71], v[192:195] cbsz:1 blgp:1\n"
    "ds_read_b128 v[104:107], v26 offset:53248\n"
    "ds_read_b128 v[108:111], v27 offset:53248\n"
    "v_mfma_f32_16x16x128_f8f6f4 v[196:199], v[112:119], a[72:79], v[196:199] cbsz:1 blgp:1\n"
    "s_waitcnt lgkmcnt(4)\n"
    "v_mfma_f32_16x16x128_f8f6f4 v[200:203], v[120:127], a[64:71], v[200:203] cbsz:1 blgp:1\n"
    "ds_read_b128 v[112:115], v26 offset:6144\n"
    "ds_read_b128 v[116:119], v27 offset:6144\n"
    "v_mfma_f32_16x16x128_f8f6f4 v[204:207], v[120:127], a[72:79], v[204:207] cbsz:1 blgp:1\n"
    "s_waitcnt lgkmcnt(4)\n"
    "v_mfma_f32_16x16x128_f8f6f4 v[192:195], v[96:103], a[128:135], v[192:195] cbsz:1 blgp:1\n"
    "ds_read_b128 v[120:123], v26 offset:38912\n"
    "ds_read_b128 v[124:127], v27 offset:38912\n"
    "v_mfma_f32_16x16x128_f8f6f4 v[196:199], v[96:103], a[136:143], v[196:199] cbsz:1 blgp:1\n"
    "s_waitcnt lgkmcnt(4)\n"
    "v_mfma_f32_16x16x128_f8f6f4 v[200:203], v[104:111], a[128:135], v[200:203] cbsz:1 blgp:1\n"
    "ds_read_b128 v[96:99], v26 offset:14336\n"
    "ds_read_b128 v[100:103], v27 offset:14336\n"
    "v_mfma_f32_16x16x128_f8f6f4 v[204:207], v[104:111], a[136:143], v[204:207] cbsz:1 blgp:1\n"
    "s_waitcnt lgkmcnt(4)\n"
    "v_mfma_f32_16x16x128_f8f6f4 v[192:195], v[112:119], a[16:23], v[192:195] cbsz:1 blgp:1\n"
    "ds_read_b128 v[104:107], v26 offset:47104\n"
    "ds_read_b128 v[108:111], v27 offset:47104\n"
    "v_mfma_f32_16x16x128_f8f6f4 v[196:199], v[112:119], a[24:31], v[196:199] cbsz:1 blgp:1\n"
    "s_waitcnt lgkmcnt(4)\n"
    "v_mfma_f32_16x16x128_f8f6f4 v[200:203], v[120:127], a[16:23], v[200:203] cbsz:1 blgp:1\n"
    "ds_read_b128 v[112:115], v26 offset:22528\n"
    "ds_read_b128 v[116:119], v27 offset:22528\n"
    "v_mfma_f32_16x16x128_f8f6f4 v[204:207], v[120:127], a[24:31], v[204:207] cbsz:1 blgp:1\n"
    "s_call_b64 s[10:11], L_slowp%=\n"
    "s_barrier\n"
    "buffer_load_dwordx4 v[152:155], v40, s[48:51], s93 offen offset:0 sc0 sc1\n"
    "buffer_load_dwordx4 v[156:159], v40, s[48:51], s93 offen offset:1024 sc0 sc1\n"
    "s_mov_b32 s93, 0x800\n"
    "buffer_load_dwordx4 v[160:163], v40, s[48:51], s93 offen offset:0 sc0 sc1\n"
    "buffer_load_dwordx4 v[164:167], v40, s[48:51], s93 offen offset:1024 sc0 sc1\n"
    "v_mfma_f32_16x16x128_f8f6f4 v[192:195], v[96:103], a[80:87], v[192:195] cbsz:1 blgp:1\n"
    "ds_read_b128 v[120:123], v26 offset:55296\n"
    "ds_read_b128 v[124:127], v27 offset:55296\n"
    "v_mfma_f32_16x16x128_f8f6f4 v[196:199], v[96:103], a[88:95], v[196:199] cbsz:1 blgp:1\n"
    "v_mfma_f32_16x16x128_f8f6f4 v[200:203], v[104:111], a[80:87], v[200:203] cbsz:1 blgp:1\n"
    "v_mfma_f32_16x16x128_f8f6f4 v[204:207], v[104:111], a[88:95], v[204:207] cbsz:1 blgp:1\n"
    "v_mfma_f32_16x16x128_f8f6f4 v[192:195], v[112:119], a[144:151], v[192:195] cbsz:1 blgp:1\n"
    "v_mfma_f32_16x16x128_f8f6f4 v[196:199], v[112:119], a[152:159], v[196:199] cbsz:1 blgp:1\n"
    "s_waitcnt lgkmcnt(0)\n"
    "v_mfma_f32_16x16x128_f8f6f4 v[200:203], v[120:127], a[144:151], v[200:203] cbsz:1 blgp:1\n"
    "v_mfma_f32_16x16x128_f8f6f4 v[204:207], v[120:127], a[152:159], v[204:207] cbsz:1 blgp:1\n"
    "s_waitcnt vmcnt(0)\n"
    "s_nop 10\n"
    "v_fma_f32 v192, v192, v43, v152\n"
    "v_fma_f32 v196, v196, v43, v156\n"
    "v_fma_f32 v193, v193, v43, v153\n"
    "v_fma_f32 v197, v197, v43, v157\n"
    "v_fma_f32 v194, v194, v43, v154\n"
    "v_fma_f32 v198, v198, v43, v158\n"
    "v_fma_f32 v195, v195, v43, v155\n"
    "v_fma_f32 v199, v199, v43, v159\n"
    "v_permlane16_swap_b32 v192, v196\n"
    "v_permlane16_swap_b32 v193, v197\n"
    "v_permlane16_swap_b32 v194, v198\n"
    "v_permlane16_swap_b32 v195, v199\n"
    "global_store_dword v64, v192, s[22:23] sc0 sc1\n"
    "global_store_dword v68, v196, s[22:23] sc0 sc1\n"
    "global_store_dword v65, v193, s[22:23] sc0 sc1\n"
    "global_store_dword v69, v197, s[22:23] sc0 sc1\n"
    "global_store_dword v66, v194, s[22:23] sc0 sc1\n"
    "global_store_dword v70, v198, s[22:23] sc0 sc1\n"
    "global_store_dword v67, v195, s[22:23] sc0 sc1\n"
    "global_store_dword v71, v199, s[22:23] sc0 sc1\n"
    "s_add_u32 s22, s22, 0x40000\n"
    "s_addc_u32 s23, s23, 0\n"
    "v_fma_f32 v200, v200, v43, v160\n"
    "v_fma_f32 v204, v204, v43, v164\n"
    "v_fma_f32 v201, v201, v43, v161\n"
    "v_fma_f32 v205, v205, v43, v165\n"
    "v_fma_f32 v202, v202, v43, v162\n"
    "v_fma_f32 v206, v206, v43, v166\n"
    "v_fma_f32 v203, v203, v43, v163\n"
    "v_fma_f32 v207, v207, v43, v167\n"
    "v_permlane16_swap_b32 v200, v204\n"
    "v_permlane16_swap_b32 v201, v205\n"
    "v_permlane16_swap_b32 v202, v206\n"
    "v_permlane16_swap_b32 v203, v207\n"
    "global_store_dword v64, v200, s[22:23] sc0 sc1\n"
    "global_store_dword v68, v204, s[22:23] sc0 sc1\n"
    "global_store_dword v65, v201, s[22:23] sc0 sc1\n"
    "global_store_dword v69, v205, s[22:23] sc0 sc1\n"
    "global_store_dword v66, v202, s[22:23] sc0 sc1\n"
    "global_store_dword v70, v206, s[22:23] sc0 sc1\n"
    "global_store_dword v67, v203, s[22:23] sc0 sc1\n"
    "global_store_dword v71, v207, s[22:23] sc0 sc1\n"
    "s_endpgm\n"
    "L_c0%=:\n"
    "s_call_b64 s[10:11], L_slowx%=\n"
    "s_branch L_r0%=\n"
    "L_c1%=:\n"
    "s_waitcnt vmcnt(0)\n"
    "s_cmp_lt_u32 s64, s67\n"
    "s_cbranch_scc1 L_r1%=\n"
    "s_call_b64 s[10:11], L_slowx%=\n"
    "s_branch L_r1%=\n"
    "L_e1%=:\n"
    "s_call_b64 s[10:11], L_burst%=\n"
    "s_branch L_f1%=\n"
    "L_u1_0%=:\n"
    "s_call_b64 s[10:11], L_urg%=\n"
    "s_branch L_v1_0%=\n"
    "L_c2%=:\n"
    "s_waitcnt vmcnt(0)\n"
    "s_cmp_lt_u32 s64, s67\n"
    "s_cbranch_scc1 L_r2%=\n"
    "s_call_b64 s[10:11], L_slowx%=\n"
    "s_branch L_r2%=\n"
    "L_e2%=:\n"
    "s_call_b64 s[10:11], L_burst%=\n"
    "s_branch L_f2%=\n"
    "L_u2_0%=:\n"
    "s_call_b64 s[10:11], L_urg%=\n"
    "s_branch L_v2_0%=\n"
    "L_c3%=:\n"
    "s_waitcnt vmcnt(0)\n"
    "s_cmp_lt_u32 s64, s67\n"
    "s_cbranch_scc1 L_r3%=\n"
    "s_call_b64 s[10:11], L_slowx%=\n"
    "s_branch L_r3%=\n"
    "L_e3%=:\n"
    "s_call_b64 s[10:11], L_burst%=\n"
    "s_branch L_f3%=\n"
    "L_u3_0%=:\n"
    "s_call_b64 s[10:11], L_urg%=\n"
    "s_branch L_v3_0%=\n"
    "L_c4%=:\n"
    "s_waitcnt vmcnt(0)\n"
    "s_cmp_lt_u32 s64, s67\n"
    "s_cbranch_scc1 L_r4%=\n"
    "s_call_b64 s[10:11], L_slowx%=\n"
    "s_branch L_r4%=\n"
    "L_e4%=:\n"
    "s_call_b64 s[10:11], L_burst%=\n"
    "s_branch L_f4%=\n"
    "L_u4_0%=:\n"
    "s_call_b64 s[10:11], L_urg%=\n"
    "s_branch L_v4_0%=\n"
    "L_p4%=:\n"
    "s_call_b64 s[10:11], L_slowq%=\n"
    "s_branch L_q4%=\n"
    "L_c5%=:\n"
    "s_waitcnt vmcnt(0)\n"
    "s_cmp_lt_u32 s64, s67\n"
    "s_cbranch_scc1 L_r5%=\n"
    "s_call_b64 s[10:11], L_slowx%=\n"
    "s_branch L_r5%=\n"
    "L_e5%=:\n"
    "s_call_b64 s[10:11], L_burst%=\n"
    "s_branch L_f5%=\n"
    "L_u5_0%=:\n"
    "s_call_b64 s[10:11], L_urg%=\n"
    "s_branch L_v5_0%=\n"
    "L_p5%=:\n"
    "s_call_b64 s[10:11], L_slowq%=\n"
    "s_branch L_q5%=\n"
    "L_c6%=:\n"
    "s_waitcnt vmcnt(0)\n"
    "s_cmp_lt_u32 s64, s67\n"
    "s_cbranch_scc1 L_r6%=\n"
    "s_call_b64 s[10:11], L_slowx%=\n"
    "s_branch L_r6%=\n"
    "L_e6%=:\n"
    "s_call_b64 s[10:11], L_burst%=\n"
    "s_branch L_f6%=\n"
    "L_u6_0%=:\n"
    "s_call_b64 s[10:11], L_urg%=\n"
    "s_branch L_v6_0%=\n"
    "L_p6%=:\n"
    "s_call_b64 s[10:11], L_slowq%=\n"
    "s_branch L_q6%=\n"
    "L_c7%=:\n"
    "s_waitcnt vmcnt(0)\n"
    "s_cmp_lt_u32 s64, s67\n"
    "s_cbranch_scc1 L_r7%=\n"
    "s_call_b64 s[10:11], L_slowx%=\n"
    "s_branch L_r7%=\n"
    "L_e7%=:\n"
    "s_call_b64 s[10:11], L_burst%=\n"
    "s_branch L_f7%=\n"
    "L_u7_0%=:\n"
    "s_call_b64 s[10:11], L_urg%=\n"
    "s_branch L_v7_0%=\n"
    "L_p7%=:\n"
    "s_call_b64 s[10:11], L_slowq%=\n"
    "s_branch L_q7%=\n"
    "L_c8%=:\n"
    "s_waitcnt vmcnt(0)\n"
    "s_cmp_lt_u32 s64, s67\n"
    "s_cbranch_scc1 L_r8%=\n"
    "s_call_b64 s[10:11], L_slowx%=\n"
    "s_branch L_r8%=\n"
    "L_e8%=:\n"
    "s_call_b64 s[10:11], L_burst%=\n"
    "s_branch L_f8%=\n"
    "L_u8_0%=:\n"
    "s_call_b64 s[10:11], L_urg%=\n"
    "s_branch L_v8_0%=\n"
    "L_p8%=:\n"
    "s_call_b64 s[10:11], L_slowq%=\n"
    "s_branch L_q8%=\n"
    "L_c9%=:\n"
    "s_waitcnt vmcnt(0)\n"
    "s_cmp_lt_u32 s64, s67\n"
    "s_cbranch_scc1 L_r9%=\n"
    "s_call_b64 s[10:11], L_slowx%=\n"
    "s_branch L_r9%=\n"
    "L_slowx%=:\n"
    "v_readlane_b32 s74, v42, 0\n"
    "v_readlane_b32 s75, v42, 16\n"
    "v_readlane_b32 s76, v42, 32\n"
    "s_min_u32 s74, s74, s75\n"
    "s_min_u32 s96, s76, s74\n"
    "s_min_u32 s79, s96, s69\n"
    "s_cmp_gt_u32 s79, s64\n"
    "s_cbranch_scc1 L_sx_o%=\n"
    "s_memrealtime s[80:81]\n"
    "L_sx_l%=:\n"
    "global_load_dword v42, v41, s[20:21] sc0 sc1\n"
    "s_waitcnt vmcnt(0) lgkmcnt(0)\n"
    "v_readlane_b32 s74, v42, 0\n"
    "v_readlane_b32 s75, v42, 16\n"
    "v_readlane_b32 s76, v42, 32\n"
    "v_readlane_b32 s78, v42, 48\n"
    "s_min_u32 s74, s74, s75\n"
    "s_min_u32 s96, s76, s74\n"
    "s_min_u32 s79, s96, s69\n"
    "s_cmp_gt_u32 s79, s64\n"
    "s_cbranch_scc1 L_sx_o%=\n"
    "v_readlane_b32 s74, v42, 56\n"
    "s_memrealtime s[76:77]\n"
    "s_lshr_b32 s75, s70, 4\n"
    "s_cmp_eq_u32 s74, s75\n"
    "s_cbranch_scc1 L_sx_t%=\n"
    "s_waitcnt lgkmcnt(0)\n"
    "s_sub_u32 s76, s76, s80\n"
    "s_cmp_gt_u32 s76, 20000000\n"
    "s_cbranch_scc1 L_sx_t%=\n"
    "s_sleep 1\n"
    "s_branch L_sx_l%=\n"
    "L_sx_t%=:\n"
    "s_waitcnt lgkmcnt(0)\n"
    "v_mov_b32 v47, s75\n"
    "v_mov_b32 v15, 4521984\n"
    "global_store_dword v15, v47, s[20:21] sc1\n"
    "v_mov_b32 v41, 4522112\n"
    "v_mov_b32 v13, -1\n"
    "global_store_dword v41, v13, s[20:21] sc0 sc1\n"
    "global_store_dword v41, v13, s[20:21] offset:128 sc0 sc1\n"
    "v_mov_b32 v43, 0x7fc00000\n"
    "v_mov_b32 v38, v41\n"
    "s_waitcnt vmcnt(0)\n"
    "s_branch L_sx_l%=\n"
    "L_sx_o%=:\n"
    "s_mov_b32 s46, s97\n"
    "s_mov_b32 m0, s72\n"
    "s_add_u32 s75, s95, 0x1000\n"
    "buffer_load_dwordx4 v25, s[44:47], s95 offen offset:0 sc0 sc1 lds\n"
    "buffer_load_dwordx4 v25, s[44:47], s95 offen offset:1024 sc0 sc1 lds\n"
    "buffer_load_dwordx4 v25, s[44:47], s95 offen offset:2048 sc0 sc1 lds\n"
    "buffer_load_dwordx4 v25, s[44:47], s95 offen offset:3072 sc0 sc1 lds\n"
    "s_add_u32 m0, s72, 0x1000\n"
    "s_add_u32 s72, s72, 0x8000\n"
    "buffer_load_dwordx4 v25, s[44:47], s75 offen offset:0 sc0 sc1 lds\n"
    "buffer_load_dwordx4 v25, s[44:47], s75 offen offset:1024 sc0 sc1 lds\n"
    "buffer_load_dwordx4 v25, s[44:47], s75 offen offset:2048 sc0 sc1 lds\n"
    "s_and_b32 s72, s72, 0x1ffff\n"
    "s_add_u32 s67, s67, 1\n"
    "s_add_u32 s95, s95, 0x10000\n"
    "global_load_dword v42, v41, s[20:21] sc0 sc1\n"
    "s_cmp_lt_u32 s67, s79\n"
    "s_cbranch_scc1 L_sx2%=\n"
    "s_waitcnt vmcnt(1)\n"
    "s_setpc_b64 s[10:11]\n"
    "L_sx2%=:\n"
    "s_mov_b32 s46, s97\n"
    "s_mov_b32 m0, s72\n"
    "s_add_u32 s75, s95, 0x1000\n"
    "buffer_load_dwordx4 v25, s[44:47], s95 offen offset:0 sc0 sc1 lds\n"
    "buffer_load_dwordx4 v25, s[44:47], s95 offen offset:1024 sc0 sc1 lds\n"
    "buffer_load_dwordx4 v25, s[44:47], s95 offen offset:2048 sc0 sc1 lds\n"
    "buffer_load_dwordx4 v25, s[44:47], s95 offen offset:3072 sc0 sc1 lds\n"
    "s_add_u32 m0, s72, 0x1000\n"
    "s_add_u32 s72, s72, 0x8000\n"
    "buffer_load_dwordx4 v25, s[44:47], s75 offen offset:0 sc0 sc1 lds\n"
    "buffer_load_dwordx4 v25, s[44:47], s75 offen offset:1024 sc0 sc1 lds\n"
    "buffer_load_dwordx4 v25, s[44:47], s75 offen offset:2048 sc0 sc1 lds\n"
    "s_and_b32 s72, s72, 0x1ffff\n"
    "s_add_u32 s67, s67, 1\n"
    "s_add_u32 s95, s95, 0x10000\n"
    "s_waitcnt vmcnt(8)\n"
    "s_setpc_b64 s[10:11]\n"
    "L_slowe%=:\n"
    "v_cmp_lt_u32 vcc, s70, v42\n"
    "s_or_b32 vcc_hi, vcc_hi, 0xffff0000\n"
    "s_cmp_eq_u64 vcc, -1\n"
    "s_cbranch_scc1 L_se_o%=\n"
    "s_memrealtime s[80:81]\n"
    "L_se_l%=:\n"
    "global_load_dword v42, v41, s[20:21] sc0 sc1\n"
    "s_waitcnt vmcnt(0) lgkmcnt(0)\n"
    "v_readlane_b32 s74, v42, 0\n"
    "v_readlane_b32 s75, v42, 16\n"
    "v_readlane_b32 s76, v42, 32\n"
    "v_readlane_b32 s78, v42, 48\n"
    "s_min_u32 s74, s74, s75\n"
    "s_min_u32 s96, s76, s74\n"
    "s_min_u32 s79, s96, s69\n"
    "v_cmp_lt_u32 vcc, s70, v42\n"
    "s_or_b32 vcc_hi, vcc_hi, 0xffff0000\n"
    "s_cmp_eq_u64 vcc, -1\n"
    "s_cbranch_scc1 L_se_o%=\n"
    "v_readlane_b32 s74, v42, 56\n"
    "s_memrealtime s[76:77]\n"
    "s_lshr_b32 s75, s70, 4\n"
    "s_cmp_eq_u32 s74, s75\n"
    "s_cbranch_scc1 L_se_t%=\n"
    "s_waitcnt lgkmcnt(0)\n"
    "s_sub_u32 s76, s76, s80\n"
    "s_cmp_gt_u32 s76, 20000000\n"
    "s_cbranch_scc1 L_se_t%=\n"
    "s_sleep 1\n"
    "s_branch L_se_l%=\n"
    "L_se_t%=:\n"
    "s_waitcnt lgkmcnt(0)\n"
    "v_mov_b32 v47, s75\n"
    "v_mov_b32 v15, 4521984\n"
    "global_store_dword v15, v47, s[20:21] sc1\n"
    "v_mov_b32 v41, 4522112\n"
    "v_mov_b32 v13, -1\n"
    "global_store_dword v41, v13, s[20:21] sc0 sc1\n"
    "global_store_dword v41, v13, s[20:21] offset:128 sc0 sc1\n"
    "v_mov_b32 v43, 0x7fc00000\n"
    "v_mov_b32 v38, v41\n"
    "s_waitcnt vmcnt(0)\n"
    "s_branch L_se_l%=\n"
    "L_se_o%=:\n"
    "s_mov_b32 s46, s97\n"
    "s_mov_b32 m0, s72\n"
    "s_add_u32 s75, s95, 0x1000\n"
    "buffer_load_dwordx4 v25, s[44:47], s95 offen offset:0 sc0 sc1 lds\n"
    "buffer_load_dwordx4 v25, s[44:47], s95 offen offset:1024 sc0 sc1 lds\n"
    "buffer_load_dwordx4 v25, s[44:47], s95 offen offset:2048 sc0 sc1 lds\n"
    "buffer_load_dwordx4 v25, s[44:47], s95 offen offset:3072 sc0 sc1 lds\n"
    "s_add_u32 m0, s72, 0x1000\n"
    "s_add_u32 s72, s72, 0x8000\n"
    "buffer_load_dwordx4 v25, s[44:47], s75 offen offset:0 sc0 sc1 lds\n"
    "buffer_load_dwordx4 v25, s[44:47], s75 offen offset:1024 sc0 sc1 lds\n"
    "buffer_load_dwordx4 v25, s[44:47], s75 offen offset:2048 sc0 sc1 lds\n"
    "s_and_b32 s72, s72, 0x1ffff\n"
    "s_add_u32 s67, s67, 1\n"
    "s_add_u32 s95, s95, 0x10000\n"
    "s_mov_b32 s46, s97\n"
    "s_mov_b32 m0, s72\n"
    "s_add_u32 s75, s95, 0x1000\n"
    "buffer_load_dwordx4 v25, s[44:47], s95 offen offset:0 sc0 sc1 lds\n"
    "buffer_load_dwordx4 v25, s[44:47], s95 offen offset:1024 sc0 sc1 lds\n"
    "buffer_load_dwordx4 v25, s[44:47], s95 offen offset:2048 sc0 sc1 lds\n"
    "buffer_load_dwordx4 v25, s[44:47], s95 offen offset:3072 sc0 sc1 lds\n"
    "s_add_u32 m0, s72, 0x1000\n"
    "s_add_u32 s72, s72, 0x8000\n"
    "buffer_load_dwordx4 v25, s[44:47], s75 offen offset:0 sc0 sc1 lds\n"
    "buffer_load_dwordx4 v25, s[44:47], s75 offen offset:1024 sc0 sc1 lds\n"
    "buffer_load_dwordx4 v25, s[44:47], s75 offen offset:2048 sc0 sc1 lds\n"
    "s_and_b32 s72, s72, 0x1ffff\n"
    "s_add_u32 s67, s67, 1\n"
    "s_add_u32 s95, s95, 0x10000\n"
    "s_setpc_b64 s[10:11]\n"
    "L_urg%=:\n"
    "v_readlane_b32 s74, v42, 0\n"
    "v_readlane_b32 s75, v42, 16\n"
    "v_readlane_b32 s76, v42, 32\n"
    "s_min_u32 s74, s74, s75\n"
    "s_min_u32 s96, s76, s74\n"
    "s_min_u32 s79, s96, s69\n"
    "s_cmp_lt_u32 s67, s79\n"
    "s_cbranch_scc0 L_urg1%=\n"
    "s_mov_b32 s46, s97\n"
    "s_mov_b32 m0, s72\n"
    "s_add_u32 s75, s95, 0x1000\n"
    "buffer_load_dwordx4 v25, s[44:47], s95 offen offset:0 sc0 sc1 lds\n"
    "buffer_load_dwordx4 v25, s[44:47], s95 offen offset:1024 sc0 sc1 lds\n"
    "buffer_load_dwordx4 v25, s[44:47], s95 offen offset:2048 sc0 sc1 lds\n"
    "buffer_load_dwordx4 v25, s[44:47], s95 offen offset:3072 sc0 sc1 lds\n"
    "s_add_u32 m0, s72, 0x1000\n"
    "s_add_u32 s72, s72, 0x8000\n"
    "buffer_load_dwordx4 v25, s[44:47], s75 offen offset:0 sc0 sc1 lds\n"
    "buffer_load_dwordx4 v25, s[44:47], s75 offen offset:1024 sc0 sc1 lds\n"
    "buffer_load_dwordx4 v25, s[44:47], s75 offen offset:2048 sc0 sc1 lds\n"
    "s_and_b32 s72, s72, 0x1ffff\n"
    "s_add_u32 s67, s67, 1\n"
    "s_add_u32 s95, s95, 0x10000\n"
    "s_mov_b32 s46, 0\n"
    "s_mov_b32 m0, 155648\n"
    "L_urg1%=:\n"
    "global_load_dword v42, v38, s[20:21] sc0 sc1\n"
    "s_setpc_b64 s[10:11]\n"
    "L_burst%=:\n"
    "s_mov_b32 s46, s97\n"
    "s_mov_b32 m0, s72\n"
    "s_add_u32 s75, s95, 0x1000\n"
    "buffer_load_dwordx4 v25, s[44:47], s95 offen offset:0 sc0 sc1 lds\n"
    "buffer_load_dwordx4 v25, s[44:47], s95 offen offset:1024 sc0 sc1 lds\n"
    "buffer_load_dwordx4 v25, s[44:47], s95 offen offset:2048 sc0 sc1 lds\n"
    "buffer_load_dwordx4 v25, s[44:47], s95 offen offset:3072 sc0 sc1 lds\n"
    "s_add_u32 m0, s72, 0x1000\n"
    "s_add_u32 s72, s72, 0x8000\n"
    "buffer_load_dwordx4 v25, s[44:47], s75 offen offset:0 sc0 sc1 lds\n"
    "buffer_load_dwordx4 v25, s[44:47], s75 offen offset:1024 sc0 sc1 lds\n"
    "buffer_load_dwordx4 v25, s[44:47], s75 offen offset:2048 sc0 sc1 lds\n"
    "s_and_b32 s72, s72, 0x1ffff\n"
    "s_add_u32 s67, s67, 1\n"
    "s_add_u32 s95, s95, 0x10000\n"
    "s_setpc_b64 s[10:11]\n"
    "L_slowq%=:\n"
    "s_mov_b32 s74, 3\n"
    "L_sq%=:\n"
    "s_waitcnt lgkmcnt(0)\n"
    "s_cmp_ge_u32 s62, s89\n"
    "s_cbranch_scc1 L_sp_o%=\n"
    "s_load_dword s62, s[20:21], s71 glc\n"
    "s_sub_u32 s74, s74, 1\n"
    "s_cbranch_scc0 L_sq%=\n"
    "L_slowp%=:\n"
    "s_memrealtime s[80:81]\n"
    "L_sp_l%=:\n"
    "global_load_dword v42, v41, s[20:21] sc0 sc1\n"
    "s_waitcnt vmcnt(0) lgkmcnt(0)\n"
    "v_readlane_b32 s74, v42, 0\n"
    "v_readlane_b32 s75, v42, 16\n"
    "v_readlane_b32 s76, v42, 32\n"
    "v_readlane_b32 s78, v42, 48\n"
    "s_min_u32 s74, s74, s75\n"
    "s_min_u32 s96, s76, s74\n"
    "s_min_u32 s79, s96, s69\n"
    "s_cmp_ge_u32 s78, s89\n"
    "s_cbranch_scc1 L_sp_o%=\n"
    "v_readlane_b32 s74, v42, 56\n"
    "s_memrealtime s[76:77]\n"
    "s_lshr_b32 s75, s70, 4\n"
    "s_cmp_eq_u32 s74, s75\n"
    "s_cbranch_scc1 L_sp_t%=\n"
    "s_waitcnt lgkmcnt(0)\n"
    "s_sub_u32 s76, s76, s80\n"
    "s_cmp_gt_u32 s76, 20000000\n"
    "s_cbranch_scc1 L_sp_t%=\n"
    "s_sleep 1\n"
    "s_branch L_sp_l%=\n"
    "L_sp_t%=:\n"
    "s_waitcnt lgkmcnt(0)\n"
    "v_mov_b32 v47, s75\n"
    "v_mov_b32 v15, 4521984\n"
    "global_store_dword v15, v47, s[20:21] sc1\n"
    "v_mov_b32 v41, 4522112\n"
    "v_mov_b32 v13, -1\n"
    "global_store_dword v41, v13, s[20:21] sc0 sc1\n"
    "global_store_dword v41, v13, s[20:21] offset:128 sc0 sc1\n"
    "v_mov_b32 v43, 0x7fc00000\n"
    "v_mov_b32 v38, v41\n"
    "s_waitcnt vmcnt(0)\n"
    "s_branch L_sp_l%=\n"
    "L_sp_o%=:\n"
    "s_setpc_b64 s[10:11]\n"
    "L_end%=:\n"
    "s_waitcnt vmcnt(0) lgkmcnt(0)\n"
    :
    : [tid] "v"(threadIdx.x), [bid] "s"(blockIdx.x), [ka] "s"(__builtin_amdgcn_kernarg_segment_ptr())
    :
      "memory", "scc", "vcc", "v12", "v13", "v14", "v15", "v16", "v17", "v18", "v19", "v20", "v21", "v22", "v23", "v24", "v25", "v26", "v27", "v28",
      "v29", "v30", "v31", "v32", "v33", "v34", "v35", "v36", "v37", "v38", "v39", "v40", "v41", "v42", "v43", "v44", "v45", "v46", "v47", "v48", "v49",
      "v50", "v51", "v52", "v53", "v54", "v55", "v56", "v57", "v58", "v59", "v60", "v61", "v62", "v63", "v64", "v65", "v66", "v67", "v68", "v69", "v70",
      "v71", "v72", "v73", "v74", "v75", "v76", "v77", "v78", "v79", "v80", "v81", "v82", "v83", "v84", "v85", "v86", "v87", "v88", "v89", "v90", "v91",
      "v92", "v93", "v94", "v95", "v96", "v97", "v98", "v99", "v100", "v101", "v102", "v103", "v104", "v105", "v106", "v107", "v108", "v109", "v110",
      "v111", "v112", "v113", "v114", "v115", "v116", "v117", "v118", "v119", "v120", "v121", "v122", "v123", "v124", "v125", "v126", "v127", "v128",
      "v129", "v130", "v131", "v132", "v133", "v134", "v135", "v136", "v137", "v138", "v139", "v140", "v141", "v142", "v143", "v144", "v145", "v146",
      "v147", "v148", "v149", "v150", "v151", "v152", "v153", "v154", "v155", "v156", "v157", "v158", "v159", "v160", "v161", "v162", "v163", "v164",
      "v165", "v166", "v167", "v168", "v169", "v170", "v171", "v172", "v173", "v174", "v175", "v176", "v177", "v178", "v179", "v180", "v181", "v182",
      "v183", "v184", "v185", "v186", "v187", "v188", "v189", "v190", "v191", "v192", "v193", "v194", "v195", "v196", "v197", "v198", "v199", "v200",
      "v201", "v202", "v203", "v204", "v205", "v206", "v207", "v208", "v209", "v210", "v211", "v212", "v213", "v214", "v215", "v216", "v217", "v218",
      "v219", "v220", "v221", "v222", "v223", "v224", "v225", "v226", "v227", "v228", "v229", "v230", "v231", "v232", "v233", "v234", "v235", "v236",
      "v237", "v238", "v239", "v240", "v241", "v242", "v243", "v244", "v245", "v246", "v247", "v248", "v249", "v250", "v251", "v252", "v253", "v254",
      "v255", "a0", "a1", "a2", "a3", "a4", "a5", "a6", "a7", "a8", "a9", "a10", "a11", "a12", "a13", "a14", "a15", "a16", "a17", "a18", "a19", "a20",
      "a21", "a22", "a23", "a24", "a25", "a26", "a27", "a28", "a29", "a30", "a31", "a32", "a33", "a34", "a35", "a36", "a37", "a38", "a39", "a40", "a41",
      "a42", "a43", "a44", "a45", "a46", "a47", "a48", "a49", "a50", "a51", "a52", "a53", "a54", "a55", "a56", "a57", "a58", "a59", "a60", "a61", "a62",
      "a63", "a64", "a65", "a66", "a67", "a68", "a69", "a70", "a71", "a72", "a73", "a74", "a75", "a76", "a77", "a78", "a79", "a80", "a81", "a82", "a83",
      "a84", "a85", "a86", "a87", "a88", "a89", "a90", "a91", "a92", "a93", "a94", "a95", "a96", "a97", "a98", "a99", "a100", "a101", "a102", "a103",
      "a104", "a105", "a106", "a107", "a108", "a109", "a110", "a111", "a112", "a113", "a114", "a115", "a116", "a117", "a118", "a119", "a120", "a121",
      "a122", "a123", "a124", "a125", "a126", "a127", "a128", "a129", "a130", "a131", "a132", "a133", "a134", "a135", "a136", "a137", "a138", "a139",
      "a140", "a141", "a142", "a143", "a144", "a145", "a146", "a147", "a148", "a149", "a150", "a151", "a152", "a153", "a154", "a155", "a156", "a157",
      "a158", "a159", "a160", "a161", "a162", "a163", "a164", "a165", "a166", "a167", "a168", "a169", "a170", "a171", "a172", "a173", "a174", "a175",
      "a176", "a177", "a178", "a179", "a180", "a181", "a182", "a183", "a184", "a185", "a186", "a187", "a188", "a189", "a190", "a191", "a192", "a193",
      "a194", "a195", "a196", "a197", "a198", "a199", "a200", "a201", "a202", "a203", "a204", "a205", "a206", "a207", "a208", "a209", "a210", "a211",
      "a212", "a213", "a214", "a215", "a216", "a217", "a218", "a219", "a220", "a221", "a222", "a223", "a224", "a225", "a226", "a227", "a228", "a229",
      "a230", "a231", "a232", "a233", "a234", "a235", "a236", "a237", "a238", "a239", "a240", "a241", "a242", "a243", "a244", "a245", "a246", "a247",
      "a248", "a249", "a250", "a251", "a252", "a253", "a254", "a255", "s10", "s11", "s12", "s13", "s14", "s15", "s16", "s17", "s18", "s19", "s20",
      "s21", "s22", "s23", "s24", "s25", "s26", "s27", "s28", "s29", "s30", "s31", "s34", "s35", "s36", "s37", "s38", "s39", "s40", "s41", "s42", "s43",
      "s44", "s45", "s46", "s47", "s48", "s49", "s50", "s51", "s52", "s53", "s54", "s55", "s56", "s57", "s58", "s59", "s60", "s61", "s62", "s63", "s64",
      "s65", "s66", "s67", "s68", "s69", "s70", "s71", "s72", "s73", "s74", "s75", "s76", "s77", "s78", "s79", "s80", "s81", "s82", "s83", "s84", "s85",
      "s86", "s87", "s88", "s89", "s90", "s91", "s92", "s93", "s94", "s95", "s96", "s97", "s98", "s99");
}

extern "C" void kernel_launch(int rank, int nranks, ncclComm_t comm,
                              hipStream_t stream,
                              void* const* d_in, const int* in_sizes, int n_in,
                              void* d_out, int out_size,
                              const int* in_dtypes, int out_dtype,
                              void* const* d_out_all,
                              void* const* d_ws_all, size_t ws_bytes) {
  if (nranks != NR || (unsigned)rank >= (unsigned)NR || ws_bytes < WS_NEED) return;
  HIP_CHECK(hipFuncSetAttribute((const void*)a2a_gemm, hipFuncAttributeMaxDynamicSharedMemorySize, LDS_BYTES));
  Params p{};
  p.x = (u64)d_in[0]; p.w = (u64)d_in[1]; p.sx = (u64)d_in[2]; p.sw = (u64)d_in[3]; p.out = (u64)d_out;
  p.ws = (u64)d_ws_all[rank]; p.rank = (unsigned)rank; p.pad = 0;
  for (int i = 0; i < NR; ++i) p.peer[i] = (u64)d_ws_all[(rank + i) & 7];
  a2a_gemm<<<256, 256, LDS_BYTES, stream>>>(p);
  HIP_CHECK(hipGetLastError());
}
```
